```python
import numpy as np

kernel_name = "gpu_gemm_a2a_m2048_k2048_n2048_bf16_silu"


ABSMAX_RTOL = 0.008

COMPUTE_DTYPE = "bfloat16"
EPILOGUE = "silu"
IN_QUANT = None
OUT_QUANT = None
IN_DTYPES = {"x": "bfloat16", "w_mat": "bfloat16"}
WORLD_SIZE = 8

def _to_bf16(x):
    u = np.ascontiguousarray(x, dtype=np.float32).view(np.uint32)
    return ((u + 0x7FFF + ((u >> 16) & 1)) & 0xFFFF0000).view(np.float32)

def setup_inputs():

    x = _to_bf16(np.random.randn(2048, 2048).astype(np.float32))
    w_mat = _to_bf16((np.random.randn(2048, 2048) / np.sqrt(2048)).astype(np.float32))
    return {"x": x, "w_mat": w_mat}

def _epilogue(y):
    return (y / (1.0 + np.exp(-np.clip(y, -60.0, 60.0)))).astype(np.float32)

def reference(x, w_mat):
    y = (x @ w_mat).astype(np.float32)
    return _epilogue(y)
```

```cpp
#include <hip/hip_runtime.h>
#include <rccl/rccl.h>
#include <cstdio>
#include <cstdint>

#define HIP_CHECK(expr) do { hipError_t _e = (expr); if (_e != hipSuccess) { \
    fprintf(stderr, "HIP error %s at %s:%d\n", hipGetErrorString(_e), __FILE__, __LINE__); \
    return; } } while (0)

typedef __attribute__((ext_vector_type(8))) __bf16 bf16x8;
typedef __attribute__((ext_vector_type(2))) __bf16 bf16x2;
typedef __attribute__((ext_vector_type(4))) short s16x4;
typedef __attribute__((ext_vector_type(8))) short s16x8;
typedef __attribute__((ext_vector_type(4))) float f32x4;
typedef __attribute__((ext_vector_type(4))) unsigned u32x4;
#define LDS_AS __attribute__((address_space(3)))
#define GLB_AS __attribute__((address_space(1)))

constexpr int NRANK = 8;
constexpr int KDIM = 2048, NDIM = 2048, MLOC = 256, NLOC = 256;
#define NW 4
constexpr int KW = KDIM / NW;
constexpr int KB = KW / 128;
constexpr int NCHUNK = 6 * KB;
constexpr int RED_BYTES = 2 * NW * 2048;
constexpr int RING = (163840 - RED_BYTES) / (NW * 4096);
constexpr int RED_OFF = NW * RING * 4096;
constexpr int LDS_BYTES = RED_OFF + RED_BYTES;
constexpr int NWG = 256;

constexpr int WS_FLAG = 0;
constexpr int WS_CNT = NWG * 128;
constexpr int WS_TMO = 2 * NWG * 128;
constexpr int WS_DBG = 2 * NWG * 128 + 4096;
constexpr unsigned long long DEADLINE_TICKS = 50000000ull;

struct Params {
    const void* x;
    const void* w;
    char* out[NRANK];
    char* ws[NRANK];
    int rank;
    int pad;
};

template <int LOFF, int GOFF>
__device__ __forceinline__ void dma16(unsigned lds_base, unsigned voff, __amdgpu_buffer_rsrc_t srd) {
    unsigned t;
    asm volatile("s_add_u32 m0, %1, %2\n\ts_mov_b32 %0, %3\n\tbuffer_load_dwordx4 %4, %5, %0 offen lds"
                 : "=&s"(t) : "s"(lds_base), "i"(LOFF), "i"(GOFF), "v"(voff), "s"(srd) : "memory", "scc");
}
template <int N> __device__ __forceinline__ void wait_vm() {
    asm volatile("s_waitcnt vmcnt(%0)" :: "i"(N) : "memory");
}
__device__ __forceinline__ void wait_lgkm0() { asm volatile("s_waitcnt lgkmcnt(0)" ::: "memory"); }
__device__ __forceinline__ void lds_barrier() { asm volatile("s_waitcnt lgkmcnt(0)\n\ts_barrier" ::: "memory"); }

constexpr bool is_w(int c) { return c < 3 * KB && (c % 3) != 0; }
constexpr int w_idx(int c) { return 2 * (c / 3) + (c % 3) - 1; }
constexpr int x_phase(int c) { return c < 3 * KB ? 0 : 1 + (c - 3 * KB) / KB; }
constexpr int x_kb(int c) { return c < 3 * KB ? c / 3 : (c - 3 * KB) % KB; }
constexpr bool phase_end(int c) { return c >= 3 * KB - 1 && (c - (3 * KB - 1)) % KB == 0; }
constexpr int cmin(int a, int b) { return a < b ? a : b; }

struct Ctx {
    unsigned lds_base;
    unsigned vx[4], vw;
    __amdgpu_buffer_rsrc_t sx, sw;
    LDS_AS char* ax[4];
    LDS_AS char* aw[2];
    LDS_AS char* ared[2];
    bf16x8 wf[KB * 4][2];
    bf16x8 xf[2][4];
    f32x4 acc[2];
};

template <int C, int I> __device__ __forceinline__ void issue_piece(Ctx& c) {
    if constexpr (C < NCHUNK) {
        constexpr int loff = (C % RING) * 4096 + I * 1024;
        if constexpr (is_w(C)) dma16<loff, (w_idx(C) * 64 + I * 16) * (NDIM * 2)>(c.lds_base, c.vw, c.sw);
        else dma16<loff, x_phase(C) * 16 * (KDIM * 2) + x_kb(C) * 256>(c.lds_base, c.vx[I], c.sx);
    }
}
template <int C> __device__ __forceinline__ void issue_chunk(Ctx& c) {
    issue_piece<C, 0>(c); issue_piece<C, 1>(c); issue_piece<C, 2>(c); issue_piece<C, 3>(c);
}
template <int C> __device__ __forceinline__ void prologue(Ctx& c) {
    if constexpr (C < RING) { issue_chunk<C>(c); prologue<C + 1>(c); }
}
constexpr int xset(int c) { return (c < 3 * KB ? c / 3 : KB + (c - 3 * KB)) & 1; }

template <int C> __device__ __forceinline__ void read_chunk(Ctx& c) {
    constexpr int so = (C % RING) * 4096;
    if constexpr (is_w(C)) {
#pragma unroll
        for (int s = 0; s < 2; ++s)
#pragma unroll
            for (int t = 0; t < 2; ++t) {
                s16x4 lo = __builtin_amdgcn_ds_read_tr16_b64_v4i16((LDS_AS s16x4*)(c.aw[t] + so + s * 2048));
                s16x4 hi = __builtin_amdgcn_ds_read_tr16_b64_v4i16((LDS_AS s16x4*)(c.aw[t] + so + s * 2048 + 256));
                s16x8 v = __builtin_shufflevector(lo, hi, 0, 1, 2, 3, 4, 5, 6, 7);
                c.wf[w_idx(C) * 2 + s][t] = __builtin_bit_cast(bf16x8, v);
            }
    } else {
#pragma unroll
        for (int s = 0; s < 4; ++s) c.xf[xset(C)][s] = *(LDS_AS bf16x8*)(c.ax[s] + so);
    }
}
template <int C, int J> __device__ __forceinline__ void mma_pair(Ctx& c) {
    if constexpr (C >= 0) {
        if constexpr (is_w(C)) {
            if constexpr (J < 2) {
                constexpr int ks = w_idx(C) * 2 + J;
                c.acc[0] = __builtin_amdgcn_mfma_f32_16x16x32_bf16(c.wf[ks][0], c.xf[xset(C)][ks & 3], c.acc[0], 0, 0, 0);
                c.acc[1] = __builtin_amdgcn_mfma_f32_16x16x32_bf16(c.wf[ks][1], c.xf[xset(C)][ks & 3], c.acc[1], 0, 0, 0);
            }
        } else if constexpr (C >= 3 * KB) {
            constexpr int ks = x_kb(C) * 4 + J;
            c.acc[0] = __builtin_amdgcn_mfma_f32_16x16x32_bf16(c.wf[ks][0], c.xf[xset(C)][J], c.acc[0], 0, 0, 0);
            c.acc[1] = __builtin_amdgcn_mfma_f32_16x16x32_bf16(c.wf[ks][1], c.xf[xset(C)][J], c.acc[1], 0, 0, 0);
        }
    }
}
template <int C> __device__ __forceinline__ void finish_phase(Ctx& c) {
    if constexpr (C >= 0 && phase_end(C)) {
        constexpr int buf = (x_phase(C) & 1) * (NW * 2048);
        *(LDS_AS f32x4*)(c.ared[0] + buf) = c.acc[0];
        *(LDS_AS f32x4*)(c.ared[1] + buf) = c.acc[1];
        lds_barrier();
        c.acc[0] = f32x4{0.f, 0.f, 0.f, 0.f};
        c.acc[1] = f32x4{0.f, 0.f, 0.f, 0.f};
    }
}
#define SB() __builtin_amdgcn_sched_barrier(0)
template <int C> __device__ __forceinline__ void step(Ctx& c) {
    if constexpr (C < NCHUNK) {
        constexpr int pend = (C == 0) ? RING - 1 : cmin(C - 2 + RING, NCHUNK - 1) - C;
        wait_vm<4 * pend>();
        read_chunk<C>(c);
        SB();
        if constexpr (C >= 1) {
            issue_piece<C - 1 + RING, 0>(c); mma_pair<C - 1, 0>(c); SB();
            issue_piece<C - 1 + RING, 1>(c); mma_pair<C - 1, 1>(c); SB();
            issue_piece<C - 1 + RING, 2>(c); mma_pair<C - 1, 2>(c); SB();
            issue_piece<C - 1 + RING, 3>(c); mma_pair<C - 1, 3>(c); SB();
            finish_phase<C - 1>(c);
        }
        wait_lgkm0();
        SB();
        step<C + 1>(c);
    } else {
        mma_pair<C - 1, 0>(c); mma_pair<C - 1, 1>(c); mma_pair<C - 1, 2>(c); mma_pair<C - 1, 3>(c);
        finish_phase<C - 1>(c);
    }
}

__device__ __forceinline__ float silu(float y) {
    float z = __builtin_amdgcn_fmed3f(y, -60.0f, 60.0f);
    float e = __builtin_amdgcn_exp2f(z * -1.4426950408889634f);
    return y * __builtin_amdgcn_rcpf(1.0f + e);
}
__device__ __forceinline__ unsigned pack2(float a, float b) {
    bf16x2 v = {(__bf16)a, (__bf16)b};
    return __builtin_bit_cast(unsigned, v);
}

#define STAMP(i)

__global__ __launch_bounds__((NW + 1) * 64) void gemm_a2a(Params p) {
    __shared__ __attribute__((aligned(1024))) char smem[LDS_BYTES];
    const unsigned lane = threadIdx.x & 63;
    const unsigned wid = __builtin_amdgcn_readfirstlane(threadIdx.x >> 6);
    const unsigned b = blockIdx.x;
    const unsigned j = b & 7, idx = b >> 3;
    const unsigned mt = idx & 3, nt = idx >> 2;
    const unsigned rank = (unsigned)p.rank & 7;
    const unsigned dst = (rank + j) & 7;
    LDS_AS char* lds = (LDS_AS char*)smem;

    if (wid < NW) {
        Ctx c;
        c.lds_base = (unsigned)(uintptr_t)lds + wid * (RING * 4096);
        c.sx = __builtin_amdgcn_make_buffer_rsrc((void*)p.x, (short)0, MLOC * KDIM * 2, 0x00020000);
        c.sw = __builtin_amdgcn_make_buffer_rsrc((void*)p.w, (short)0, KDIM * NDIM * 2, 0x00020000);
        const unsigned kbase = wid * KW;
        const unsigned g = lane >> 4, m = lane & 15;
#pragma unroll
        for (int i = 0; i < 4; ++i)
            c.vx[i] = (mt * 64 + 4 * i + g) * (KDIM * 2) + kbase * 2 + ((m ^ g ^ (4 * i)) << 4);
        c.vw = (kbase + (lane >> 2)) * (NDIM * 2) + (dst * NLOC + nt * 32) * 2 + (((lane & 3) ^ ((lane >> 4) & 2)) << 4);
        prologue<0>(c);
        LDS_AS char* ring = lds + wid * (RING * 4096);
#pragma unroll
        for (int s = 0; s < 4; ++s) c.ax[s] = ring + m * 256 + ((g ^ m ^ (4 * s)) << 4);
#pragma unroll
        for (int t = 0; t < 2; ++t) {
            c.aw[t] = ring + (8 * g + (m >> 2)) * 64 + 8 * (lane & 3) + ((t ^ (g & 1)) << 5);
            const unsigned f = ((m & 3) << 1) | ((m >> 2) & 1);
            c.ared[t] = lds + RED_OFF + wid * 2048 + m * 128 + (((4 * t + g) ^ f) << 4);
        }
        c.acc[0] = f32x4{0.f, 0.f, 0.f, 0.f};
        c.acc[1] = f32x4{0.f, 0.f, 0.f, 0.f};
        step<0>(c);
        return;
    }

    char* myws = p.ws[rank];
    STAMP(0);
    unsigned* cnt = (unsigned*)(myws + WS_CNT + b * 128);
    const unsigned seq = __hip_atomic_load(cnt, __ATOMIC_RELAXED, __HIP_MEMORY_SCOPE_AGENT) + 1;
    const unsigned m = lane >> 2, cg = lane & 3;
    const unsigned f = ((m & 3) << 1) | ((m >> 2) & 1);
    LDS_AS char* r0 = lds + RED_OFF + m * 128 + (((2 * cg) ^ f) << 4);
    LDS_AS char* r1 = lds + RED_OFF + m * 128 + (((2 * cg + 1) ^ f) << 4);
    char* optr = p.out[dst] + (size_t)((rank * MLOC + mt * 64 + m) * NLOC + nt * 32 + cg * 8) * 2;
    if (lane == 0) __hip_atomic_store(cnt, seq, __ATOMIC_RELAXED, __HIP_MEMORY_SCOPE_AGENT);
#pragma unroll
    for (int ph = 0; ph < 4; ++ph) {
        lds_barrier();
        STAMP(1 + ph);
        const int buf = (ph & 1) * (NW * 2048);
        f32x4 a = *(LDS_AS f32x4*)(r0 + buf), d = *(LDS_AS f32x4*)(r1 + buf);
#pragma unroll
        for (int w = 1; w < NW; ++w) {
            a += *(LDS_AS f32x4*)(r0 + buf + w * 2048);
            d += *(LDS_AS f32x4*)(r1 + buf + w * 2048);
        }
        u32x4 o = {pack2(silu(a[0]), silu(a[1])), pack2(silu(a[2]), silu(a[3])),
                   pack2(silu(d[0]), silu(d[1])), pack2(silu(d[2]), silu(d[3]))};
        asm volatile("global_store_dwordx4 %0, %1, off sc0 sc1\n\ts_nop 1"
                     :: "v"(optr + ph * 16 * NLOC * 2), "v"(o) : "memory");
    }
    STAMP(5);
    if (j == 0) return;
    asm volatile("s_waitcnt vmcnt(0)" ::: "memory");
    STAMP(6);
    if (lane == 0) {
        __hip_atomic_store((unsigned*)(p.ws[dst] + WS_FLAG + b * 128), seq, __ATOMIC_RELAXED, __HIP_MEMORY_SCOPE_SYSTEM);
        unsigned* flag = (unsigned*)(myws + WS_FLAG + b * 128);
        unsigned* tmo = (unsigned*)(myws + WS_TMO);
        const unsigned long long t0 = __builtin_amdgcn_s_memrealtime();
        while (__hip_atomic_load(flag, __ATOMIC_RELAXED, __HIP_MEMORY_SCOPE_SYSTEM) < seq) {
            if (__hip_atomic_load(tmo, __ATOMIC_RELAXED, __HIP_MEMORY_SCOPE_AGENT) == seq
                || __builtin_amdgcn_s_memrealtime() - t0 > DEADLINE_TICKS) {
                __hip_atomic_store(tmo, seq, __ATOMIC_RELAXED, __HIP_MEMORY_SCOPE_AGENT);
                __hip_atomic_store((unsigned short*)(p.out[rank] + (size_t)(rank * MLOC * NLOC) * 2),
                                   (unsigned short)0x7FC0, __ATOMIC_RELAXED, __HIP_MEMORY_SCOPE_SYSTEM);
                break;
            }
            __builtin_amdgcn_s_sleep(1);
        }
    }
    STAMP(7);
}

extern "C" void kernel_launch(int rank, int nranks, ncclComm_t comm,
                              hipStream_t stream,
                              void* const* d_in, const int* in_sizes, int n_in,
                              void* d_out, int out_size,
                              const int* in_dtypes, int out_dtype,
                              void* const* d_out_all,
                              void* const* d_ws_all, size_t ws_bytes) {
    if (nranks != NRANK || n_in < 2 || (unsigned)rank >= (unsigned)NRANK) return;
    Params p;
    p.x = d_in[0];
    p.w = d_in[1];
    for (int i = 0; i < NRANK; ++i) {
        p.out[i] = (char*)d_out_all[i];
        p.ws[i] = (char*)d_ws_all[i];
    }
    p.rank = rank;
    p.pad = 0;
    gemm_a2a<<<dim3(NWG), dim3((NW + 1) * 64), 0, stream>>>(p);
    HIP_CHECK(hipGetLastError());
}
```

```cpp
#include <hip/hip_runtime.h>
#include <rccl/rccl.h>
#include <cstdio>
#include <cstdint>

#define HIP_CHECK(expr) do { hipError_t _e = (expr); if (_e != hipSuccess) { \
    fprintf(stderr, "HIP error %s at %s:%d\n", hipGetErrorString(_e), __FILE__, __LINE__); \
    return; } } while (0)

typedef __attribute__((ext_vector_type(8))) __bf16 bf16x8;
typedef __attribute__((ext_vector_type(2))) __bf16 bf16x2;
typedef __attribute__((ext_vector_type(4))) short s16x4;
typedef __attribute__((ext_vector_type(8))) short s16x8;
typedef __attribute__((ext_vector_type(4))) float f32x4;
typedef __attribute__((ext_vector_type(4))) unsigned u32x4;
#define LDS_AS __attribute__((address_space(3)))

constexpr int NRANK = 8;
constexpr int KDIM = 2048, NDIM = 2048, MLOC = 256, NLOC = 256;
constexpr int NW = 4;
constexpr int UNIT = 12288;
constexpr int NSLOT = 3;
constexpr int WAVE_LDS = NSLOT * UNIT;
constexpr int JUNK_OFF = NW * WAVE_LDS;
constexpr int LDS_BYTES = JUNK_OFF + NW * 1024;
constexpr int NWG = 256;
#define DLY 4

constexpr int WS_FLAG = 0;
constexpr int WS_CNT = NWG * NW * 128;
constexpr int WS_TMO = 2 * NWG * NW * 128;
constexpr int WS_DBG = WS_TMO + 4096;
constexpr unsigned long long DEADLINE_TICKS = 50000000ull;

struct Params {
    const void* x;
    const void* w;
    unsigned rank;
    unsigned pad;
    char* myws;
    char* myout;
    char* out[NRANK];
    char* ws[NRANK];
};

template <int LOFF, int GOFF>
__device__ __forceinline__ void dma16(unsigned lds_base, unsigned voff, unsigned sbase, __amdgpu_buffer_rsrc_t srd) {
    unsigned t;
    asm volatile("s_add_u32 m0, %1, %2\n\ts_add_u32 %0, %6, %3\n\tbuffer_load_dwordx4 %4, %5, %0 offen lds"
                 : "=&s"(t) : "s"(lds_base), "i"(LOFF), "i"(GOFF), "v"(voff), "s"(srd), "s"(sbase) : "memory", "scc");
}
template <int N> __device__ __forceinline__ void wait_vm() {
    asm volatile("s_waitcnt vmcnt(%0)" :: "i"(N) : "memory");
}
__device__ __forceinline__ void lds_barrier() { asm volatile("s_waitcnt lgkmcnt(0)\n\ts_barrier" ::: "memory"); }
#define SB() __builtin_amdgcn_sched_barrier(0)

struct Ctx {
    unsigned lds_base;
    unsigned kbase, leader;
    unsigned vx[2], vw;
    __amdgpu_buffer_rsrc_t sx, sw;
    LDS_AS char* ax[2];
    LDS_AS char* aw[4];
    bf16x8 wf[2][4];
    bf16x8 xf[2][2];
    f32x4 acc[2][4];
};
#define STAMP(i)

template <int U> __device__ __forceinline__ unsigned block_of(const Ctx& c) {
    return c.kbase ^ (c.leader ? 16u * (U & 1) + (U >> 1) : 4u * U);
}
template <int U, int P> __device__ __forceinline__ void piece(Ctx& c, unsigned kb) {
    constexpr int loff = (U % NSLOT) * UNIT + P * 1024;
    if constexpr (P < 8) dma16<loff, ((P >> 1) * 16 + (P & 1) * 4) * (NDIM * 2)>(c.lds_base, c.vw, kb << 18, c.sw);
    else dma16<loff, (P - 8) * 8 * (KDIM * 2)>(c.lds_base, c.vx[P & 1], kb << 7, c.sx);
}
template <int U> __device__ __forceinline__ void issue_unit(Ctx& c) {
    const unsigned kb = block_of<U>(c);
    piece<U, 8>(c, kb); piece<U, 0>(c, kb); piece<U, 1>(c, kb); piece<U, 9>(c, kb); piece<U, 2>(c, kb); piece<U, 3>(c, kb);
    piece<U, 10>(c, kb); piece<U, 4>(c, kb); piece<U, 5>(c, kb); piece<U, 11>(c, kb); piece<U, 6>(c, kb); piece<U, 7>(c, kb);
}
template <int U> __device__ __forceinline__ void warm_unit(Ctx& c, unsigned junk) {
    const unsigned kb = block_of<U>(c);
    if (c.leader && kb >= 4) {
        dma16<0, 32 * (KDIM * 2)>(junk, c.vx[0], kb << 7, c.sx);
        dma16<0, 40 * (KDIM * 2)>(junk, c.vx[0], kb << 7, c.sx);
        dma16<0, 48 * (KDIM * 2)>(junk, c.vx[0], kb << 7, c.sx);
        dma16<0, 56 * (KDIM * 2)>(junk, c.vx[0], kb << 7, c.sx);
    }
}
template <int U> __device__ __forceinline__ void read_unit(Ctx& c) {
    constexpr int so = (U % NSLOT) * UNIT;
#pragma unroll
    for (int s = 0; s < 2; ++s) {
#pragma unroll
        for (int h = 0; h < 2; ++h) c.xf[s][h] = *(LDS_AS bf16x8*)(c.ax[s] + so + h * 2048);
#pragma unroll
        for (int t = 0; t < 4; ++t) {
            s16x4 lo = __builtin_amdgcn_ds_read_tr16_b64_v4i16((LDS_AS s16x4*)(c.aw[t] + so + s * 4096));
            s16x4 hi = __builtin_amdgcn_ds_read_tr16_b64_v4i16((LDS_AS s16x4*)(c.aw[t] + so + s * 4096 + 1024));
            c.wf[s][t] = __builtin_bit_cast(bf16x8, __builtin_shufflevector(lo, hi, 0, 1, 2, 3, 4, 5, 6, 7));
        }
    }
}
template <int S, int T> __device__ __forceinline__ void mma2(Ctx& c) {
    c.acc[0][T] = __builtin_amdgcn_mfma_f32_16x16x32_bf16(c.wf[S][T], c.xf[S][0], c.acc[0][T], 0, 0, 0);
    c.acc[1][T] = __builtin_amdgcn_mfma_f32_16x16x32_bf16(c.wf[S][T], c.xf[S][1], c.acc[1][T], 0, 0, 0);
}
template <int U> __device__ __forceinline__ void step(Ctx& c) {
    if constexpr (U < 8) {
        wait_vm<(U < 7) ? 12 : 0>();
        STAMP(3 + U);
        read_unit<U>(c);
        SB();
        if constexpr (U + 2 < 8) {
            const unsigned kb = block_of<U + 2>(c);
            piece<U + 2, 8>(c, kb); piece<U + 2, 0>(c, kb); piece<U + 2, 1>(c, kb); piece<U + 2, 9>(c, kb); SB();
            mma2<0, 0>(c); SB(); piece<U + 2, 2>(c, kb); SB();
            mma2<0, 1>(c); SB(); piece<U + 2, 3>(c, kb); SB();
            mma2<0, 2>(c); SB(); piece<U + 2, 10>(c, kb); SB();
            mma2<0, 3>(c); SB(); piece<U + 2, 4>(c, kb); SB();
            mma2<1, 0>(c); SB(); piece<U + 2, 5>(c, kb); SB();
            mma2<1, 1>(c); SB(); piece<U + 2, 11>(c, kb); SB();
            mma2<1, 2>(c); SB(); piece<U + 2, 6>(c, kb); SB();
            mma2<1, 3>(c); SB(); piece<U + 2, 7>(c, kb); SB();
        } else {
            mma2<0, 0>(c); mma2<0, 1>(c); mma2<0, 2>(c); mma2<0, 3>(c);
            mma2<1, 0>(c); mma2<1, 1>(c); mma2<1, 2>(c); mma2<1, 3>(c);
            SB();
        }
        step<U + 1>(c);
    }
}

__device__ __forceinline__ float silu(float y) {
    float z = __builtin_amdgcn_fmed3f(y, -60.0f, 60.0f);
    float e = __builtin_amdgcn_exp2f(z * -1.4426950408889634f);
    return y * __builtin_amdgcn_rcpf(1.0f + e);
}
__device__ __forceinline__ unsigned pack2(float a, float b) {
    bf16x2 v = {(__bf16)a, (__bf16)b};
    return __builtin_bit_cast(unsigned, v);
}

__global__ __launch_bounds__(NW * 64) void gemm_a2a(Params p) {
    __shared__ __attribute__((aligned(1024))) char smem[LDS_BYTES];
    Ctx c;
    STAMP(0);
    const unsigned lane = threadIdx.x & 63;
    const unsigned wid = __builtin_amdgcn_readfirstlane(threadIdx.x >> 6);
    const unsigned b = blockIdx.x;
    const unsigned j = b & 7, mt = (b >> 3) & 7, nt = b >> 6;
    const unsigned d = (j & 1) * 4 + nt;
    const unsigned sub = (j >> 1) ^ nt;
    const unsigned rank = p.rank & 7;
    const unsigned dst = (rank + d) & 7;
    LDS_AS char* lds = (LDS_AS char*)smem;

    c.lds_base = (unsigned)(uintptr_t)lds + wid * WAVE_LDS;
    c.sx = __builtin_amdgcn_make_buffer_rsrc((void*)p.x, (short)0, MLOC * KDIM * 2, 0x00020000);
    c.sw = __builtin_amdgcn_make_buffer_rsrc((void*)p.w, (short)0, KDIM * NDIM * 2, 0x00020000);
    c.leader = (mt & 1) ^ 1;
    const unsigned cs = (mt >> 1) ^ nt;
    c.kbase = c.leader ? 4 * (wid + 1) + cs : (wid ^ cs);
    const unsigned g = lane >> 4, m = lane & 15;
    c.vx[0] = (mt * 32 + (lane >> 3)) * (KDIM * 2) + (((lane & 7) ^ g) << 4);
    c.vx[1] = c.vx[0] ^ 64;
    c.vw = (((lane >> 3) & 3) + (lane >> 5) * 8) * (NDIM * 2) + (dst * NLOC + sub * 64) * 2 + (((lane & 7) ^ (g << 1)) << 4);
    const unsigned junk = (unsigned)(uintptr_t)lds + JUNK_OFF + wid * 1024;
    for (unsigned i = 0; i < wid; ++i) __builtin_amdgcn_s_sleep(DLY);
    STAMP(1);
    warm_unit<0>(c, junk);
    issue_unit<0>(c);
    unsigned* cnt = (unsigned*)(p.myws + WS_CNT + (b * NW + wid) * 128);
    unsigned seq;
    asm volatile("global_load_dword %0, %1, %2 sc1" : "=v"(seq) : "v"(0u), "s"(cnt) : "memory");
    __builtin_amdgcn_s_barrier();
    warm_unit<1>(c, junk);
    issue_unit<1>(c);
    STAMP(2);

    LDS_AS char* ring = lds + wid * WAVE_LDS;
#pragma unroll
    for (int s = 0; s < 2; ++s) c.ax[s] = ring + 8192 + m * 128 + (((4 * s + g) ^ (m >> 1)) << 4);
    const unsigned q = m >> 2;
#pragma unroll
    for (int t = 0; t < 4; ++t)
        c.aw[t] = ring + (g >> 1) * 2048 + (4 * (g & 1) + q) * 128 + ((t ^ (2 * (g & 1) + (q >> 1))) << 5) + 8 * (lane & 3);
#pragma unroll
    for (int h = 0; h < 2; ++h)
#pragma unroll
        for (int t = 0; t < 4; ++t) c.acc[h][t] = f32x4{0.f, 0.f, 0.f, 0.f};
    step<0>(c);
    asm volatile("" : "+v"(seq));
    seq += 1;

    {
        LDS_AS char* pw = ring + 2 * UNIT + m * 256;
#pragma unroll
        for (int h = 0; h < 2; ++h)
#pragma unroll
            for (int t = 0; t < 4; ++t)
                *(LDS_AS f32x4*)(pw + h * 4096 + (((4 * t + g) ^ (m & 7)) << 4)) = c.acc[h][t];
    }
    const unsigned row = wid * 8 + (lane >> 3), cg = lane & 7;
    char* optr = p.out[d] + (size_t)((rank * MLOC + mt * 32 + row) * NLOC + sub * 64 + cg * 8) * 2;
    LDS_AS char* r0 = lds + 2 * UNIT + row * 256 + (((2 * cg) ^ (row & 7)) << 4);
    LDS_AS char* r1 = lds + 2 * UNIT + row * 256 + (((2 * cg + 1) ^ (row & 7)) << 4);
    lds_barrier();
    STAMP(11);
    f32x4 a[NW], e[NW];
#pragma unroll
    for (int w = 0; w < NW; ++w) {
        a[w] = *(LDS_AS f32x4*)(r0 + w * WAVE_LDS);
        e[w] = *(LDS_AS f32x4*)(r1 + w * WAVE_LDS);
    }
    const f32x4 ya = (a[0] + a[1]) + (a[2] + a[3]), ye = (e[0] + e[1]) + (e[2] + e[3]);
    u32x4 o = {pack2(silu(ya[0]), silu(ya[1])), pack2(silu(ya[2]), silu(ya[3])),
               pack2(silu(ye[0]), silu(ye[1])), pack2(silu(ye[2]), silu(ye[3]))};
    asm volatile("global_store_dwordx4 %0, %1, off sc0 sc1\n\ts_nop 1" :: "v"(optr), "v"(o) : "memory");
    STAMP(12);
    if (lane == 0) __hip_atomic_store(cnt, seq, __ATOMIC_RELAXED, __HIP_MEMORY_SCOPE_AGENT);
    if (d != 0) {
        unsigned* rflag = (unsigned*)(p.ws[d] + WS_FLAG + (b * NW + wid) * 128);
        asm volatile("s_waitcnt vmcnt(0)" ::: "memory");
        STAMP(13);
        if (lane == 0) {
            __hip_atomic_store(rflag, seq, __ATOMIC_RELAXED, __HIP_MEMORY_SCOPE_SYSTEM);
            unsigned* flag = (unsigned*)(p.myws + WS_FLAG + (b * NW + wid) * 128);
            unsigned* tmo = (unsigned*)(p.myws + WS_TMO);
            const unsigned long long t0 = __builtin_amdgcn_s_memrealtime();
            unsigned spins = 0;
            while (__hip_atomic_load(flag, __ATOMIC_RELAXED, __HIP_MEMORY_SCOPE_SYSTEM) < seq) {
                if ((++spins & 63) == 0
                    && (__hip_atomic_load(tmo, __ATOMIC_RELAXED, __HIP_MEMORY_SCOPE_AGENT) == seq
                        || __builtin_amdgcn_s_memrealtime() - t0 > DEADLINE_TICKS)) {
                    __hip_atomic_store(tmo, seq, __ATOMIC_RELAXED, __HIP_MEMORY_SCOPE_AGENT);
                    __hip_atomic_store((unsigned short*)(p.myout + (size_t)(rank * MLOC * NLOC) * 2),
                                       (unsigned short)0x7FC0, __ATOMIC_RELAXED, __HIP_MEMORY_SCOPE_SYSTEM);
                    break;
                }
            }
        }
        STAMP(14);
    }
}

extern "C" void kernel_launch(int rank, int nranks, ncclComm_t comm,
                              hipStream_t stream,
                              void* const* d_in, const int* in_sizes, int n_in,
                              void* d_out, int out_size,
                              const int* in_dtypes, int out_dtype,
                              void* const* d_out_all,
                              void* const* d_ws_all, size_t ws_bytes) {
    if (nranks != NRANK || n_in < 2 || (unsigned)rank >= (unsigned)NRANK) return;
    Params p;
    p.x = d_in[0];
    p.w = d_in[1];
    p.rank = (unsigned)rank;
    p.pad = 0;
    p.myws = (char*)d_ws_all[rank];
    p.myout = (char*)d_out;
    for (int i = 0; i < NRANK; ++i) {
        p.out[i] = (char*)d_out_all[(rank + i) % NRANK];
        p.ws[i] = (char*)d_ws_all[(rank + i) % NRANK];
    }
    gemm_a2a<<<dim3(NWG), dim3(NW * 64), 0, stream>>>(p);
    HIP_CHECK(hipGetLastError());
}
```

```cpp
#include <hip/hip_runtime.h>
#include <rccl/rccl.h>
#include <cstdio>
#include <cstdint>

#define HIP_CHECK(expr) do { hipError_t _e = (expr); if (_e != hipSuccess) { \
    fprintf(stderr, "HIP error %s at %s:%d\n", hipGetErrorString(_e), __FILE__, __LINE__); \
    return; } } while (0)

typedef __attribute__((ext_vector_type(8))) __bf16 bf16x8;
typedef __attribute__((ext_vector_type(2))) __bf16 bf16x2;
typedef __attribute__((ext_vector_type(4))) short s16x4;
typedef __attribute__((ext_vector_type(4))) float f32x4;
typedef __attribute__((ext_vector_type(4))) unsigned u32x4;
typedef __attribute__((ext_vector_type(2))) unsigned u32x2;
#define LDS_AS __attribute__((address_space(3)))
#define GLB_AS __attribute__((address_space(1)))

constexpr int NRANK = 8;
constexpr int KDIM = 2048, NDIM = 2048, MLOC = 256, NLOC = 256;
constexpr int NW = 4;
constexpr int UNIT = 12288;
constexpr int NSLOT = 3;
constexpr int WAVE_LDS = NSLOT * UNIT;
constexpr int JUNK_OFF = NW * WAVE_LDS;
constexpr int LDS_BYTES = JUNK_OFF + 4096;
constexpr int NWG = 256;
#define DLY 4
#define LDLY 0
#define FST 1
#define PROTO 1

constexpr int WS_FLAG = 0;
constexpr int WS_CNT = NWG * NW * 128;
constexpr int WS_TMO = 2 * NWG * NW * 128;
constexpr int WS_DBG = WS_TMO + 4096;
constexpr unsigned long long DEADLINE_TICKS = 50000000ull;
constexpr unsigned SENTINEL = 0xFF81FF81u;

struct Params {
    unsigned srdx[4];
    unsigned srdw[4];
    char* myws;
    char* myout;
    unsigned rank;
    unsigned pad[3];
    char* out[NRANK];
    char* ws[NRANK];
};

template <int LOFF, int GOFF>
__device__ __forceinline__ void dma16(unsigned lds_base, unsigned voff, unsigned sbase, u32x4 srd) {
    unsigned t;
    asm volatile("s_add_u32 m0, %1, %2\n\ts_add_u32 %0, %6, %3\n\tbuffer_load_dwordx4 %4, %5, %0 offen lds"
                 : "=&s"(t) : "s"(lds_base), "i"(LOFF), "i"(GOFF), "v"(voff), "s"(srd), "s"(sbase) : "memory", "scc");
}
template <int N> __device__ __forceinline__ void wait_vm() {
    asm volatile("s_waitcnt vmcnt(%0)" :: "i"(N) : "memory");
}
__device__ __forceinline__ void lds_barrier() { asm volatile("s_waitcnt lgkmcnt(0)\n\ts_barrier" ::: "memory"); }
__device__ __forceinline__ void store16_sys(GLB_AS char* ptr, u32x4 v) {
    asm volatile("global_store_dwordx4 %0, %1, off sc0 sc1\n\ts_nop 1" :: "v"(ptr), "v"(v) : "memory");
}
#define SB() __builtin_amdgcn_sched_barrier(0)
__device__ __forceinline__ unsigned ssel(unsigned cond, unsigned a, unsigned b) {
    unsigned r;
    asm("s_cmp_lg_u32 %1, 0\n\ts_cselect_b32 %0, %2, %3" : "=s"(r) : "s"(cond), "s"(a), "s"(b) : "scc");
    return r;
}

struct Ctx {
    unsigned lds_base;
    unsigned kbase, kfirst, lmask;
    unsigned vx[2], vw;
    u32x4 sx, sw;
    LDS_AS char* ax[2][2];
    LDS_AS char* aw[4];
    bf16x8 wf[2][4];
    bf16x8 xf[2][2];
    f32x4 acc[2][4];
};
#define STAMP(i)
#define LSTAMP(i)

template <int U> __device__ __forceinline__ unsigned block_of(const Ctx& c) {
    constexpr unsigned tl = 18u * (U & 1) ^ (U >> 1), tf = (U < 2 && FST == 2) ? 2u * U : 4u * U;
    return ((U < 2 && FST == 2) ? c.kfirst : c.kbase) ^ tf ^ ((tl ^ tf) & c.lmask);
}
template <int U, int I> __device__ __forceinline__ void piece(Ctx& c, unsigned kb) {
    constexpr int so = (U % NSLOT) * UNIT;
    if constexpr (I < 4) dma16<so + 8192 + I * 1024, I * 8 * (KDIM * 2)>(c.lds_base, c.vx[I & 1], kb << 7, c.sx);
    else dma16<so + (I - 4) * 1024, (((I - 4) >> 1) * 16 + (I & 1) * 4) * (NDIM * 2)>(c.lds_base, c.vw, kb << 18, c.sw);
}
template <int U> __device__ __forceinline__ void issue_x(Ctx& c, unsigned kb) {
    piece<U, 0>(c, kb); piece<U, 1>(c, kb); piece<U, 2>(c, kb); piece<U, 3>(c, kb);
}
template <int U> __device__ __forceinline__ void issue_w(Ctx& c, unsigned kb) {
    piece<U, 4>(c, kb); piece<U, 5>(c, kb); piece<U, 6>(c, kb); piece<U, 7>(c, kb);
    piece<U, 8>(c, kb); piece<U, 9>(c, kb); piece<U, 10>(c, kb); piece<U, 11>(c, kb);
}
__device__ __forceinline__ void issue_x_at(Ctx& c, unsigned base, unsigned soff) {
    dma16<0, 0>(base, c.vx[0], soff, c.sx);
    dma16<1024, 8 * (KDIM * 2)>(base, c.vx[1], soff, c.sx);
    dma16<2048, 16 * (KDIM * 2)>(base, c.vx[0], soff, c.sx);
    dma16<3072, 24 * (KDIM * 2)>(base, c.vx[1], soff, c.sx);
}
template <int U, int S> __device__ __forceinline__ void read_x(Ctx& c) {
    constexpr int so = (U % NSLOT) * UNIT;
    c.xf[S][0] = *(LDS_AS bf16x8*)(c.ax[S][0] + so);
    c.xf[S][1] = *(LDS_AS bf16x8*)(c.ax[S][1] + so);
}
template <int U, int S, int T> __device__ __forceinline__ void read_w1(Ctx& c) {
    constexpr int so = (U % NSLOT) * UNIT;
    s16x4 lo = __builtin_amdgcn_ds_read_tr16_b64_v4i16((LDS_AS s16x4*)(c.aw[T] + so + S * 4096));
    s16x4 hi = __builtin_amdgcn_ds_read_tr16_b64_v4i16((LDS_AS s16x4*)(c.aw[T] + so + S * 4096 + 1024));
    c.wf[S][T] = __builtin_bit_cast(bf16x8, __builtin_shufflevector(lo, hi, 0, 1, 2, 3, 4, 5, 6, 7));
}
template <int U, int S> __device__ __forceinline__ void read_w(Ctx& c) {
    read_w1<U, S, 2>(c); read_w1<U, S, 3>(c); read_w1<U, S, 0>(c); read_w1<U, S, 1>(c);
}
template <int S, int H, int T> __device__ __forceinline__ void mma(Ctx& c) {
    c.acc[H][T] = __builtin_amdgcn_mfma_f32_16x16x32_bf16(c.wf[S][T], c.xf[S][H], c.acc[H][T], 0, 0, 0);
}
template <int S, int T> __device__ __forceinline__ void mma2(Ctx& c) { mma<S, 0, T>(c); mma<S, 1, T>(c); }

template <int U> __device__ __forceinline__ void step(Ctx& c) {
    LSTAMP(3 + U);
    const unsigned kb = block_of<U + 2>(c);
    piece<U + 2, 0>(c, kb); read_x<U, 0>(c); read_w1<U, 0, 2>(c); SB();
    piece<U + 2, 1>(c, kb); read_w1<U, 0, 3>(c); read_w1<U, 0, 0>(c); SB();
    piece<U + 2, 2>(c, kb); read_w1<U, 0, 1>(c); read_x<U, 1>(c); SB();
    piece<U + 2, 3>(c, kb); read_w1<U, 1, 2>(c); read_w1<U, 1, 3>(c); SB();
    piece<U + 2, 4>(c, kb); read_w1<U, 1, 0>(c); read_w1<U, 1, 1>(c); SB();
    mma2<0, 2>(c); SB(); piece<U + 2, 5>(c, kb); SB();
    mma2<0, 3>(c); SB(); piece<U + 2, 6>(c, kb); SB();
    mma2<0, 0>(c); SB(); piece<U + 2, 7>(c, kb); SB();
    mma2<0, 1>(c); SB(); piece<U + 2, 8>(c, kb); SB();
    mma2<1, 2>(c); SB(); piece<U + 2, 9>(c, kb); SB();
    mma2<1, 3>(c); SB(); piece<U + 2, 10>(c, kb); SB();
    mma2<1, 0>(c); SB(); piece<U + 2, 11>(c, kb); SB();
    mma2<1, 1>(c); SB();
}

__device__ __forceinline__ float silu(float y) {
    float z = __builtin_amdgcn_fmed3f(y, -60.0f, 60.0f);
    float e = __builtin_amdgcn_exp2f(z * -1.4426950408889634f);
    return y * __builtin_amdgcn_rcpf(1.0f + e);
}
__device__ __forceinline__ unsigned pack2(float a, float b) {
    bf16x2 v = {(__bf16)a, (__bf16)b};
    return __builtin_bit_cast(unsigned, v);
}
__device__ __forceinline__ bool expired(const Params& p, unsigned seq, unsigned long long t0) {
    unsigned* tmo = (unsigned*)(p.myws + WS_TMO);
    if (__hip_atomic_load(tmo, __ATOMIC_RELAXED, __HIP_MEMORY_SCOPE_AGENT) == seq
        || __builtin_amdgcn_s_memrealtime() - t0 > DEADLINE_TICKS) {
        __hip_atomic_store(tmo, seq, __ATOMIC_RELAXED, __HIP_MEMORY_SCOPE_AGENT);
        __hip_atomic_store((unsigned short*)(p.myout + (size_t)((p.rank & 7) * MLOC * NLOC) * 2),
                           (unsigned short)0x7FC0, __ATOMIC_RELAXED, __HIP_MEMORY_SCOPE_SYSTEM);
        return true;
    }
    return false;
}

__global__ __launch_bounds__(512) void gemm_a2a(Params p) {
    __shared__ __attribute__((aligned(1024))) char smem[LDS_BYTES];
    Ctx c;
    STAMP(0);
    const unsigned b = blockIdx.x;
    const unsigned j = b & 7, mt = (b >> 3) & 7, nt = (b >> 6) & 3;
    const unsigned peer = (j & 1) * 4 + nt;
    u32x4 kws;
    unsigned rank;
    GLB_AS char* outd;
    {
        const void* ka = (const void*)__builtin_amdgcn_kernarg_segment_ptr();
        asm volatile("s_load_dwordx4 %0, %5, 0x0\n\ts_load_dwordx4 %1, %5, 0x10\n\ts_load_dwordx4 %2, %5, 0x20\n\t"
                     "s_load_dword %3, %5, 0x30\n\ts_load_dwordx2 %4, %5, %6 offset:0x40"
                     : "=&s"(c.sx), "=&s"(c.sw), "=&s"(kws), "=&s"(rank), "=&s"(outd) : "s"(ka), "s"(peer * 8));
        SB();
    }
    const unsigned lane = threadIdx.x & 63;
    const unsigned wid = __builtin_amdgcn_readfirstlane(threadIdx.x >> 6);
    const unsigned sub = (j >> 1) ^ nt;
    LDS_AS char* lds = (LDS_AS char*)smem;
    c.lds_base = (unsigned)(uintptr_t)lds + wid * WAVE_LDS;
    c.lmask = (mt & 1) - 1;
    const unsigned cs = (mt >> 1) ^ nt;
    const unsigned kl = 4 * (wid + FST) + (cs ^ ((wid + FST) >> 1));
    c.kbase = ssel(c.lmask, kl, wid ^ cs);
    c.kfirst = ssel(c.lmask, kl, (wid & 1) * 4 + (cs ^ (wid >> 1)));
    const unsigned g = lane >> 4, m = lane & 15;
    c.vx[0] = (mt * 32 + (lane >> 3)) * (KDIM * 2) + (((lane & 7) ^ g) << 4);
    c.vx[1] = c.vx[0] ^ 64;
    c.vw = (((lane >> 3) & 3) + (lane >> 5) * 8) * (NDIM * 2) + (peer * 4 + sub) * 128 + (((lane & 7) ^ (g << 1)) << 4);
    const unsigned junk = (unsigned)(uintptr_t)lds + JUNK_OFF;
    const unsigned kb0 = block_of<0>(c), kb1 = block_of<1>(c);
    unsigned xb0 = ssel(c.lmask, junk, c.lds_base + 8192), xs0 = (kb0 << 7) + (32 * (KDIM * 2) & c.lmask);
    asm volatile("" : "+v"(c.vx[0]), "+v"(c.vx[1]), "+v"(c.vw), "+s"(xb0), "+s"(xs0));
    if (wid & 1) __builtin_amdgcn_s_sleep(DLY);
    if (wid & 2) __builtin_amdgcn_s_sleep(2 * DLY);
    if (LDLY && c.lmask) __builtin_amdgcn_s_sleep(LDLY);
    asm volatile("s_waitcnt lgkmcnt(0)" : "+s"(c.sx), "+s"(c.sw), "+s"(kws), "+s"(rank), "+s"(outd));
    LSTAMP(1);
    issue_x_at(c, xb0, xs0);
    issue_w<0>(c, kb0);
    __builtin_amdgcn_s_barrier();
    if (c.lmask) {
        issue_x<0>(c, kb0);
        if (kb1 >= 4 * FST) issue_x_at(c, junk, (kb1 << 7) + 32 * (KDIM * 2));
    }
    rank &= 7;
    GLB_AS char* const myws = (GLB_AS char*)(((unsigned long long)kws[1] << 32) | kws[0]);
    GLB_AS char* const myout = (GLB_AS char*)(((unsigned long long)kws[3] << 32) | kws[2]);
    const unsigned arm = wid * 16 + (j >> 1) * 4 + (mt >> 1);
    const bool armer = ((peer ^ rank) | c.lmask) == 0 && (arm >> 3) != rank;
    if (__builtin_expect(armer, 0)) {
        GLB_AS char* fp = myout + (size_t)arm * (32 * NLOC * 2) + lane * 16;
        const u32x4 sv = {SENTINEL, SENTINEL, SENTINEL, SENTINEL};
#pragma unroll
        for (int i = 0; i < 16; ++i) store16_sys(fp + i * 1024, sv);
    }
    GLB_AS unsigned* cnt = (GLB_AS unsigned*)(myws + WS_CNT + (b * NW + wid) * 128);
    unsigned seq;
    asm volatile("global_load_dword %0, %1, %2 sc1" : "=v"(seq) : "v"(0u), "s"(cnt) : "memory");
    LDS_AS char* ring = lds + wid * WAVE_LDS;
    const unsigned q = m >> 2;
    piece<1, 4>(c, kb1); piece<1, 5>(c, kb1); SB();
#pragma unroll
    for (int h = 0; h < 2; ++h) c.ax[0][h] = ring + 8192 + (h ^ (wid & 1)) * 2048 + m * 128 + ((g ^ (m >> 1)) << 4);
    asm volatile("" : "+v"(c.ax[0][0]), "+v"(c.ax[0][1]));
    piece<1, 6>(c, kb1); SB();
#pragma unroll
    for (int h = 0; h < 2; ++h) c.ax[1][h] = (LDS_AS char*)((unsigned)(uintptr_t)c.ax[0][h] ^ 64);
    c.aw[0] = ring + (g >> 1) * 2048 + (4 * (g & 1) + q) * 128 + 8 * (lane & 3);
    asm volatile("" : "+v"(c.ax[1][0]), "+v"(c.ax[1][1]), "+v"(c.aw[0]));
    piece<1, 7>(c, kb1); SB();
    c.aw[0] += ((wid & 2) ^ (2 * (g & 1) + (q >> 1))) << 5;
#pragma unroll
    for (int t = 1; t < 4; ++t) c.aw[t] = (LDS_AS char*)((unsigned)(uintptr_t)c.aw[0] ^ (t << 5));
    asm volatile("" : "+v"(c.aw[0]), "+v"(c.aw[1]), "+v"(c.aw[2]), "+v"(c.aw[3]));
    piece<1, 8>(c, kb1); piece<1, 9>(c, kb1); piece<1, 10>(c, kb1); piece<1, 11>(c, kb1);
    issue_x<1>(c, kb1);
    LSTAMP(2);
#pragma unroll
    for (int h = 0; h < 2; ++h)
#pragma unroll
        for (int t = 0; t < 4; ++t) c.acc[h][t] = f32x4{0.f, 0.f, 0.f, 0.f};

    asm volatile("s_waitcnt vmcnt(12)" : "+v"(seq) :: "memory");
    seq += 1;
    if (__builtin_expect(armer, 0) && lane == 0) {
        __hip_atomic_store((unsigned*)(p.ws[(arm >> 3) & 7] + WS_FLAG + (rank * 8 + (arm & 7)) * 128), seq, __ATOMIC_RELAXED, __HIP_MEMORY_SCOPE_SYSTEM);
        __hip_atomic_store((GLB_AS unsigned*)(myws + WS_FLAG + arm * 128 + 4), seq, __ATOMIC_RELAXED, __HIP_MEMORY_SCOPE_SYSTEM);
    }
    step<0>(c); wait_vm<12>();
    step<1>(c); wait_vm<12>();
    step<2>(c); wait_vm<12>();
    step<3>(c); wait_vm<12>();
    step<4>(c); wait_vm<12>();
    step<5>(c);
    const GLB_AS char* syncp = myws + WS_FLAG + (peer * 8 + mt) * 128;
    u32x2 sy;
    asm volatile("global_load_dwordx2 %0, %1, %2 sc0 sc1" : "=v"(sy) : "v"(0u), "s"(syncp) : "memory");
    LDS_AS char* pw = ring + 2 * UNIT + lane * 16;
    LDS_AS char* pr1 = lds + (wid ^ 1) * WAVE_LDS + 2 * UNIT + lane * 16;
    LDS_AS char* pr2 = lds + (wid ^ 2) * WAVE_LDS + 2 * UNIT + 2048 + lane * 16;
    LDS_AS char* pr3 = lds + (wid ^ 3) * WAVE_LDS + 2 * UNIT + 4096 + lane * 16;
    GLB_AS char* optr = outd + (size_t)((rank * MLOC + mt * 32 + (wid & 1) * 16 + m) * NLOC + sub * 64 + (wid & 2) * 16
                                 + (g & 1) * 16 + (g >> 1) * 8) * 2;
    asm volatile("" : "+v"(pw), "+v"(pr1), "+v"(pr2), "+v"(pr3), "+v"(optr));
    wait_vm<13>();
    LSTAMP(9);
    read_x<6, 0>(c); read_w<6, 0>(c); read_x<6, 1>(c); read_w<6, 1>(c);
    SB();
    mma2<0, 2>(c); mma2<0, 3>(c); mma2<0, 0>(c); mma2<0, 1>(c);
    mma2<1, 2>(c); mma2<1, 3>(c); mma2<1, 0>(c); mma2<1, 1>(c);
    SB();
    wait_vm<5>();
    read_x<7, 0>(c); read_w<7, 0>(c); read_x<7, 1>(c);
    SB();
    mma2<0, 2>(c); mma2<0, 3>(c); mma2<0, 0>(c); mma2<0, 1>(c);
    SB();
    wait_vm<1>();
    LSTAMP(10);
    read_w<7, 1>(c);
    SB();
    mma<1, 0, 2>(c); mma<1, 1, 2>(c); mma<1, 0, 3>(c); SB();
    *(LDS_AS f32x4*)(pw + 2048) = c.acc[0][2]; SB();
    mma<1, 1, 3>(c); SB();
    *(LDS_AS f32x4*)(pw + 4096) = c.acc[1][2]; SB();
    mma<1, 1, 0>(c); SB();
    *(LDS_AS f32x4*)(pw + 3072) = c.acc[0][3]; SB();
    mma<1, 1, 1>(c); SB();
    *(LDS_AS f32x4*)(pw + 5120) = c.acc[1][3]; SB();
    mma<1, 0, 0>(c); SB();
    *(LDS_AS f32x4*)(pw + 0) = c.acc[1][0]; SB();
    mma<1, 0, 1>(c); SB();
    *(LDS_AS f32x4*)(pw + 1024) = c.acc[1][1]; SB();
    lds_barrier();
    LSTAMP(11);
    const f32x4 a1 = *(LDS_AS f32x4*)(pr1), a2 = *(LDS_AS f32x4*)(pr2), a3 = *(LDS_AS f32x4*)(pr3);
    const f32x4 b1 = *(LDS_AS f32x4*)(pr1 + 1024), b2 = *(LDS_AS f32x4*)(pr2 + 1024), b3 = *(LDS_AS f32x4*)(pr3 + 1024);
    SB();
    const f32x4 y0 = (c.acc[0][0] + a1) + (a2 + a3), y1 = (c.acc[0][1] + b1) + (b2 + b3);
    const auto s0 = __builtin_amdgcn_permlane16_swap(pack2(silu(y0[0]), silu(y0[1])), pack2(silu(y1[0]), silu(y1[1])), false, false);
    const auto s1 = __builtin_amdgcn_permlane16_swap(pack2(silu(y0[2]), silu(y0[3])), pack2(silu(y1[2]), silu(y1[3])), false, false);
    const u32x4 o = {s0[0], s1[0], s0[1], s1[1]};
    asm volatile("s_waitcnt vmcnt(0)" : "+v"(sy) :: "memory");
    bool ok = true;
    if (peer != rank && __builtin_amdgcn_readfirstlane(sy[0] < seq || sy[1] < seq)) {
        const unsigned long long t0 = __builtin_amdgcn_s_memrealtime();
        for (unsigned spins = 1;; ++spins) {
            asm volatile("global_load_dwordx2 %0, %1, %2 sc0 sc1\n\ts_waitcnt vmcnt(0)" : "=&v"(sy) : "v"(0u), "s"(syncp) : "memory");
            if (__builtin_amdgcn_readfirstlane(sy[0] >= seq && sy[1] >= seq)) break;
            if ((spins & 63) == 0 && __builtin_amdgcn_readfirstlane(expired(p, seq, t0))) { ok = false; break; }
            __builtin_amdgcn_s_sleep(1);
        }
    }
    if (ok) store16_sys(optr, o);
    STAMP(12);
    if (lane == 0) __hip_atomic_store(cnt, seq, __ATOMIC_RELAXED, __HIP_MEMORY_SCOPE_AGENT);
    STAMP(13);
    if (peer != rank && ok) {
        const GLB_AS char* mine = myout + (size_t)((peer * MLOC + mt * 32 + wid * 8 + (lane >> 3)) * NLOC + sub * 64 + (lane & 7) * 8) * 2;
        const unsigned long long t0 = __builtin_amdgcn_s_memrealtime();
        for (unsigned spins = 1;; ++spins) {
            u32x4 r;
            asm volatile("global_load_dwordx4 %0, %1, off sc0 sc1\n\ts_waitcnt vmcnt(0)" : "=&v"(r) : "v"(mine) : "memory");
            if (__all(r[0] != SENTINEL && r[1] != SENTINEL && r[2] != SENTINEL && r[3] != SENTINEL)) break;
            if ((spins & 63) == 0 && __builtin_amdgcn_readfirstlane(expired(p, seq, t0))) break;
        }
    }
    STAMP(14);
}

extern "C" void kernel_launch(int rank, int nranks, ncclComm_t comm,
                              hipStream_t stream,
                              void* const* d_in, const int* in_sizes, int n_in,
                              void* d_out, int out_size,
                              const int* in_dtypes, int out_dtype,
                              void* const* d_out_all,
                              void* const* d_ws_all, size_t ws_bytes) {
    if (nranks != NRANK || n_in < 2 || (unsigned)rank >= (unsigned)NRANK) return;
    Params p;
    const uintptr_t ax = (uintptr_t)d_in[0], aw = (uintptr_t)d_in[1];
    p.srdx[0] = (unsigned)ax; p.srdx[1] = (unsigned)(ax >> 32) & 0xffffu; p.srdx[2] = MLOC * KDIM * 2; p.srdx[3] = 0x00020000u;
    p.srdw[0] = (unsigned)aw; p.srdw[1] = (unsigned)(aw >> 32) & 0xffffu; p.srdw[2] = KDIM * NDIM * 2; p.srdw[3] = 0x00020000u;
    p.myws = (char*)d_ws_all[rank];
    p.myout = (char*)d_out;
    p.rank = (unsigned)rank;
    p.pad[0] = p.pad[1] = p.pad[2] = 0;
    for (int i = 0; i < NRANK; ++i) {
        p.out[i] = (char*)d_out_all[i];
        p.ws[i] = (char*)d_ws_all[i];
    }
    gemm_a2a<<<dim3(NWG), dim3(NW * 64), 0, stream>>>(p);
    HIP_CHECK(hipGetLastError());
}
```

```cpp
#include <hip/hip_runtime.h>
#include <rccl/rccl.h>
#include <cstdio>
#include <cstdint>

#define HIP_CHECK(expr) do { hipError_t _e = (expr); if (_e != hipSuccess) { \
    fprintf(stderr, "HIP error %s at %s:%d\n", hipGetErrorString(_e), __FILE__, __LINE__); \
    return; } } while (0)

typedef __attribute__((ext_vector_type(8))) __bf16 bf16x8;
typedef __attribute__((ext_vector_type(2))) __bf16 bf16x2;
typedef __attribute__((ext_vector_type(4))) short s16x4;
typedef __attribute__((ext_vector_type(4))) float f32x4;
typedef __attribute__((ext_vector_type(4))) unsigned u32x4;
typedef __attribute__((ext_vector_type(2))) unsigned u32x2;
#define LDS_AS __attribute__((address_space(3)))
#define GLB_AS __attribute__((address_space(1)))

constexpr int NRANK = 8;
constexpr int KDIM = 2048, NDIM = 2048, MLOC = 256, NLOC = 256;
constexpr int NW = 4;
constexpr int UNIT = 12288;
constexpr int NSLOT = 3;
constexpr int WAVE_LDS = NSLOT * UNIT + 4096;
constexpr int LDS_BYTES = NW * WAVE_LDS;
constexpr int voff(int v) { return v < 3 ? v * UNIT + 8192 + 2048 : NSLOT * UNIT + (v - 3) * 2048; }
constexpr int xoff(int u) { return u == 3 ? NSLOT * UNIT : (u % NSLOT) * UNIT + 8192; }
constexpr int NWG = 256;
#define DLY 4
#define LDLY 8
#define RET 4
#define SYNC_STRIDE 8
#define NA 4
#define NB 0
#define PROTO 1

constexpr int WS_FLAG = 0;
constexpr int WS_CNT = NWG * NW * 128;
constexpr int WS_TMO = 2 * NWG * NW * 128;
constexpr int WS_JUNK = WS_TMO + 4096;
constexpr int WS_DBG = WS_JUNK + 32768;
constexpr unsigned long long DEADLINE_TICKS = 50000000ull;
constexpr unsigned SENTINEL = 0xFF81FF81u;

struct Params {
    unsigned srdx[4];
    unsigned srdw[4];
    char* myws;
    char* myout;
    unsigned rank;
    unsigned pad[3];
    char* out[NRANK];
    char* ws[NRANK];
};

template <int LOFF, int GOFF>
__device__ __forceinline__ void dma16(unsigned lds_base, unsigned voff, unsigned sbase, u32x4 srd) {
    unsigned t;
    asm volatile("s_add_u32 m0, %1, %2\n\ts_add_u32 %0, %6, %3\n\tbuffer_load_dwordx4 %4, %5, %0 offen lds"
                 : "=&s"(t) : "s"(lds_base), "i"(LOFF), "i"(GOFF), "v"(voff), "s"(srd), "s"(sbase) : "memory", "scc");
}
template <int N> __device__ __forceinline__ void wait_vm() {
    asm volatile("s_waitcnt vmcnt(%0)" :: "i"(N) : "memory");
}
__device__ __forceinline__ void lds_barrier() { asm volatile("s_waitcnt lgkmcnt(0)\n\ts_barrier" ::: "memory"); }
__device__ __forceinline__ void store8_sys(GLB_AS char* ptr, u32x2 v) {
    asm volatile("global_store_dwordx2 %0, %1, off sc0 sc1" :: "v"(ptr), "v"(v) : "memory");
}
__device__ __forceinline__ void store16_sys(GLB_AS char* ptr, u32x4 v) {
    asm volatile("global_store_dwordx4 %0, %1, off sc0 sc1\n\ts_nop 1" :: "v"(ptr), "v"(v) : "memory");
}
__device__ __forceinline__ void barriers(unsigned n) {
    for (unsigned i = 0; i < n; ++i) asm volatile("s_barrier" ::: "memory");
}
#define SB() __builtin_amdgcn_sched_barrier(0)
__device__ __forceinline__ unsigned ssel(unsigned cond, unsigned a, unsigned b) {
    unsigned r;
    asm("s_cmp_lg_u32 %1, 0\n\ts_cselect_b32 %0, %2, %3" : "=s"(r) : "s"(cond), "s"(a), "s"(b) : "scc");
    return r;
}

struct Ctx {
    unsigned lds_base;
    unsigned kbase, lmask;
    unsigned vx[2], vw;
    u32x4 sx, sw;
    LDS_AS char* ax[2];
    LDS_AS char* aw[4];
    bf16x8 wf[8][2][4];
    bf16x8 xf[8][2][2];
    bf16x8 xv[RET][2];
    f32x4 acc[2][4];
};
#define STAMP(i)
#define LSTAMP(i)
#define TSTAMP(i)

template <int U> __device__ __forceinline__ unsigned block_of(const Ctx& c) {
    constexpr unsigned tl = 18u * (U & 1) ^ (U >> 1), tf = 4u * U;
    return c.kbase ^ tf ^ ((tl ^ tf) & c.lmask);
}
constexpr bool full(int u) { return u < 8 - RET; }
constexpr int nloads(int u) { return u == 8 ? 2 * RET : (u == 2 || u == 3) ? 8 : full(u) ? 12 : 10; }
template <int U, int I, bool ALLX = false> __device__ __forceinline__ void piece(Ctx& c, unsigned kb) {
    constexpr int so = (U % NSLOT) * UNIT;
    if constexpr (U == 8) {
        if constexpr (I < 2 * RET)
            dma16<voff(I / 2) + (I & 1) * 1024, (16 + (I & 1) * 8) * (KDIM * 2)>(c.lds_base, c.vx[I & 1], block_of<8 - RET + I / 2>(c) << 7, c.sx);
    } else if constexpr (I < 4) {
        if constexpr (I < 2 || full(U) || ALLX) dma16<xoff(U) + I * 1024, I * 8 * (KDIM * 2)>(c.lds_base, c.vx[I & 1], kb << 7, c.sx);
    } else dma16<so + (I - 4) * 1024, (((I - 4) >> 1) * 16 + (I & 1) * 4) * (NDIM * 2)>(c.lds_base, c.vw, kb << 18, c.sw);
}
template <int U, int K> __device__ __forceinline__ void pseq(Ctx& c, unsigned kb) {
    constexpr int order[12] = {0, 1, 4, 5, 6, 2, 7, 3, 8, 9, 10, 11};
    if constexpr (U == 2 || U == 3) {
        if constexpr (K == 0) {
            if (!c.lmask) { piece<U, 0>(c, kb); piece<U, 1>(c, kb); piece<U, 2>(c, kb); piece<U, 3>(c, kb); }
        } else if constexpr (order[K] >= 4) piece<U, order[K]>(c, kb);
    } else if constexpr (U < 8) piece<U, order[K]>(c, kb);
    else if constexpr (U == 8 && K < NA) piece<8, K>(c, kb);
}
template <int U> __device__ __forceinline__ void issue_x(Ctx& c, unsigned kb) {
    piece<U, 0, true>(c, kb); piece<U, 1, true>(c, kb); piece<U, 2, true>(c, kb); piece<U, 3, true>(c, kb);
}
template <int U> __device__ __forceinline__ void issue_w(Ctx& c, unsigned kb) {
    piece<U, 4>(c, kb); piece<U, 5>(c, kb); piece<U, 6>(c, kb); piece<U, 7>(c, kb);
    piece<U, 8>(c, kb); piece<U, 9>(c, kb); piece<U, 10>(c, kb); piece<U, 11>(c, kb);
}
template <int U, int S> __device__ __forceinline__ void read_x(Ctx& c) {
    constexpr int so = xoff(U);
    c.xf[U][S][0] = *(LDS_AS bf16x8*)(c.ax[S] + so);
    if constexpr (full(U)) c.xf[U][S][1] = *(LDS_AS bf16x8*)(c.ax[S] + so + 2048);
}
template <int U, int S, int T> __device__ __forceinline__ void read_w1(Ctx& c) {
    constexpr int so = (U % NSLOT) * UNIT;
    s16x4 lo = __builtin_amdgcn_ds_read_tr16_b64_v4i16((LDS_AS s16x4*)(c.aw[T] + so + S * 4096));
    s16x4 hi = __builtin_amdgcn_ds_read_tr16_b64_v4i16((LDS_AS s16x4*)(c.aw[T] + so + S * 4096 + 1024));
    c.wf[U][S][T] = __builtin_bit_cast(bf16x8, __builtin_shufflevector(lo, hi, 0, 1, 2, 3, 4, 5, 6, 7));
}
template <int U, int S> __device__ __forceinline__ void read_w(Ctx& c) {
    read_w1<U, S, 0>(c); read_w1<U, S, 1>(c); read_w1<U, S, 2>(c); read_w1<U, S, 3>(c);
}
template <int U, int S, int T> __device__ __forceinline__ void mma(Ctx& c) {
    c.acc[0][T] = __builtin_amdgcn_mfma_f32_16x16x32_bf16(c.wf[U][S][T], c.xf[U][S][0], c.acc[0][T], 0, 0, 0);
    if constexpr (full(U))
        c.acc[1][T] = __builtin_amdgcn_mfma_f32_16x16x32_bf16(c.wf[U][S][T], c.xf[U][S][1], c.acc[1][T], 0, 0, 0);
}
template <int V> __device__ __forceinline__ void read_v(Ctx& c) {
    c.xv[V][0] = *(LDS_AS bf16x8*)(c.ax[0] + voff(V));
    c.xv[V][1] = *(LDS_AS bf16x8*)(c.ax[1] + voff(V));
}
template <int V, int S> __device__ __forceinline__ void mma_v(Ctx& c) {
#pragma unroll
    for (int t = 0; t < 4; ++t)
        c.acc[1][t] = __builtin_amdgcn_mfma_f32_16x16x32_bf16(c.wf[8 - RET + V][S][t], c.xv[V][S], c.acc[1][t], 0, 0, 0);
}
template <int V, int E> __device__ __forceinline__ void read_vs(Ctx& c) {
    if constexpr (V < E) { read_v<V>(c); read_vs<V + 1, E>(c); }
}
template <int V, int E> __device__ __forceinline__ void mma_vs(Ctx& c) {
    if constexpr (V < E) { mma_v<V, 0>(c); mma_v<V, 1>(c); mma_vs<V + 1, E>(c); }
}

template <int I, int E> __device__ __forceinline__ void issue_v(Ctx& c) {
    if constexpr (I < E) { piece<8, I>(c, 0); issue_v<I + 1, E>(c); }
}
template <int V, int S, int T> __device__ __forceinline__ void mv(Ctx& c) {
    c.acc[1][T] = __builtin_amdgcn_mfma_f32_16x16x32_bf16(c.wf[8 - RET + V][S][T], c.xv[V][S], c.acc[1][T], 0, 0, 0);
    SB();
}
template <int V> __device__ __forceinline__ void pass2(Ctx& c) {
    if constexpr (V < RET) {
        if constexpr (V + 1 < RET) {
            wait_vm<2 * RET - 2 * (V + 2) + 1>();
            read_v<V + 1>(c);
            SB();
        }
        mma_v<V, 0>(c); mma_v<V, 1>(c);
        SB();
        pass2<V + 1>(c);
    }
}

template <int U, int N = U + 2> __device__ __forceinline__ void step(Ctx& c) {
    LSTAMP(3 + U);
    const unsigned kb = block_of<N & 7>(c);
    pseq<N, 0>(c, kb); read_x<U, 0>(c); read_w1<U, 0, 0>(c); SB();
    pseq<N, 1>(c, kb); read_w1<U, 0, 1>(c); read_w1<U, 0, 2>(c); SB();
    pseq<N, 2>(c, kb); read_w1<U, 0, 3>(c); read_x<U, 1>(c); SB();
    pseq<N, 3>(c, kb); read_w1<U, 1, 0>(c); read_w1<U, 1, 1>(c); SB();
    pseq<N, 4>(c, kb); read_w1<U, 1, 2>(c); read_w1<U, 1, 3>(c); SB();
    mma<U, 0, 0>(c); SB(); pseq<N, 5>(c, kb); SB();
    mma<U, 0, 1>(c); SB(); pseq<N, 6>(c, kb); SB();
    mma<U, 0, 2>(c); SB(); pseq<N, 7>(c, kb); SB();
    mma<U, 0, 3>(c); SB(); pseq<N, 8>(c, kb); SB();
    mma<U, 1, 0>(c); SB(); pseq<N, 9>(c, kb); SB();
    mma<U, 1, 1>(c); SB(); pseq<N, 10>(c, kb); SB();
    mma<U, 1, 2>(c); SB(); pseq<N, 11>(c, kb); SB();
    mma<U, 1, 3>(c); SB();
}

__device__ __forceinline__ unsigned pack2(float a, float b) {
    bf16x2 v = {(__bf16)a, (__bf16)b};
    return __builtin_bit_cast(unsigned, v);
}
#define PIN1(a) do { asm volatile("" : "+v"(a)); SB(); } while (0)
#define PIN2(a, b) do { asm volatile("" : "+v"(a), "+v"(b)); SB(); } while (0)
__device__ __forceinline__ u32x2 finish4(const f32x4 a0, const f32x4 a1, const f32x4 a2, const f32x4 a3) {
    const f32x4 y = (a0 + a1) + (a2 + a3);
    f32x4 e;
#pragma unroll
    for (int i = 0; i < 4; ++i) e[i] = __builtin_amdgcn_fmed3f(y[i], -60.0f, 60.0f);
    e *= -1.4426950408889634f;
    PIN1(e);
#pragma unroll
    for (int i = 0; i < 4; ++i) e[i] = __builtin_amdgcn_exp2f(e[i]);
    PIN1(e);
    e += 1.0f;
    PIN1(e);
#pragma unroll
    for (int i = 0; i < 4; ++i) e[i] = __builtin_amdgcn_rcpf(e[i]);
    PIN1(e);
    e *= y;
    return u32x2{pack2(e[0], e[1]), pack2(e[2], e[3])};
}
__device__ __forceinline__ bool expired(const Params& p, unsigned seq, unsigned long long t0) {
    unsigned* tmo = (unsigned*)(p.myws + WS_TMO);
    if (__hip_atomic_load(tmo, __ATOMIC_RELAXED, __HIP_MEMORY_SCOPE_AGENT) == seq
        || __builtin_amdgcn_s_memrealtime() - t0 > DEADLINE_TICKS) {
        __hip_atomic_store(tmo, seq, __ATOMIC_RELAXED, __HIP_MEMORY_SCOPE_AGENT);
        __hip_atomic_store((unsigned short*)(p.myout + (size_t)((p.rank & 7) * MLOC * NLOC) * 2),
                           (unsigned short)0x7FC0, __ATOMIC_RELAXED, __HIP_MEMORY_SCOPE_SYSTEM);
        return true;
    }
    return false;
}

__global__ __launch_bounds__(512) void gemm_a2a(Params p) {
    __shared__ __attribute__((aligned(1024))) char smem[LDS_BYTES];
    Ctx c;
    STAMP(0);
    const unsigned b = blockIdx.x;
    const unsigned j = b & 7, mt = (b >> 3) & 7, nt = (b >> 6) & 3;
    const unsigned peer = (j & 1) * 4 + nt;
    u32x4 kws;
    unsigned rank;
    GLB_AS char* outd;
    {
        const void* ka = (const void*)__builtin_amdgcn_kernarg_segment_ptr();
        asm volatile("s_load_dwordx4 %0, %5, 0x0\n\ts_load_dwordx4 %1, %5, 0x10\n\ts_load_dwordx4 %2, %5, 0x20\n\t"
                     "s_load_dword %3, %5, 0x30\n\ts_load_dwordx2 %4, %5, %6 offset:0x40"
                     : "=&s"(c.sx), "=&s"(c.sw), "=&s"(kws), "=&s"(rank), "=&s"(outd) : "s"(ka), "s"(peer * 8));
        SB();
    }
    const unsigned lane = threadIdx.x & 63;
    const unsigned wid = __builtin_amdgcn_readfirstlane(threadIdx.x >> 6);
    const unsigned sub = (j >> 1) ^ nt;
    LDS_AS char* lds = (LDS_AS char*)smem;
    c.lds_base = (unsigned)(uintptr_t)lds + wid * WAVE_LDS;
    c.lmask = ((mt ^ nt) & 1) - 1;
    const unsigned cs = (mt >> 1) ^ (nt & 2);
    c.kbase = ssel(c.lmask, 4 * (wid + 1) + (cs ^ ((wid + 1) >> 1)), wid ^ cs);
    const unsigned g = lane >> 4, m = lane & 15;
    c.vx[0] = (mt * 32 + (lane >> 3)) * (KDIM * 2) + (((lane & 7) ^ g) << 4);
    c.vx[1] = c.vx[0] ^ 64;
    const unsigned kb0 = block_of<0>(c), kb1 = block_of<1>(c);
    const unsigned nsl = wid + (((peer * 4 + sub) & 7) == 3 ? 0 : (LDLY / DLY) & c.lmask);
    asm volatile("" : "+v"(c.vx[0]), "+v"(c.vx[1]));
    asm volatile("s_waitcnt lgkmcnt(0)" : "+s"(c.sx), "+s"(c.sw), "+s"(kws), "+s"(rank), "+s"(outd));
    if (__builtin_expect(nsl != 0, 0)) {
        for (unsigned i = 0; i < nsl; ++i) __builtin_amdgcn_s_sleep(DLY);
    }
    LSTAMP(1);
    issue_x<0>(c, kb0);
    c.vw = (((lane >> 3) & 3) + (lane >> 5) * 8) * (NDIM * 2) + (peer * 4 + sub) * 128 + (((lane & 7) ^ (g << 1)) << 4);
    issue_w<0>(c, kb0);
    if (c.lmask) issue_x<2>(c, kb0 ^ 1);
    barriers(1 + (wid & c.lmask));
    rank &= 7;
    GLB_AS char* const myws = (GLB_AS char*)(((unsigned long long)kws[1] << 32) | kws[0]);
    GLB_AS char* const myout = (GLB_AS char*)(((unsigned long long)kws[3] << 32) | kws[2]);
    const unsigned arm = wid * 16 + (j >> 1) * 4 + (mt >> 1);
    const bool armer = ((peer ^ rank) | c.lmask) == 0 && (arm >> 3) != rank;
    if (__builtin_expect(armer, 0)) {
        GLB_AS char* fp = myout + (size_t)arm * (32 * NLOC * 2) + lane * 16;
        const u32x4 sv = {SENTINEL, SENTINEL, SENTINEL, SENTINEL};
#pragma unroll
        for (int i = 0; i < 16; ++i) store16_sys(fp + i * 1024, sv);
    }
    GLB_AS unsigned* cnt = (GLB_AS unsigned*)(myws + WS_CNT + (b * NW + wid) * 128);
    unsigned seq;
    asm volatile("global_load_dword %0, %1, %2 sc1" : "=v"(seq) : "v"(0u), "s"(cnt) : "memory");
    LDS_AS char* ring = lds + wid * WAVE_LDS;
    const unsigned q = m >> 2;
    piece<1, 4>(c, kb1); piece<1, 5>(c, kb1); SB();
    c.ax[0] = ring + m * 128 + ((g ^ (m >> 1)) << 4);
    c.ax[1] = (LDS_AS char*)(uintptr_t)((unsigned)(uintptr_t)c.ax[0] ^ 64);
    asm volatile("" : "+v"(c.ax[0]), "+v"(c.ax[1]));
    piece<1, 6>(c, kb1); SB();
    c.aw[0] = ring + (g >> 1) * 2048 + (4 * (g & 1) + q) * 128 + 8 * (lane & 3);
    asm volatile("" : "+v"(c.aw[0]));
    piece<1, 7>(c, kb1); SB();
    c.aw[0] += (2 * (g & 1) + (q >> 1)) << 5;
#pragma unroll
    for (int t = 1; t < 4; ++t) c.aw[t] = (LDS_AS char*)(uintptr_t)((unsigned)(uintptr_t)c.aw[0] ^ (t << 5));
    asm volatile("" : "+v"(c.aw[0]), "+v"(c.aw[1]), "+v"(c.aw[2]), "+v"(c.aw[3]));
    piece<1, 8>(c, kb1); piece<1, 9>(c, kb1); piece<1, 10>(c, kb1); piece<1, 11>(c, kb1);
    issue_x<1>(c, kb1);
    if (c.lmask) issue_x<3>(c, kb1 ^ 1);
    barriers((4 - wid) & c.lmask);
    LSTAMP(2);
#pragma unroll
    for (int h = 0; h < 2; ++h)
#pragma unroll
        for (int t = 0; t < 4; ++t) c.acc[h][t] = f32x4{0.f, 0.f, 0.f, 0.f};

    if (c.lmask) wait_vm<17>();
    else asm volatile("s_waitcnt vmcnt(12)" : "+v"(seq) :: "memory");
    if (__builtin_expect(armer, 0) && lane == 0) {
        __hip_atomic_store((unsigned*)(p.ws[(arm >> 3) & 7] + WS_FLAG + (rank * 8 + (arm & 7)) * SYNC_STRIDE), seq + 1, __ATOMIC_RELAXED, __HIP_MEMORY_SCOPE_SYSTEM);
        __hip_atomic_store((GLB_AS unsigned*)(myws + WS_FLAG + arm * SYNC_STRIDE + 4), seq + 1, __ATOMIC_RELAXED, __HIP_MEMORY_SCOPE_SYSTEM);
    }
    step<0>(c);
    asm volatile("s_waitcnt vmcnt(12)" : "+v"(seq) :: "memory");
    seq += 1;
    step<1>(c); wait_vm<nloads(3)>();
    step<2>(c); wait_vm<nloads(4)>();
    step<3>(c); wait_vm<nloads(5)>();
    step<4>(c);
    static_assert(RET >= 3 && RET <= 5, "voff() needs units 5..7 without rows 16..31 and unit 3 with them");
    wait_vm<nloads(6)>();
    LSTAMP(8);
    {
        const unsigned kb7 = block_of<7>(c);
        piece<7, 0>(c, kb7); read_x<5, 0>(c); read_x<5, 1>(c); SB();
        piece<7, 1>(c, kb7); read_w1<5, 0, 0>(c); SB();
        piece<7, 4>(c, kb7); read_w1<5, 0, 1>(c); SB();
        piece<7, 5>(c, kb7); read_w1<5, 0, 2>(c); SB();
        piece<7, 6>(c, kb7); read_w1<5, 0, 3>(c); SB();
        piece<7, 7>(c, kb7); read_w1<5, 1, 0>(c); SB();
        piece<7, 8>(c, kb7); read_w1<5, 1, 1>(c); SB();
        piece<7, 9>(c, kb7); read_w1<5, 1, 2>(c); SB();
        piece<7, 10>(c, kb7); read_w1<5, 1, 3>(c); SB();
        piece<7, 11>(c, kb7); SB();
    }
    LDS_AS char* pw0 = ring + m * 256 + ((g ^ (m & 7)) << 4);
    LDS_AS char* pw1 = (LDS_AS char*)(uintptr_t)((unsigned)(uintptr_t)pw0 ^ 64);
    const unsigned row = wid * 4 + g;
    LDS_AS char* pr0 = lds + row * 256 + ((m ^ (row & 7)) << 4);
    LDS_AS char* pr2 = pr0 + 2 * WAVE_LDS;
    GLB_AS char* optr = outd + (size_t)((rank * MLOC + mt * 32 + row) * NLOC + sub * 64 + m * 4) * 2;
    GLB_AS char* optr2 = optr + 16 * NLOC * 2;
    const GLB_AS char* syncp = myws + WS_FLAG + (peer * 8 + mt) * SYNC_STRIDE;
    TSTAMP(1);
    u32x2 sy;
    __builtin_amdgcn_s_barrier();
    asm volatile("global_load_dwordx2 %0, %1, %2 sc0 sc1" : "=v"(sy) : "v"(0u), "s"(syncp) : "memory");
    mma<5, 0, 0>(c); mma<5, 0, 1>(c); SB();
    wait_vm<nloads(7) + 1>();
    LSTAMP(9);
    read_x<6, 0>(c); read_x<6, 1>(c); read_w1<6, 0, 0>(c); SB(); mma<5, 0, 2>(c); SB(); pseq<8, 0>(c, 0); SB();
    read_w1<6, 0, 1>(c); read_w1<6, 0, 2>(c); SB(); mma<5, 0, 3>(c); SB(); pseq<8, 1>(c, 0); SB();
    read_w1<6, 0, 3>(c); read_w1<6, 1, 0>(c); SB(); mma<5, 1, 0>(c); SB(); pseq<8, 2>(c, 0); SB();
    read_w1<6, 1, 1>(c); read_w1<6, 1, 2>(c); SB(); mma<5, 1, 1>(c); SB(); pseq<8, 3>(c, 0); SB();
    read_w1<6, 1, 3>(c); SB(); mma<5, 1, 2>(c); mma<5, 1, 3>(c); SB(); pseq<8, 4>(c, 0); SB();
    mma<6, 0, 0>(c); mma<6, 0, 1>(c); SB(); pseq<8, 5>(c, 0); SB();
    mma<6, 0, 2>(c); mma<6, 0, 3>(c); SB(); pseq<8, 6>(c, 0); SB(); pseq<8, 7>(c, 0); SB();
    asm volatile("" : "+v"(pw0), "+v"(pw1), "+v"(pr0), "+v"(pr2), "+v"(optr), "+v"(optr2));
    TSTAMP(2);
    wait_vm<4 + 1 + NA>();
    read_x<7, 0>(c); read_w<7, 0>(c); read_x<7, 1>(c);
    SB();
    mma<6, 1, 0>(c); mma<6, 1, 1>(c); mma<6, 1, 2>(c); mma<6, 1, 3>(c);
    SB();
    wait_vm<1 + NA>();
    LSTAMP(10);
    TSTAMP(3);
    read_w<7, 1>(c);
    SB();
    mma<7, 0, 0>(c); SB(); mma<7, 0, 1>(c); SB(); mma<7, 0, 2>(c); SB(); mma<7, 0, 3>(c); SB();
    mma<7, 1, 0>(c); SB(); mma<7, 1, 1>(c); SB(); mma<7, 1, 2>(c); SB();
    *(LDS_AS f32x4*)(pw0) = c.acc[0][0];
    SB(); mma<7, 1, 3>(c); SB();
    *(LDS_AS f32x4*)(pw1) = c.acc[0][1];
    *(LDS_AS f32x4*)(pw0 + 128) = c.acc[0][2];
    *(LDS_AS f32x4*)(pw1 + 128) = c.acc[0][3];
    SB();
    issue_v<NA, 2 * RET>(c);
    asm volatile("s_waitcnt vmcnt(%1)" : "+v"(sy) : "i"(2 * RET - 4) : "memory");
    TSTAMP(6);
    read_v<0>(c); read_v<1>(c);
    bool ok = true;
    if (__builtin_expect(peer != rank && __builtin_amdgcn_readfirstlane(sy[0] < seq || sy[1] < seq), 0)) {
        const unsigned long long t0 = __builtin_amdgcn_s_memrealtime();
        for (unsigned spins = 1;; ++spins) {
            asm volatile("global_load_dwordx2 %0, %1, %2 sc0 sc1\n\ts_waitcnt vmcnt(0)" : "=&v"(sy) : "v"(0u), "s"(syncp) : "memory");
            if (__builtin_amdgcn_readfirstlane(sy[0] >= seq && sy[1] >= seq)) break;
            if ((spins & 63) == 0 && __builtin_amdgcn_readfirstlane(expired(p, seq, t0))) {
                ok = false;
                optr = optr2 = myws + WS_JUNK + threadIdx.x * 8;
                break;
            }
            __builtin_amdgcn_s_sleep(1);
        }
    }
    asm volatile("s_waitcnt lgkmcnt(4)\n\ts_barrier" ::: "memory");
    TSTAMP(7);
    {
        const f32x4 a0 = *(LDS_AS f32x4*)(pr0), a1 = *(LDS_AS f32x4*)(pr0 + WAVE_LDS);
        const f32x4 a2 = *(LDS_AS f32x4*)(pr2), a3 = *(LDS_AS f32x4*)(pr2 + WAVE_LDS);
        SB();
        mv<0, 0, 0>(c); mv<0, 0, 1>(c); mv<0, 0, 2>(c); mv<0, 0, 3>(c); mv<0, 1, 0>(c); mv<0, 1, 1>(c);
        f32x4 y = a0 + a1, z = a2 + a3, e;
        PIN2(y, z);
        mv<0, 1, 2>(c);
        y += z;
        z[0] = __builtin_amdgcn_fmed3f(y[0], -60.0f, 60.0f); z[1] = __builtin_amdgcn_fmed3f(y[1], -60.0f, 60.0f);
        PIN2(y, z);
        mv<0, 1, 3>(c);
        z[2] = __builtin_amdgcn_fmed3f(y[2], -60.0f, 60.0f); z[3] = __builtin_amdgcn_fmed3f(y[3], -60.0f, 60.0f);
        z *= -1.4426950408889634f;
        PIN1(z);
        mv<1, 0, 0>(c);
        e[0] = __builtin_amdgcn_exp2f(z[0]); e[1] = __builtin_amdgcn_exp2f(z[1]); e[2] = z[2]; e[3] = z[3];
        PIN1(e);
        mv<1, 0, 1>(c);
        e[2] = __builtin_amdgcn_exp2f(e[2]); e[3] = __builtin_amdgcn_exp2f(e[3]);
        PIN1(e);
        mv<1, 0, 2>(c);
        e += 1.0f;
        e[0] = __builtin_amdgcn_rcpf(e[0]);
        PIN1(e);
        mv<1, 0, 3>(c);
        e[1] = __builtin_amdgcn_rcpf(e[1]); e[2] = __builtin_amdgcn_rcpf(e[2]);
        PIN1(e);
        mv<1, 1, 0>(c);
        e[3] = __builtin_amdgcn_rcpf(e[3]);
        e *= y;
        const u32x2 o = {pack2(e[0], e[1]), pack2(e[2], e[3])};
        TSTAMP(8);
        store8_sys(optr, o);
        SB();
    }
    LSTAMP(11);
    TSTAMP(9);
    if constexpr (RET > 2) { wait_vm<2 * RET - 6 + 1>(); read_v<2>(c); SB(); }
    mv<1, 1, 1>(c); mv<1, 1, 2>(c); mv<1, 1, 3>(c);
    pass2<2>(c);
    TSTAMP(10);
    *(LDS_AS f32x4*)(pw0 + UNIT) = c.acc[1][0];
    *(LDS_AS f32x4*)(pw1 + UNIT) = c.acc[1][1];
    *(LDS_AS f32x4*)(pw0 + UNIT + 128) = c.acc[1][2];
    *(LDS_AS f32x4*)(pw1 + UNIT + 128) = c.acc[1][3];
    lds_barrier();
    TSTAMP(11);
    {
        const f32x4 a0 = *(LDS_AS f32x4*)(pr0 + UNIT), a1 = *(LDS_AS f32x4*)(pr0 + UNIT + WAVE_LDS);
        const f32x4 a2 = *(LDS_AS f32x4*)(pr2 + UNIT), a3 = *(LDS_AS f32x4*)(pr2 + UNIT + WAVE_LDS);
        SB();
        store8_sys(optr2, finish4(a0, a1, a2, a3));
    }
    STAMP(12);
    if (lane == 0) __hip_atomic_store(cnt, seq, __ATOMIC_RELAXED, __HIP_MEMORY_SCOPE_AGENT);
    STAMP(13);
    if (peer != rank && ok) {
        const GLB_AS char* mine = myout + (size_t)((peer * MLOC + mt * 32 + wid * 8 + (lane >> 3)) * NLOC + sub * 64 + (lane & 7) * 8) * 2;
        const unsigned long long t0 = __builtin_amdgcn_s_memrealtime();
#define POLL 3
#define PSL 0
#define PLD(r) asm volatile("global_load_dwordx4 %0, %1, off sc0 sc1\n\ts_sleep %2" : "+v"(r) : "v"(mine), "i"(PSL) : "memory")
#define PCK(r) asm volatile("s_waitcnt vmcnt(2)" : "+v"(r) :: "memory"); \
               if (__all(r[0] != SENTINEL && r[1] != SENTINEL && r[2] != SENTINEL && r[3] != SENTINEL)) break;
        u32x4 r0 = {0, 0, 0, 0}, r1 = r0, r2 = r0;
        PLD(r0); PLD(r1); PLD(r2);
        for (unsigned spins = 1;; ++spins) {
            PCK(r0); PLD(r0);
            PCK(r1); PLD(r1);
            PCK(r2); PLD(r2);
            if ((spins & 63) == 0) {
                asm volatile("s_waitcnt vmcnt(0)" : "+v"(r0), "+v"(r1), "+v"(r2) :: "memory");
                if (__builtin_amdgcn_readfirstlane(expired(p, seq, t0))) break;
            }
        }
        asm volatile("s_waitcnt vmcnt(0)" : "+v"(r0), "+v"(r1), "+v"(r2) :: "memory");
    }
    STAMP(14);
}

extern "C" void kernel_launch(int rank, int nranks, ncclComm_t comm,
                              hipStream_t stream,
                              void* const* d_in, const int* in_sizes, int n_in,
                              void* d_out, int out_size,
                              const int* in_dtypes, int out_dtype,
                              void* const* d_out_all,
                              void* const* d_ws_all, size_t ws_bytes) {
    if (nranks != NRANK || n_in < 2 || (unsigned)rank >= (unsigned)NRANK) return;
    Params p;
    const uintptr_t ax = (uintptr_t)d_in[0], aw = (uintptr_t)d_in[1];
    p.srdx[0] = (unsigned)ax; p.srdx[1] = (unsigned)(ax >> 32) & 0xffffu; p.srdx[2] = MLOC * KDIM * 2; p.srdx[3] = 0x00020000u;
    p.srdw[0] = (unsigned)aw; p.srdw[1] = (unsigned)(aw >> 32) & 0xffffu; p.srdw[2] = KDIM * NDIM * 2; p.srdw[3] = 0x00020000u;
    p.myws = (char*)d_ws_all[rank];
    p.myout = (char*)d_out;
    p.rank = (unsigned)rank;
    p.pad[0] = p.pad[1] = p.pad[2] = 0;
    for (int i = 0; i < NRANK; ++i) {
        p.out[i] = (char*)d_out_all[i];
        p.ws[i] = (char*)d_ws_all[i];
    }
    gemm_a2a<<<dim3(NWG), dim3(NW * 64), 0, stream>>>(p);
    HIP_CHECK(hipGetLastError());
}
```

```cpp
#include <hip/hip_runtime.h>
#include <rccl/rccl.h>
#include <cstddef>
#include <cstdint>
#include <cstdio>

#define HIP_CHECK(expr) do { hipError_t _e = (expr); if (_e != hipSuccess) { \
    fprintf(stderr, "HIP error %s at %s:%d\n", hipGetErrorString(_e), __FILE__, __LINE__); \
    return; } } while (0)

typedef __attribute__((ext_vector_type(8))) __bf16 bf16x8;
typedef __attribute__((ext_vector_type(2))) __bf16 bf16x2;
typedef __attribute__((ext_vector_type(4))) short s16x4;
typedef __attribute__((ext_vector_type(4))) float f32x4;
typedef __attribute__((ext_vector_type(4))) unsigned u32x4;
typedef __attribute__((ext_vector_type(2))) unsigned u32x2;
#define LDS_AS __attribute__((address_space(3)))
#define GLB_AS __attribute__((address_space(1)))

constexpr int NRANK = 8;
constexpr int KDIM = 2048, NDIM = 2048, MLOC = 256, NLOC = 256;
constexpr int NWG = 256;
constexpr int NW = 4;
constexpr int UNIT = 12288;
constexpr int NSLOT = 3;
constexpr int WAVE_LDS = NSLOT * UNIT + 4096;
constexpr int LDS_BYTES = NW * WAVE_LDS;
constexpr int RET = 4;
constexpr int NA = 4;
constexpr int NB = 2;
constexpr int DLY = 6, LDLY = 6, FDLY = 4;
static_assert(RET >= 3 && RET <= 5 && NA >= 4 && NA + NB <= 2 * RET, "see voff() and the counted waits");
constexpr int xoff(int u) { return u == 3 ? NSLOT * UNIT : (u % NSLOT) * UNIT + 8192; }
constexpr int voff(int v) { return v < 3 ? v * UNIT + 8192 + 2048 : NSLOT * UNIT + (v - 3) * 2048; }

constexpr int WS_SYNC = 0;
constexpr int WS_TMO = 4096;
constexpr int WS_JUNK = 8192;
constexpr int WS_CNT = 16384;
constexpr int WS_END = WS_CNT + NWG * NW * 128;
constexpr unsigned long long DEADLINE_TICKS = 50000000ull;
constexpr unsigned SENTINEL = 0xFF81FF81u;

struct Params {
    unsigned srdx[4];
    unsigned srdw[4];
    char* myws;
    char* myout;
    unsigned rank;
    unsigned pad[3];
    char* out[NRANK];
    char* ws[NRANK];
};
static_assert(offsetof(Params, srdw) == 0x10 && offsetof(Params, myws) == 0x20 && offsetof(Params, myout) == 0x28
              && offsetof(Params, rank) == 0x30 && offsetof(Params, out) == 0x40, "offsets used by the argument loads");

#define SB() __builtin_amdgcn_sched_barrier(0)
#define PIN1(a) do { asm volatile("" : "+v"(a)); SB(); } while (0)
#define PIN2(a, b) do { asm volatile("" : "+v"(a), "+v"(b)); SB(); } while (0)

template <int LOFF, int GOFF>
__device__ __forceinline__ void dma16(unsigned lds_base, unsigned voff, unsigned sbase, u32x4 srd) {
    unsigned t;
    asm volatile("s_add_u32 m0, %1, %2\n\ts_add_u32 %0, %6, %3\n\tbuffer_load_dwordx4 %4, %5, %0 offen lds"
                 : "=&s"(t) : "s"(lds_base), "i"(LOFF), "i"(GOFF), "v"(voff), "s"(srd), "s"(sbase) : "memory", "scc");
}
template <int N> __device__ __forceinline__ void wait_vm() {
    asm volatile("s_waitcnt vmcnt(%0)" :: "i"(N) : "memory");
}
__device__ __forceinline__ void lds_barrier() { asm volatile("s_waitcnt lgkmcnt(0)\n\ts_barrier" ::: "memory"); }
__device__ __forceinline__ void barriers(unsigned n) {
    for (unsigned i = 0; i < n; ++i) asm volatile("s_barrier" ::: "memory");
}
__device__ __forceinline__ void store8_sys(GLB_AS char* ptr, u32x2 v) {
    asm volatile("global_store_dwordx2 %0, %1, off sc0 sc1\n\ts_nop 1" :: "v"(ptr), "v"(v) : "memory");
}
__device__ __forceinline__ void store16_sys(GLB_AS char* ptr, u32x4 v) {
    asm volatile("global_store_dwordx4 %0, %1, off sc0 sc1\n\ts_nop 1" :: "v"(ptr), "v"(v) : "memory");
}
__device__ __forceinline__ unsigned ssel(unsigned cond, unsigned a, unsigned b) {
    unsigned r;
    asm("s_cmp_lg_u32 %1, 0\n\ts_cselect_b32 %0, %2, %3" : "=s"(r) : "s"(cond), "s"(a), "s"(b) : "scc");
    return r;
}
__device__ __forceinline__ LDS_AS char* lxor(LDS_AS char* p, unsigned v) {
    return (LDS_AS char*)(uintptr_t)((unsigned)(uintptr_t)p ^ v);
}

struct Ctx {
    unsigned lds_base;
    unsigned kbase, lmask;
    unsigned vx[2], vw;
    u32x4 sx, sw;
    LDS_AS char* ax[2];
    LDS_AS char* aw[4];
    bf16x8 wf[8][2][4];
    bf16x8 xf[8][2][2];
    bf16x8 xv[RET][2];
    f32x4 acc[2][4];
};

template <int U> __device__ __forceinline__ unsigned block_of(const Ctx& c) {
    constexpr unsigned tl = 18u * (U & 1) ^ (U >> 1), tf = 4u * U;
    return c.kbase ^ tf ^ ((tl ^ tf) & c.lmask);
}
constexpr bool full(int u) { return u < 8 - RET; }
constexpr int nloads(int u) { return (u == 2 || u == 3) ? 8 : full(u) ? 12 : 10; }
template <int U, int I, bool ALLX = false> __device__ __forceinline__ void piece(Ctx& c, unsigned kb) {
    constexpr int so = (U % NSLOT) * UNIT;
    if constexpr (U == 8) {
        dma16<voff(I / 2) + (I & 1) * 1024, (16 + (I & 1) * 8) * (KDIM * 2)>(c.lds_base, c.vx[I & 1], block_of<8 - RET + I / 2>(c) << 7, c.sx);
    } else if constexpr (I < 4) {
        if constexpr (I < 2 || full(U) || ALLX) dma16<xoff(U) + I * 1024, I * 8 * (KDIM * 2)>(c.lds_base, c.vx[I & 1], kb << 7, c.sx);
    } else dma16<so + (I - 4) * 1024, (((I - 4) >> 1) * 16 + (I & 1) * 4) * (NDIM * 2)>(c.lds_base, c.vw, kb << 18, c.sw);
}
template <int U, int K> __device__ __forceinline__ void pseq(Ctx& c, unsigned kb) {
    constexpr int order[12] = {0, 1, 4, 5, 6, 2, 7, 3, 8, 9, 10, 11};
    if constexpr (U == 2 || U == 3) {
        if constexpr (K == 0) {
            if (!c.lmask) { piece<U, 0>(c, kb); piece<U, 1>(c, kb); piece<U, 2>(c, kb); piece<U, 3>(c, kb); }
        } else if constexpr (order[K] >= 4) piece<U, order[K]>(c, kb);
    } else if constexpr (U < 8) piece<U, order[K]>(c, kb);
    else if constexpr (K < NA) piece<8, K>(c, kb);
}
template <int U> __device__ __forceinline__ void issue_x(Ctx& c, unsigned kb) {
    piece<U, 0, true>(c, kb); piece<U, 1, true>(c, kb); piece<U, 2, true>(c, kb); piece<U, 3, true>(c, kb);
}
template <int U> __device__ __forceinline__ void issue_w(Ctx& c, unsigned kb) {
    piece<U, 4>(c, kb); piece<U, 5>(c, kb); piece<U, 6>(c, kb); piece<U, 7>(c, kb);
    piece<U, 8>(c, kb); piece<U, 9>(c, kb); piece<U, 10>(c, kb); piece<U, 11>(c, kb);
}
template <int I, int E> __device__ __forceinline__ void issue_v(Ctx& c) {
    if constexpr (I < E) { piece<8, I>(c, 0); issue_v<I + 1, E>(c); }
}
template <int U, int S> __device__ __forceinline__ void read_x(Ctx& c) {
    c.xf[U][S][0] = *(LDS_AS bf16x8*)(c.ax[S] + xoff(U));
    if constexpr (full(U)) c.xf[U][S][1] = *(LDS_AS bf16x8*)(c.ax[S] + xoff(U) + 2048);
}
template <int U, int S, int T> __device__ __forceinline__ void read_w1(Ctx& c) {
    constexpr int so = (U % NSLOT) * UNIT + S * 4096;
    s16x4 lo = __builtin_amdgcn_ds_read_tr16_b64_v4i16((LDS_AS s16x4*)(c.aw[T] + so));
    s16x4 hi = __builtin_amdgcn_ds_read_tr16_b64_v4i16((LDS_AS s16x4*)(c.aw[T] + so + 1024));
    c.wf[U][S][T] = __builtin_bit_cast(bf16x8, __builtin_shufflevector(lo, hi, 0, 1, 2, 3, 4, 5, 6, 7));
}
template <int U, int S> __device__ __forceinline__ void read_w(Ctx& c) {
    read_w1<U, S, 0>(c); read_w1<U, S, 1>(c); read_w1<U, S, 2>(c); read_w1<U, S, 3>(c);
}
template <int V> __device__ __forceinline__ void read_v(Ctx& c) {
    c.xv[V][0] = *(LDS_AS bf16x8*)(c.ax[0] + voff(V));
    c.xv[V][1] = *(LDS_AS bf16x8*)(c.ax[1] + voff(V));
}
template <int U, int S, int T> __device__ __forceinline__ void mma(Ctx& c) {
    c.acc[0][T] = __builtin_amdgcn_mfma_f32_16x16x32_bf16(c.wf[U][S][T], c.xf[U][S][0], c.acc[0][T], 0, 0, 0);
    if constexpr (full(U))
        c.acc[1][T] = __builtin_amdgcn_mfma_f32_16x16x32_bf16(c.wf[U][S][T], c.xf[U][S][1], c.acc[1][T], 0, 0, 0);
    SB();
}
template <int V, int S, int T> __device__ __forceinline__ void mv(Ctx& c) {
    c.acc[1][T] = __builtin_amdgcn_mfma_f32_16x16x32_bf16(c.wf[8 - RET + V][S][T], c.xv[V][S], c.acc[1][T], 0, 0, 0);
    SB();
}
template <int V> __device__ __forceinline__ void pass2(Ctx& c) {
    if constexpr (V < RET) {
        if constexpr (V + 1 < RET) {
            wait_vm<2 * RET - 2 * (V + 2) + 1>();
            read_v<V + 1>(c);
            SB();
        }
        mv<V, 0, 0>(c); mv<V, 0, 1>(c); mv<V, 0, 2>(c); mv<V, 0, 3>(c);
        mv<V, 1, 0>(c); mv<V, 1, 1>(c); mv<V, 1, 2>(c); mv<V, 1, 3>(c);
        pass2<V + 1>(c);
    }
}
template <int U> __device__ __forceinline__ void step(Ctx& c) {
    constexpr int N = U + 2;
    const unsigned kb = block_of<N>(c);
    pseq<N, 0>(c, kb); read_x<U, 0>(c); read_w1<U, 0, 0>(c); SB();
    pseq<N, 1>(c, kb); read_w1<U, 0, 1>(c); read_w1<U, 0, 2>(c); SB();
    pseq<N, 2>(c, kb); read_w1<U, 0, 3>(c); read_x<U, 1>(c); SB();
    pseq<N, 3>(c, kb); read_w1<U, 1, 0>(c); read_w1<U, 1, 1>(c); SB();
    pseq<N, 4>(c, kb); read_w1<U, 1, 2>(c); read_w1<U, 1, 3>(c); SB();
    mma<U, 0, 0>(c); pseq<N, 5>(c, kb); SB();
    mma<U, 0, 1>(c); pseq<N, 6>(c, kb); SB();
    mma<U, 0, 2>(c); pseq<N, 7>(c, kb); SB();
    mma<U, 0, 3>(c); pseq<N, 8>(c, kb); SB();
    mma<U, 1, 0>(c); pseq<N, 9>(c, kb); SB();
    mma<U, 1, 1>(c); pseq<N, 10>(c, kb); SB();
    mma<U, 1, 2>(c); pseq<N, 11>(c, kb); SB();
    mma<U, 1, 3>(c);
}

__device__ __forceinline__ unsigned pack2(float a, float b) {
    bf16x2 v = {(__bf16)a, (__bf16)b};
    return __builtin_bit_cast(unsigned, v);
}
__device__ __forceinline__ bool expired(const Params& p, unsigned seq, unsigned long long t0) {
    unsigned* tmo = (unsigned*)(p.myws + WS_TMO);
    if (__hip_atomic_load(tmo, __ATOMIC_RELAXED, __HIP_MEMORY_SCOPE_AGENT) == seq
        || __builtin_amdgcn_s_memrealtime() - t0 > DEADLINE_TICKS) {
        __hip_atomic_store(tmo, seq, __ATOMIC_RELAXED, __HIP_MEMORY_SCOPE_AGENT);
        __hip_atomic_store((unsigned short*)(p.myout + (size_t)((p.rank & 7) * MLOC * NLOC) * 2),
                           (unsigned short)0x7FC0, __ATOMIC_RELAXED, __HIP_MEMORY_SCOPE_SYSTEM);
        return true;
    }
    return false;
}
__device__ __forceinline__ unsigned poll_arrived(const GLB_AS char* ptr) {
    unsigned n;
    unsigned long long t;
#define POLL_LD(r) "global_load_dwordx4 v[" r "], %2, off sc0 sc1\n\t"
#define POLL_CK(a, b, c, d) \
    "s_waitcnt vmcnt(2)\n\t" \
    "v_cmp_eq_u32 vcc, %3, v" a "\n\tv_cmp_eq_u32 %1, %3, v" b "\n\ts_or_b64 %1, %1, vcc\n\t" \
    "v_cmp_eq_u32 vcc, %3, v" c "\n\ts_or_b64 %1, %1, vcc\n\t" \
    "v_cmp_eq_u32 vcc, %3, v" d "\n\ts_or_b64 %1, %1, vcc\n\ts_cbranch_scc0 2f\n\t"
    asm volatile(POLL_LD("0:3") POLL_LD("4:7") POLL_LD("8:11")
                 "s_movk_i32 %0, 64\n"
                 "1:\n\t"
                 POLL_CK("0", "1", "2", "3") POLL_LD("0:3")
                 POLL_CK("4", "5", "6", "7") POLL_LD("4:7")
                 POLL_CK("8", "9", "10", "11") POLL_LD("8:11")
                 "s_sub_u32 %0, %0, 1\n\ts_cmp_lg_u32 %0, 0\n\ts_cbranch_scc1 1b\n"
                 "2:\n\t"
                 "s_waitcnt vmcnt(0)"
                 : "=&s"(n), "=&s"(t) : "v"(ptr), "s"(SENTINEL)
                 : "memory", "scc", "vcc", "v0", "v1", "v2", "v3", "v4", "v5", "v6", "v7", "v8", "v9", "v10", "v11");
    return n;
}

__global__ __launch_bounds__(512) void gemm_a2a(Params p) {
    __shared__ __attribute__((aligned(1024))) char smem[LDS_BYTES];
    Ctx c;
    const unsigned b = blockIdx.x;
    const unsigned j = b & 7, mt = (b >> 3) & 7, nt = (b >> 6) & 3;
    const unsigned peer = (j & 1) * 4 + nt;
    u32x4 kws;
    unsigned rank;
    GLB_AS char* outd;
    {
        const void* ka = (const void*)__builtin_amdgcn_kernarg_segment_ptr();
        asm volatile("s_load_dwordx4 %0, %5, 0x0\n\ts_load_dwordx4 %1, %5, 0x10\n\ts_load_dwordx4 %2, %5, 0x20\n\t"
                     "s_load_dword %3, %5, 0x30\n\ts_load_dwordx2 %4, %5, %6 offset:0x40"
                     : "=&s"(c.sx), "=&s"(c.sw), "=&s"(kws), "=&s"(rank), "=&s"(outd) : "s"(ka), "s"(peer * 8));
        SB();
    }
    const unsigned lane = threadIdx.x & 63;
    const unsigned wid = __builtin_amdgcn_readfirstlane(threadIdx.x >> 6);
    const unsigned sub = (j >> 1) ^ nt;
    LDS_AS char* lds = (LDS_AS char*)smem;
    c.lds_base = (unsigned)(uintptr_t)lds + wid * WAVE_LDS;
    c.lmask = ((mt ^ nt) & 1) - 1;
    const unsigned cs = (mt >> 1) ^ (nt & 2);
    c.kbase = ssel(c.lmask, 4 * (wid + 1) + (cs ^ ((wid + 1) >> 1)), wid ^ cs) ^ ((j & 1) * 16 + (j & 2) * 4);
    const unsigned g = lane >> 4, m = lane & 15;
    c.vx[0] = (mt * 32 + (lane >> 3)) * (KDIM * 2) + (((lane & 7) ^ g) << 4);
    c.vx[1] = c.vx[0] ^ 64;
    const unsigned kb0 = block_of<0>(c), kb1 = block_of<1>(c);
    const unsigned nsl = ssel(c.lmask, wid * DLY + (((peer * 4 + sub) & 7) == 3 ? 0 : LDLY), wid * FDLY);
    asm volatile("" : "+v"(c.vx[0]), "+v"(c.vx[1]));
    asm volatile("s_waitcnt lgkmcnt(0)" : "+s"(c.sx), "+s"(c.sw), "+s"(kws), "+s"(rank), "+s"(outd));
    rank &= 7;
    GLB_AS char* const myws = (GLB_AS char*)(((unsigned long long)kws[1] << 32) | kws[0]);
    GLB_AS char* const myout = (GLB_AS char*)(((unsigned long long)kws[3] << 32) | kws[2]);
    unsigned warm = 0;
    if (__builtin_expect(nsl != 0, 0)) {
        if (wid == 3) {
            const unsigned long long wp = ((unsigned long long)c.sw[1] << 32) | c.sw[0];
            asm volatile("s_load_dword %0, %1, %2" : "+s"(warm) : "s"(outd), "s"(peer << 17) : "memory");
            asm volatile("s_load_dword %0, %1, %2" : "+s"(warm) : "s"(myout), "s"(rank << 17) : "memory");
            asm volatile("s_load_dword %0, %1, %2" : "+s"(warm) : "s"(myws), "i"(WS_TMO + 256) : "memory");
            asm volatile("s_load_dword %0, %1, %2" : "+s"(warm) : "s"(wp), "s"(mt << 20) : "memory");
        }
        for (unsigned i = 0; i < nsl; i += 2) __builtin_amdgcn_s_sleep(1);
    }
    issue_x<0>(c, kb0);
    c.vw = (((lane >> 3) & 3) + (lane >> 5) * 8) * (NDIM * 2) + (peer * 4 + sub) * 128 + (((lane & 7) ^ (g << 1)) << 4);
    issue_w<0>(c, kb0);
    if (c.lmask) issue_x<2>(c, kb0 ^ 1);
    barriers(1 + (wid & c.lmask));
    const unsigned arm = wid * 16 + (j >> 1) * 4 + (mt >> 1);
    const bool armer = ((peer ^ rank) | c.lmask) == 0 && (arm >> 3) != rank;
    if (__builtin_expect(armer, 0)) {
        GLB_AS char* fp = myout + (size_t)arm * (32 * NLOC * 2) + lane * 16;
        const u32x4 sv = {SENTINEL, SENTINEL, SENTINEL, SENTINEL};
#pragma unroll
        for (int i = 0; i < 16; ++i) store16_sys(fp + i * 1024, sv);
    }
    GLB_AS unsigned* cnt = (GLB_AS unsigned*)(myws + WS_CNT + (b * NW + wid) * 128);
    unsigned seq;
    asm volatile("global_load_dword %0, %1, %2 sc1" : "=v"(seq) : "v"(0u), "s"(cnt) : "memory");
    LDS_AS char* ring = lds + wid * WAVE_LDS;
    const unsigned q = m >> 2;
    piece<1, 4>(c, kb1); piece<1, 5>(c, kb1); SB();
    c.ax[0] = ring + m * 128 + ((g ^ (m >> 1)) << 4);
    c.ax[1] = lxor(c.ax[0], 64);
    asm volatile("" : "+v"(c.ax[0]), "+v"(c.ax[1]));
    piece<1, 6>(c, kb1); SB();
    c.aw[0] = ring + (g >> 1) * 2048 + (4 * (g & 1) + q) * 128 + 8 * (lane & 3);
    asm volatile("" : "+v"(c.aw[0]));
    piece<1, 7>(c, kb1); SB();
    c.aw[0] += (2 * (g & 1) + (q >> 1)) << 5;
#pragma unroll
    for (int t = 1; t < 4; ++t) c.aw[t] = lxor(c.aw[0], t << 5);
    asm volatile("" : "+v"(c.aw[0]), "+v"(c.aw[1]), "+v"(c.aw[2]), "+v"(c.aw[3]));
    piece<1, 8>(c, kb1); piece<1, 9>(c, kb1); piece<1, 10>(c, kb1); piece<1, 11>(c, kb1);
    issue_x<1>(c, kb1);
    if (c.lmask) issue_x<3>(c, kb1 ^ 1);
    barriers((4 - wid) & c.lmask);
#pragma unroll
    for (int h = 0; h < 2; ++h)
#pragma unroll
        for (int t = 0; t < 4; ++t) c.acc[h][t] = f32x4{0.f, 0.f, 0.f, 0.f};

    if (c.lmask) wait_vm<17>();
    else wait_vm<13>();
    if (__builtin_expect(armer, 0)) {
        asm volatile("s_waitcnt vmcnt(12)" : "+v"(seq) :: "memory");
        if (lane == 0) {
            __hip_atomic_store((unsigned*)(p.ws[(arm >> 3) & 7] + WS_SYNC + (rank * 8 + (arm & 7)) * 8), seq + 1,
                               __ATOMIC_RELAXED, __HIP_MEMORY_SCOPE_SYSTEM);
            __hip_atomic_store((GLB_AS unsigned*)(myws + WS_SYNC + arm * 8 + 4), seq + 1,
                               __ATOMIC_RELAXED, __HIP_MEMORY_SCOPE_SYSTEM);
        }
    }
    asm volatile("s_waitcnt lgkmcnt(0)" : "+s"(warm) :: "memory");
    step<0>(c);
    asm volatile("s_waitcnt vmcnt(12)" : "+v"(seq) :: "memory");
    seq += 1;
    step<1>(c); wait_vm<nloads(3)>();
    step<2>(c); wait_vm<nloads(4)>();
    step<3>(c); wait_vm<nloads(5)>();
    step<4>(c); wait_vm<nloads(6)>();
    {
        const unsigned kb7 = block_of<7>(c);
        piece<7, 0>(c, kb7); read_x<5, 0>(c); read_x<5, 1>(c); SB();
        piece<7, 1>(c, kb7); read_w1<5, 0, 0>(c); SB();
        piece<7, 4>(c, kb7); read_w1<5, 0, 1>(c); SB();
        piece<7, 5>(c, kb7); read_w1<5, 0, 2>(c); SB();
        piece<7, 6>(c, kb7); read_w1<5, 0, 3>(c); SB();
        piece<7, 7>(c, kb7); read_w1<5, 1, 0>(c); SB();
        piece<7, 8>(c, kb7); read_w1<5, 1, 1>(c); SB();
        piece<7, 9>(c, kb7); read_w1<5, 1, 2>(c); SB();
        piece<7, 10>(c, kb7); read_w1<5, 1, 3>(c); SB();
        piece<7, 11>(c, kb7); SB();
    }
    LDS_AS char* pw0 = ring + m * 256 + ((g ^ (m & 7)) << 4);
    LDS_AS char* pw1 = lxor(pw0, 64);
    const unsigned row = wid * 4 + g;
    LDS_AS char* pr0 = lds + row * 256 + ((m ^ (row & 7)) << 4);
    LDS_AS char* pr2 = pr0 + 2 * WAVE_LDS;
    GLB_AS char* optr = outd + (size_t)((rank * MLOC + mt * 32 + row) * NLOC + sub * 64 + m * 4) * 2;
    GLB_AS char* optr2 = optr + 16 * NLOC * 2;
    const GLB_AS char* syncp = myws + WS_SYNC + (peer * 8 + mt) * 8;
    u32x2 sy;
    __builtin_amdgcn_s_barrier();
    asm volatile("global_load_dwordx2 %0, %1, %2 sc0 sc1" : "=v"(sy) : "v"(0u), "s"(syncp) : "memory");
    mma<5, 0, 0>(c); mma<5, 0, 1>(c);
    wait_vm<nloads(7) + 1>();
    read_x<6, 0>(c); read_x<6, 1>(c); read_w1<6, 0, 0>(c); SB(); mma<5, 0, 2>(c); pseq<8, 0>(c, 0); SB();
    read_w1<6, 0, 1>(c); read_w1<6, 0, 2>(c); SB(); mma<5, 0, 3>(c); pseq<8, 1>(c, 0); SB();
    read_w1<6, 0, 3>(c); read_w1<6, 1, 0>(c); SB(); mma<5, 1, 0>(c); pseq<8, 2>(c, 0); SB();
    read_w1<6, 1, 1>(c); read_w1<6, 1, 2>(c); SB(); mma<5, 1, 1>(c); pseq<8, 3>(c, 0); SB();
    read_w1<6, 1, 3>(c); SB(); mma<5, 1, 2>(c); mma<5, 1, 3>(c); pseq<8, 4>(c, 0); SB();
    mma<6, 0, 0>(c); mma<6, 0, 1>(c); pseq<8, 5>(c, 0); SB();
    mma<6, 0, 2>(c); mma<6, 0, 3>(c); pseq<8, 6>(c, 0); pseq<8, 7>(c, 0); SB();
    asm volatile("" : "+v"(pw0), "+v"(pw1), "+v"(pr0), "+v"(pr2), "+v"(optr), "+v"(optr2));
    wait_vm<4 + 1 + NA>();
    read_x<7, 0>(c); read_w<7, 0>(c); read_x<7, 1>(c);
    SB();
    mma<6, 1, 0>(c); mma<6, 1, 1>(c); mma<6, 1, 2>(c); mma<6, 1, 3>(c);
    wait_vm<1 + NA>();
    read_w<7, 1>(c);
    SB();
    mma<7, 0, 0>(c); mma<7, 0, 1>(c); mma<7, 0, 2>(c); mma<7, 0, 3>(c);
    mma<7, 1, 0>(c); mma<7, 1, 1>(c); mma<7, 1, 2>(c);
    *(LDS_AS f32x4*)(pw0) = c.acc[0][0];
    SB();
    mma<7, 1, 3>(c);
    *(LDS_AS f32x4*)(pw1) = c.acc[0][1];
    *(LDS_AS f32x4*)(pw0 + 128) = c.acc[0][2];
    *(LDS_AS f32x4*)(pw1 + 128) = c.acc[0][3];
    SB();
    issue_v<NA, NA + NB>(c);
    lds_barrier();
    const f32x4 a0 = *(LDS_AS f32x4*)(pr0), a1 = *(LDS_AS f32x4*)(pr0 + WAVE_LDS);
    const f32x4 a2 = *(LDS_AS f32x4*)(pr2), a3 = *(LDS_AS f32x4*)(pr2 + WAVE_LDS);
    SB();
    asm volatile("s_waitcnt vmcnt(%1)" : "+v"(sy) : "i"(NA + NB - 4) : "memory");
    read_v<0>(c); read_v<1>(c);
    SB();
    issue_v<NA + NB, 2 * RET>(c);
    bool ok = true;
    if (__builtin_expect(peer != rank && __builtin_amdgcn_readfirstlane(sy[0] < seq || sy[1] < seq), 0)) {
        const unsigned long long t0 = __builtin_amdgcn_s_memrealtime();
        for (unsigned spins = 1;; ++spins) {
            asm volatile("global_load_dwordx2 %0, %1, %2 sc0 sc1\n\ts_waitcnt vmcnt(0)" : "=&v"(sy) : "v"(0u), "s"(syncp) : "memory");
            if (__builtin_amdgcn_readfirstlane(sy[0] >= seq && sy[1] >= seq)) break;
            if ((spins & 63) == 0 && __builtin_amdgcn_readfirstlane(expired(p, seq, t0))) {
                ok = false;
                optr = optr2 = myws + WS_JUNK + threadIdx.x * 8;
                break;
            }
            __builtin_amdgcn_s_sleep(1);
        }
    }
    {
        f32x4 y = a0 + a1, z = a2 + a3, e;
        PIN2(y, z);
        mv<0, 0, 0>(c);
        y += z;
        z[0] = __builtin_amdgcn_fmed3f(y[0], -60.0f, 60.0f); z[1] = __builtin_amdgcn_fmed3f(y[1], -60.0f, 60.0f);
        PIN2(y, z);
        mv<0, 0, 1>(c);
        z[2] = __builtin_amdgcn_fmed3f(y[2], -60.0f, 60.0f); z[3] = __builtin_amdgcn_fmed3f(y[3], -60.0f, 60.0f);
        z *= -1.4426950408889634f;
        PIN1(z);
        mv<0, 0, 2>(c);
        e[0] = __builtin_amdgcn_exp2f(z[0]); e[1] = __builtin_amdgcn_exp2f(z[1]); e[2] = z[2]; e[3] = z[3];
        PIN1(e);
        mv<0, 0, 3>(c);
        e[2] = __builtin_amdgcn_exp2f(e[2]); e[3] = __builtin_amdgcn_exp2f(e[3]);
        PIN1(e);
        mv<0, 1, 0>(c);
        e += 1.0f;
        e[0] = __builtin_amdgcn_rcpf(e[0]);
        PIN1(e);
        mv<0, 1, 1>(c);
        e[1] = __builtin_amdgcn_rcpf(e[1]); e[2] = __builtin_amdgcn_rcpf(e[2]);
        PIN1(e);
        mv<0, 1, 2>(c);
        e[3] = __builtin_amdgcn_rcpf(e[3]);
        e *= y;
        store8_sys(optr, u32x2{pack2(e[0], e[1]), pack2(e[2], e[3])});
        SB();
    }
    mv<0, 1, 3>(c);
    pass2<1>(c);
    *(LDS_AS f32x4*)(pw0 + UNIT) = c.acc[1][0];
    *(LDS_AS f32x4*)(pw1 + UNIT) = c.acc[1][1];
    *(LDS_AS f32x4*)(pw0 + UNIT + 128) = c.acc[1][2];
    *(LDS_AS f32x4*)(pw1 + UNIT + 128) = c.acc[1][3];
    lds_barrier();
    {
        const f32x4 b0 = *(LDS_AS f32x4*)(pr0 + UNIT), b1 = *(LDS_AS f32x4*)(pr0 + UNIT + WAVE_LDS);
        const f32x4 b2 = *(LDS_AS f32x4*)(pr2 + UNIT), b3 = *(LDS_AS f32x4*)(pr2 + UNIT + WAVE_LDS);
        SB();
        const f32x4 y = (b0 + b1) + (b2 + b3);
        f32x4 e;
#pragma unroll
        for (int i = 0; i < 4; ++i) e[i] = __builtin_amdgcn_fmed3f(y[i], -60.0f, 60.0f);
        e *= -1.4426950408889634f;
        PIN1(e);
#pragma unroll
        for (int i = 0; i < 4; ++i) e[i] = __builtin_amdgcn_exp2f(e[i]);
        PIN1(e);
        e += 1.0f;
        PIN1(e);
#pragma unroll
        for (int i = 0; i < 4; ++i) e[i] = __builtin_amdgcn_rcpf(e[i]);
        PIN1(e);
        e *= y;
        store8_sys(optr2, u32x2{pack2(e[0], e[1]), pack2(e[2], e[3])});
    }
    if (lane == 0) __hip_atomic_store(cnt, seq, __ATOMIC_RELAXED, __HIP_MEMORY_SCOPE_AGENT);
    if (peer != rank && ok) {
        const GLB_AS char* mine = myout + (size_t)((peer * MLOC + mt * 32 + wid * 8 + (lane >> 3)) * NLOC + sub * 64 + (lane & 7) * 8) * 2;
        if (__builtin_expect(!poll_arrived(mine), 0)) {
            const unsigned long long t0 = __builtin_amdgcn_s_memrealtime();
            while (!poll_arrived(mine))
                if (__builtin_amdgcn_readfirstlane(expired(p, seq, t0))) break;
        }
    }
}

extern "C" void kernel_launch(int rank, int nranks, ncclComm_t comm,
                              hipStream_t stream,
                              void* const* d_in, const int* in_sizes, int n_in,
                              void* d_out, int out_size,
                              const int* in_dtypes, int out_dtype,
                              void* const* d_out_all,
                              void* const* d_ws_all, size_t ws_bytes) {
    if (nranks != NRANK || (unsigned)rank >= (unsigned)NRANK || n_in < 2 || in_sizes[0] != MLOC * KDIM
        || in_sizes[1] != KDIM * NDIM || out_size != NRANK * MLOC * NLOC || ws_bytes < (size_t)WS_END) {
        fprintf(stderr, "kernel_launch: unexpected problem shape\n");
        return;
    }
    Params p;
    const uintptr_t ax = (uintptr_t)d_in[0], aw = (uintptr_t)d_in[1];
    p.srdx[0] = (unsigned)ax; p.srdx[1] = (unsigned)(ax >> 32) & 0xffffu; p.srdx[2] = MLOC * KDIM * 2; p.srdx[3] = 0x00020000u;
    p.srdw[0] = (unsigned)aw; p.srdw[1] = (unsigned)(aw >> 32) & 0xffffu; p.srdw[2] = KDIM * NDIM * 2; p.srdw[3] = 0x00020000u;
    p.myws = (char*)d_ws_all[rank];
    p.myout = (char*)d_out;
    p.rank = (unsigned)rank;
    p.pad[0] = p.pad[1] = p.pad[2] = 0;
    for (int i = 0; i < NRANK; ++i) {
        p.out[i] = (char*)d_out_all[i];
        p.ws[i] = (char*)d_ws_all[i];
    }
    gemm_a2a<<<dim3(NWG), dim3(NW * 64), 0, stream>>>(p);
    HIP_CHECK(hipGetLastError());
}
```

```cpp
#include <hip/hip_runtime.h>
#include <rccl/rccl.h>
#include <cstddef>
#include <cstdint>
#include <cstdio>

#define HIP_CHECK(expr) do { hipError_t _e = (expr); if (_e != hipSuccess) { \
    fprintf(stderr, "HIP error %s at %s:%d\n", hipGetErrorString(_e), __FILE__, __LINE__); \
    return; } } while (0)

typedef __attribute__((ext_vector_type(8))) __bf16 bf16x8;
typedef __attribute__((ext_vector_type(2))) __bf16 bf16x2;
typedef __attribute__((ext_vector_type(4))) short s16x4;
typedef __attribute__((ext_vector_type(4))) float f32x4;
typedef __attribute__((ext_vector_type(4))) unsigned u32x4;
typedef __attribute__((ext_vector_type(2))) unsigned u32x2;
#define LDS_AS __attribute__((address_space(3)))
#define GLB_AS __attribute__((address_space(1)))

constexpr int NRANK = 8;
constexpr int KDIM = 2048, NDIM = 2048, MLOC = 256, NLOC = 256;
constexpr int NWG = 256;
constexpr int NW = 4;
constexpr int UNIT = 12288;
constexpr int NSLOT = 3;
constexpr int WAVE_LDS = NSLOT * UNIT + 4096;
constexpr int LDS_BYTES = NW * WAVE_LDS;
constexpr int RET = 4;
constexpr int NA = 4;
constexpr int NB = 2;
constexpr int DLY = 6, LDLY = 6, FDLY = 4;
static_assert(RET >= 3 && RET <= 5 && NA >= 4 && NA + NB <= 2 * RET, "see voff() and the counted waits");
constexpr int xoff(int u) { return u == 3 ? NSLOT * UNIT : (u % NSLOT) * UNIT + 8192; }
constexpr int voff(int v) { return v < 3 ? v * UNIT + 8192 + 2048 : NSLOT * UNIT + (v - 3) * 2048; }

constexpr int WS_SYNC = 0;
constexpr int WS_TMO = 4096;
constexpr int WS_JUNK = 8192;
constexpr int WS_CNT = 16384;
constexpr int WS_END = WS_CNT + NWG * NW * 128;
constexpr unsigned long long DEADLINE_TICKS = 50000000ull;
constexpr unsigned SENTINEL = 0xFF81FF81u;

struct Params {
    unsigned srdx[4];
    unsigned srdw[4];
    char* myws;
    char* myout;
    unsigned rank;
    unsigned pad[3];
    char* out[NRANK];
    char* ws[NRANK];
};
static_assert(offsetof(Params, srdw) == 0x10 && offsetof(Params, myws) == 0x20 && offsetof(Params, myout) == 0x28
              && offsetof(Params, rank) == 0x30 && offsetof(Params, out) == 0x40, "offsets used by the argument loads");

#define SB() __builtin_amdgcn_sched_barrier(0)
#define PIN1(a) do { asm volatile("" : "+v"(a)); SB(); } while (0)
#define PIN2(a, b) do { asm volatile("" : "+v"(a), "+v"(b)); SB(); } while (0)

template <int LOFF, int GOFF>
__device__ __forceinline__ void dma16(unsigned lds_base, unsigned voff, unsigned sbase, u32x4 srd) {
    unsigned t;
    asm volatile("s_add_u32 m0, %1, %2\n\ts_add_u32 %0, %6, %3\n\tbuffer_load_dwordx4 %4, %5, %0 offen lds"
                 : "=&s"(t) : "s"(lds_base), "i"(LOFF), "i"(GOFF), "v"(voff), "s"(srd), "s"(sbase) : "memory", "scc");
}
template <int N> __device__ __forceinline__ void wait_vm() {
    asm volatile("s_waitcnt vmcnt(%0)" :: "i"(N) : "memory");
}
__device__ __forceinline__ void lds_barrier() { asm volatile("s_waitcnt lgkmcnt(0)\n\ts_barrier" ::: "memory"); }
__device__ __forceinline__ void barriers(unsigned n) {
    for (unsigned i = 0; i < n; ++i) asm volatile("s_barrier" ::: "memory");
}
__device__ __forceinline__ void store8_sys(GLB_AS char* ptr, u32x2 v) {
    asm volatile("global_store_dwordx2 %0, %1, off sc0 sc1\n\ts_nop 1" :: "v"(ptr), "v"(v) : "memory");
}
__device__ __forceinline__ void store16_sys(GLB_AS char* ptr, u32x4 v) {
    asm volatile("global_store_dwordx4 %0, %1, off sc0 sc1\n\ts_nop 1" :: "v"(ptr), "v"(v) : "memory");
}
__device__ __forceinline__ unsigned ssel(unsigned cond, unsigned a, unsigned b) {
    unsigned r;
    asm("s_cmp_lg_u32 %1, 0\n\ts_cselect_b32 %0, %2, %3" : "=s"(r) : "s"(cond), "s"(a), "s"(b) : "scc");
    return r;
}
__device__ __forceinline__ LDS_AS char* lxor(LDS_AS char* p, unsigned v) {
    return (LDS_AS char*)(uintptr_t)((unsigned)(uintptr_t)p ^ v);
}

struct Ctx {
    unsigned lds_base;
    unsigned kbase, lmask;
    unsigned vx[2], vw;
    u32x4 sx, sw;
    LDS_AS char* ax[2];
    LDS_AS char* aw[4];
    bf16x8 wf[8][2][4];
    bf16x8 xf[8][2][2];
    bf16x8 xv[RET][2];
    f32x4 acc[2][4];
};

template <int U> __device__ __forceinline__ unsigned block_of(const Ctx& c) {
    constexpr unsigned tl = 18u * (U & 1) ^ (U >> 1), tf = 4u * U;
    return c.kbase ^ tf ^ ((tl ^ tf) & c.lmask);
}
constexpr bool full(int u) { return u < 8 - RET; }
constexpr int nloads(int u) { return (u == 2 || u == 3) ? 8 : full(u) ? 12 : 10; }
template <int U, int I, bool ALLX = false> __device__ __forceinline__ void piece(Ctx& c, unsigned kb) {
    constexpr int so = (U % NSLOT) * UNIT;
    if constexpr (U == 8) {
        dma16<voff(I / 2) + (I & 1) * 1024, (16 + (I & 1) * 8) * (KDIM * 2)>(c.lds_base, c.vx[I & 1], block_of<8 - RET + I / 2>(c) << 7, c.sx);
    } else if constexpr (I < 4) {
        if constexpr (I < 2 || full(U) || ALLX) dma16<xoff(U) + I * 1024, I * 8 * (KDIM * 2)>(c.lds_base, c.vx[I & 1], kb << 7, c.sx);
    } else dma16<so + (I - 4) * 1024, (((I - 4) >> 1) * 16 + (I & 1) * 4) * (NDIM * 2)>(c.lds_base, c.vw, kb << 18, c.sw);
}
template <int U, int K> __device__ __forceinline__ void pseq(Ctx& c, unsigned kb) {
    constexpr int order[12] = {0, 1, 4, 5, 6, 2, 7, 3, 8, 9, 10, 11};
    if constexpr (U == 2 || U == 3) {
        if constexpr (K == 0) {
            if (!c.lmask) { piece<U, 0>(c, kb); piece<U, 1>(c, kb); piece<U, 2>(c, kb); piece<U, 3>(c, kb); }
        } else if constexpr (order[K] >= 4) piece<U, order[K]>(c, kb);
    } else if constexpr (U < 8) piece<U, order[K]>(c, kb);
    else if constexpr (K < NA) piece<8, K>(c, kb);
}
template <int U> __device__ __forceinline__ void issue_x(Ctx& c, unsigned kb) {
    piece<U, 0, true>(c, kb); piece<U, 1, true>(c, kb); piece<U, 2, true>(c, kb); piece<U, 3, true>(c, kb);
}
template <int U> __device__ __forceinline__ void issue_w(Ctx& c, unsigned kb) {
    piece<U, 4>(c, kb); piece<U, 5>(c, kb); piece<U, 6>(c, kb); piece<U, 7>(c, kb);
    piece<U, 8>(c, kb); piece<U, 9>(c, kb); piece<U, 10>(c, kb); piece<U, 11>(c, kb);
}
template <int I, int E> __device__ __forceinline__ void issue_v(Ctx& c) {
    if constexpr (I < E) { piece<8, I>(c, 0); issue_v<I + 1, E>(c); }
}
template <int U, int S> __device__ __forceinline__ void read_x(Ctx& c) {
    c.xf[U][S][0] = *(LDS_AS bf16x8*)(c.ax[S] + xoff(U));
    if constexpr (full(U)) c.xf[U][S][1] = *(LDS_AS bf16x8*)(c.ax[S] + xoff(U) + 2048);
}
template <int U, int S, int T> __device__ __forceinline__ void read_w1(Ctx& c) {
    constexpr int so = (U % NSLOT) * UNIT + S * 4096;
    s16x4 lo = __builtin_amdgcn_ds_read_tr16_b64_v4i16((LDS_AS s16x4*)(c.aw[T] + so));
    s16x4 hi = __builtin_amdgcn_ds_read_tr16_b64_v4i16((LDS_AS s16x4*)(c.aw[T] + so + 1024));
    c.wf[U][S][T] = __builtin_bit_cast(bf16x8, __builtin_shufflevector(lo, hi, 0, 1, 2, 3, 4, 5, 6, 7));
}
template <int U, int S> __device__ __forceinline__ void read_w(Ctx& c) {
    read_w1<U, S, 0>(c); read_w1<U, S, 1>(c); read_w1<U, S, 2>(c); read_w1<U, S, 3>(c);
}
template <int V> __device__ __forceinline__ void read_v(Ctx& c) {
    c.xv[V][0] = *(LDS_AS bf16x8*)(c.ax[0] + voff(V));
    c.xv[V][1] = *(LDS_AS bf16x8*)(c.ax[1] + voff(V));
}
template <int U, int S, int T> __device__ __forceinline__ void mma(Ctx& c) {
    c.acc[0][T] = __builtin_amdgcn_mfma_f32_16x16x32_bf16(c.wf[U][S][T], c.xf[U][S][0], c.acc[0][T], 0, 0, 0);
    if constexpr (full(U))
        c.acc[1][T] = __builtin_amdgcn_mfma_f32_16x16x32_bf16(c.wf[U][S][T], c.xf[U][S][1], c.acc[1][T], 0, 0, 0);
    SB();
}
template <int V, int S, int T> __device__ __forceinline__ void mv(Ctx& c) {
    c.acc[1][T] = __builtin_amdgcn_mfma_f32_16x16x32_bf16(c.wf[8 - RET + V][S][T], c.xv[V][S], c.acc[1][T], 0, 0, 0);
    SB();
}
template <int V> __device__ __forceinline__ void pass2(Ctx& c) {
    if constexpr (V < RET) {
        if constexpr (V + 1 < RET) {
            wait_vm<2 * RET - 2 * (V + 2) + 1>();
            read_v<V + 1>(c);
            SB();
        }
        mv<V, 0, 0>(c); mv<V, 0, 1>(c); mv<V, 0, 2>(c); mv<V, 0, 3>(c);
        mv<V, 1, 0>(c); mv<V, 1, 1>(c); mv<V, 1, 2>(c); mv<V, 1, 3>(c);
        pass2<V + 1>(c);
    }
}
template <int U> __device__ __forceinline__ void step(Ctx& c) {
    constexpr int N = U + 2;
    const unsigned kb = block_of<N>(c);
    pseq<N, 0>(c, kb); read_x<U, 0>(c); read_w1<U, 0, 0>(c); SB();
    pseq<N, 1>(c, kb); read_w1<U, 0, 1>(c); read_w1<U, 0, 2>(c); SB();
    pseq<N, 2>(c, kb); read_w1<U, 0, 3>(c); read_x<U, 1>(c); SB();
    pseq<N, 3>(c, kb); read_w1<U, 1, 0>(c); read_w1<U, 1, 1>(c); SB();
    pseq<N, 4>(c, kb); read_w1<U, 1, 2>(c); read_w1<U, 1, 3>(c); SB();
    mma<U, 0, 0>(c); pseq<N, 5>(c, kb); SB();
    mma<U, 0, 1>(c); pseq<N, 6>(c, kb); SB();
    mma<U, 0, 2>(c); pseq<N, 7>(c, kb); SB();
    mma<U, 0, 3>(c); pseq<N, 8>(c, kb); SB();
    mma<U, 1, 0>(c); pseq<N, 9>(c, kb); SB();
    mma<U, 1, 1>(c); pseq<N, 10>(c, kb); SB();
    mma<U, 1, 2>(c); pseq<N, 11>(c, kb); SB();
    mma<U, 1, 3>(c);
}

__device__ __forceinline__ unsigned pack2(float a, float b) {
    bf16x2 v = {(__bf16)a, (__bf16)b};
    return __builtin_bit_cast(unsigned, v);
}
__device__ __forceinline__ bool expired(const Params& p, unsigned seq, unsigned long long t0) {
    unsigned* tmo = (unsigned*)(p.myws + WS_TMO);
    if (__hip_atomic_load(tmo, __ATOMIC_RELAXED, __HIP_MEMORY_SCOPE_AGENT) == seq
        || __builtin_amdgcn_s_memrealtime() - t0 > DEADLINE_TICKS) {
        __hip_atomic_store(tmo, seq, __ATOMIC_RELAXED, __HIP_MEMORY_SCOPE_AGENT);
        __hip_atomic_store((unsigned short*)(p.myout + (size_t)((p.rank & 7) * MLOC * NLOC) * 2),
                           (unsigned short)0x7FC0, __ATOMIC_RELAXED, __HIP_MEMORY_SCOPE_SYSTEM);
        return true;
    }
    return false;
}
__device__ __forceinline__ unsigned poll_arrived(const GLB_AS char* ptr) {
    unsigned n;
    unsigned long long t;
#define POLL_LD(r) "global_load_dwordx4 v[" r "], %2, off sc0 sc1\n\t"
#define POLL_CK(a, b, c, d) \
    "s_waitcnt vmcnt(2)\n\t" \
    "v_cmp_eq_u32 vcc, %3, v" a "\n\tv_cmp_eq_u32 %1, %3, v" b "\n\ts_or_b64 %1, %1, vcc\n\t" \
    "v_cmp_eq_u32 vcc, %3, v" c "\n\ts_or_b64 %1, %1, vcc\n\t" \
    "v_cmp_eq_u32 vcc, %3, v" d "\n\ts_or_b64 %1, %1, vcc\n\ts_cbranch_scc0 2f\n\t"
    asm volatile(POLL_LD("0:3") POLL_LD("4:7") POLL_LD("8:11")
                 "s_movk_i32 %0, 64\n"
                 "1:\n\t"
                 POLL_CK("0", "1", "2", "3") POLL_LD("0:3")
                 POLL_CK("4", "5", "6", "7") POLL_LD("4:7")
                 POLL_CK("8", "9", "10", "11") POLL_LD("8:11")
                 "s_sub_u32 %0, %0, 1\n\ts_cmp_lg_u32 %0, 0\n\ts_cbranch_scc1 1b\n"
                 "2:\n\t"
                 "s_waitcnt vmcnt(0)"
                 : "=&s"(n), "=&s"(t) : "v"(ptr), "s"(SENTINEL)
                 : "memory", "scc", "vcc", "v0", "v1", "v2", "v3", "v4", "v5", "v6", "v7", "v8", "v9", "v10", "v11");
    return n;
}

__global__ __launch_bounds__(512) void gemm_a2a(Params p) {
    __shared__ __attribute__((aligned(1024))) char smem[LDS_BYTES];
    Ctx c;
    u32x4 kws;
    unsigned rank;
    GLB_AS char* outd;
    const void* ka = (const void*)__builtin_amdgcn_kernarg_segment_ptr();
    asm volatile("s_load_dwordx4 %0, %4, 0x0\n\ts_load_dwordx4 %1, %4, 0x10\n\ts_load_dwordx4 %2, %4, 0x20\n\t"
                 "s_load_dword %3, %4, 0x30" : "=&s"(c.sx), "=&s"(c.sw), "=&s"(kws), "=&s"(rank) : "s"(ka));
    SB();
    unsigned b = blockIdx.x;
    asm volatile("" : "+s"(b));
    const unsigned j = b & 7, mt = (b >> 3) & 7, nt = (b >> 6) & 3;
    const unsigned peer = (j & 1) * 4 + nt;
    const unsigned lane = threadIdx.x & 63;
    const unsigned wid = __builtin_amdgcn_readfirstlane(threadIdx.x >> 6);
    const unsigned sub = (j >> 1) ^ nt;
    LDS_AS char* lds = (LDS_AS char*)smem;
    c.lds_base = (unsigned)(uintptr_t)lds + wid * WAVE_LDS;
    c.lmask = ((mt ^ nt) & 1) - 1;
    const unsigned cs = (mt >> 1) ^ (nt & 2);
    c.kbase = ssel(c.lmask, 4 * (wid + 1) + (cs ^ ((wid + 1) >> 1)), wid ^ cs) ^ ((j & 1) * 16 + (j & 2) * 4);
    const unsigned g = lane >> 4, m = lane & 15;
    c.vx[0] = (mt * 32 + (lane >> 3)) * (KDIM * 2) + (((lane & 7) ^ g) << 4);
    c.vx[1] = c.vx[0] ^ 64;
    const unsigned nsl = ssel(c.lmask, wid * DLY + (((peer * 4 + sub) & 7) == 3 ? 0 : LDLY), wid * FDLY);
    asm volatile("" : "+v"(c.vx[0]), "+v"(c.vx[1]));
    asm volatile("" : "+s"(c.kbase), "+s"(c.lds_base));
    const unsigned kb0 = block_of<0>(c), kb1 = block_of<1>(c);
    asm volatile("s_waitcnt lgkmcnt(0)" : "+s"(c.sx), "+s"(c.sw), "+s"(kws), "+s"(rank));
    rank &= 7;
    GLB_AS char* const myws = (GLB_AS char*)(((unsigned long long)kws[1] << 32) | kws[0]);
    GLB_AS char* const myout = (GLB_AS char*)(((unsigned long long)kws[3] << 32) | kws[2]);
    unsigned warm = 0;
    if (__builtin_expect(nsl != 0, 0)) {
        if (wid == 3) {
            const unsigned long long wp = ((unsigned long long)c.sw[1] << 32) | c.sw[0];
            char* po = p.out[peer];
            asm volatile("" : "+s"(po));
            asm volatile("s_load_dword %0, %1, %2" : "+s"(warm) : "s"(wp), "s"(mt << 20) : "memory");
            asm volatile("s_load_dword %0, %1, %2" : "+s"(warm) : "s"(po), "s"(peer << 17) : "memory");
            asm volatile("s_load_dword %0, %1, %2" : "+s"(warm) : "s"(myout), "s"(rank << 17) : "memory");
            asm volatile("s_load_dword %0, %1, %2" : "+s"(warm) : "s"(myws), "i"(WS_TMO + 256) : "memory");
        }
        for (unsigned i = 0; i < nsl; i += 2) __builtin_amdgcn_s_sleep(1);
    }
    issue_x<0>(c, kb0);
    c.vw = (((lane >> 3) & 3) + (lane >> 5) * 8) * (NDIM * 2) + (peer * 4 + sub) * 128 + (((lane & 7) ^ (g << 1)) << 4);
    issue_w<0>(c, kb0);
    asm volatile("s_load_dwordx2 %0, %1, %2 offset:0x40" : "=&s"(outd) : "s"(ka), "s"(peer * 8));
    if (c.lmask) issue_x<2>(c, kb0 ^ 1);
    barriers(1 + (wid & c.lmask));
    const unsigned arm = wid * 16 + (j >> 1) * 4 + (mt >> 1);
    const bool armer = ((peer ^ rank) | c.lmask) == 0 && (arm >> 3) != rank;
    if (__builtin_expect(armer, 0)) {
        asm volatile("s_load_dword %0, %1, %2" : "+s"(warm) : "s"(p.ws[(arm >> 3) & 7]), "i"(WS_TMO + 256) : "memory");
        GLB_AS char* fp = myout + (size_t)arm * (32 * NLOC * 2) + lane * 16;
        const u32x4 sv = {SENTINEL, SENTINEL, SENTINEL, SENTINEL};
#pragma unroll
        for (int i = 0; i < 16; ++i) store16_sys(fp + i * 1024, sv);
    }
    GLB_AS unsigned* cnt = (GLB_AS unsigned*)(myws + WS_CNT + (b * NW + wid) * 128);
    unsigned seq;
    asm volatile("global_load_dword %0, %1, %2 sc1" : "=v"(seq) : "v"(0u), "s"(cnt) : "memory");
    LDS_AS char* ring = lds + wid * WAVE_LDS;
    const unsigned q = m >> 2;
    piece<1, 4>(c, kb1); piece<1, 5>(c, kb1); SB();
    c.ax[0] = ring + m * 128 + ((g ^ (m >> 1)) << 4);
    c.ax[1] = lxor(c.ax[0], 64);
    asm volatile("" : "+v"(c.ax[0]), "+v"(c.ax[1]));
    piece<1, 6>(c, kb1); SB();
    c.aw[0] = ring + (g >> 1) * 2048 + (4 * (g & 1) + q) * 128 + 8 * (lane & 3);
    asm volatile("" : "+v"(c.aw[0]));
    piece<1, 7>(c, kb1); SB();
    c.aw[0] += (2 * (g & 1) + (q >> 1)) << 5;
#pragma unroll
    for (int t = 1; t < 4; ++t) c.aw[t] = lxor(c.aw[0], t << 5);
    asm volatile("" : "+v"(c.aw[0]), "+v"(c.aw[1]), "+v"(c.aw[2]), "+v"(c.aw[3]));
    piece<1, 8>(c, kb1); piece<1, 9>(c, kb1); piece<1, 10>(c, kb1); piece<1, 11>(c, kb1);
    issue_x<1>(c, kb1);
    if (c.lmask) issue_x<3>(c, kb1 ^ 1);
    barriers((4 - wid) & c.lmask);
#pragma unroll
    for (int h = 0; h < 2; ++h)
#pragma unroll
        for (int t = 0; t < 4; ++t) c.acc[h][t] = f32x4{0.f, 0.f, 0.f, 0.f};

    if (c.lmask) wait_vm<17>();
    else wait_vm<13>();
    if (__builtin_expect(armer, 0)) {
        asm volatile("s_waitcnt vmcnt(12)" : "+v"(seq) :: "memory");
        if (lane == 0) {
            __hip_atomic_store((unsigned*)(p.ws[(arm >> 3) & 7] + WS_SYNC + (rank * 8 + (arm & 7)) * 8), seq + 1,
                               __ATOMIC_RELAXED, __HIP_MEMORY_SCOPE_SYSTEM);
            __hip_atomic_store((GLB_AS unsigned*)(myws + WS_SYNC + arm * 8 + 4), seq + 1,
                               __ATOMIC_RELAXED, __HIP_MEMORY_SCOPE_SYSTEM);
        }
    }
    asm volatile("s_waitcnt lgkmcnt(0)" : "+s"(warm), "+s"(outd) :: "memory");
    step<0>(c);
    asm volatile("s_waitcnt vmcnt(12)" : "+v"(seq) :: "memory");
    seq += 1;
    step<1>(c); wait_vm<nloads(3)>();
    step<2>(c); wait_vm<nloads(4)>();
    step<3>(c); wait_vm<nloads(5)>();
    step<4>(c); wait_vm<nloads(6)>();
    {
        const unsigned kb7 = block_of<7>(c);
        piece<7, 0>(c, kb7); read_x<5, 0>(c); read_x<5, 1>(c); SB();
        piece<7, 1>(c, kb7); read_w1<5, 0, 0>(c); SB();
        piece<7, 4>(c, kb7); read_w1<5, 0, 1>(c); SB();
        piece<7, 5>(c, kb7); read_w1<5, 0, 2>(c); SB();
        piece<7, 6>(c, kb7); read_w1<5, 0, 3>(c); SB();
        piece<7, 7>(c, kb7); read_w1<5, 1, 0>(c); SB();
        piece<7, 8>(c, kb7); read_w1<5, 1, 1>(c); SB();
        piece<7, 9>(c, kb7); read_w1<5, 1, 2>(c); SB();
        piece<7, 10>(c, kb7); read_w1<5, 1, 3>(c); SB();
        piece<7, 11>(c, kb7); SB();
    }
    LDS_AS char* pw0 = ring + m * 256 + ((g ^ (m & 7)) << 4);
    LDS_AS char* pw1 = lxor(pw0, 64);
    const unsigned row = wid * 4 + g;
    LDS_AS char* pr0 = lds + row * 256 + ((m ^ (row & 7)) << 4);
    LDS_AS char* pr2 = pr0 + 2 * WAVE_LDS;
    GLB_AS char* optr = outd + (size_t)((rank * MLOC + mt * 32 + row) * NLOC + sub * 64 + m * 4) * 2;
    GLB_AS char* optr2 = optr + 16 * NLOC * 2;
    const GLB_AS char* syncp = myws + WS_SYNC + (peer * 8 + mt) * 8;
    u32x2 sy;
    __builtin_amdgcn_s_barrier();
    asm volatile("global_load_dwordx2 %0, %1, %2 sc0 sc1" : "=v"(sy) : "v"(0u), "s"(syncp) : "memory");
    mma<5, 0, 0>(c); mma<5, 0, 1>(c);
    wait_vm<nloads(7) + 1>();
    read_x<6, 0>(c); read_x<6, 1>(c); read_w1<6, 0, 0>(c); SB(); mma<5, 0, 2>(c); pseq<8, 0>(c, 0); SB();
    read_w1<6, 0, 1>(c); read_w1<6, 0, 2>(c); SB(); mma<5, 0, 3>(c); pseq<8, 1>(c, 0); SB();
    read_w1<6, 0, 3>(c); read_w1<6, 1, 0>(c); SB(); mma<5, 1, 0>(c); pseq<8, 2>(c, 0); SB();
    read_w1<6, 1, 1>(c); read_w1<6, 1, 2>(c); SB(); mma<5, 1, 1>(c); pseq<8, 3>(c, 0); SB();
    read_w1<6, 1, 3>(c); SB(); mma<5, 1, 2>(c); mma<5, 1, 3>(c); pseq<8, 4>(c, 0); SB();
    mma<6, 0, 0>(c); mma<6, 0, 1>(c); pseq<8, 5>(c, 0); SB();
    mma<6, 0, 2>(c); mma<6, 0, 3>(c); pseq<8, 6>(c, 0); pseq<8, 7>(c, 0); SB();
    asm volatile("" : "+v"(pw0), "+v"(pw1), "+v"(pr0), "+v"(pr2), "+v"(optr), "+v"(optr2));
    wait_vm<4 + 1 + NA>();
    read_x<7, 0>(c); read_w<7, 0>(c); read_x<7, 1>(c);
    SB();
    mma<6, 1, 0>(c); mma<6, 1, 1>(c); mma<6, 1, 2>(c); mma<6, 1, 3>(c);
    wait_vm<1 + NA>();
    read_w<7, 1>(c);
    SB();
    mma<7, 0, 0>(c); mma<7, 0, 1>(c); mma<7, 0, 2>(c); mma<7, 0, 3>(c);
    mma<7, 1, 0>(c); mma<7, 1, 1>(c); mma<7, 1, 2>(c);
    *(LDS_AS f32x4*)(pw0) = c.acc[0][0];
    SB();
    mma<7, 1, 3>(c);
    *(LDS_AS f32x4*)(pw1) = c.acc[0][1];
    *(LDS_AS f32x4*)(pw0 + 128) = c.acc[0][2];
    *(LDS_AS f32x4*)(pw1 + 128) = c.acc[0][3];
    SB();
    issue_v<NA, NA + NB>(c);
    lds_barrier();
    const f32x4 a0 = *(LDS_AS f32x4*)(pr0), a1 = *(LDS_AS f32x4*)(pr0 + WAVE_LDS);
    const f32x4 a2 = *(LDS_AS f32x4*)(pr2), a3 = *(LDS_AS f32x4*)(pr2 + WAVE_LDS);
    SB();
    asm volatile("s_waitcnt vmcnt(%1)" : "+v"(sy) : "i"(NA + NB - 4) : "memory");
    read_v<0>(c); read_v<1>(c);
    SB();
    issue_v<NA + NB, 2 * RET>(c);
    bool ok = true;
    if (__builtin_expect(peer != rank && __builtin_amdgcn_readfirstlane(sy[0] < seq || sy[1] < seq), 0)) {
        const unsigned long long t0 = __builtin_amdgcn_s_memrealtime();
        for (unsigned spins = 1;; ++spins) {
            asm volatile("global_load_dwordx2 %0, %1, %2 sc0 sc1\n\ts_waitcnt vmcnt(0)" : "=&v"(sy) : "v"(0u), "s"(syncp) : "memory");
            if (__builtin_amdgcn_readfirstlane(sy[0] >= seq && sy[1] >= seq)) break;
            if ((spins & 63) == 0 && __builtin_amdgcn_readfirstlane(expired(p, seq, t0))) {
                ok = false;
                optr = optr2 = myws + WS_JUNK + threadIdx.x * 8;
                break;
            }
            __builtin_amdgcn_s_sleep(1);
        }
    }
    {
        f32x4 y = a0 + a1, z = a2 + a3, e;
        PIN2(y, z);
        mv<0, 0, 0>(c);
        y += z;
        z[0] = __builtin_amdgcn_fmed3f(y[0], -60.0f, 60.0f); z[1] = __builtin_amdgcn_fmed3f(y[1], -60.0f, 60.0f);
        PIN2(y, z);
        mv<0, 0, 1>(c);
        z[2] = __builtin_amdgcn_fmed3f(y[2], -60.0f, 60.0f); z[3] = __builtin_amdgcn_fmed3f(y[3], -60.0f, 60.0f);
        z *= -1.4426950408889634f;
        PIN1(z);
        mv<0, 0, 2>(c);
        e[0] = __builtin_amdgcn_exp2f(z[0]); e[1] = __builtin_amdgcn_exp2f(z[1]); e[2] = z[2]; e[3] = z[3];
        PIN1(e);
        mv<0, 0, 3>(c);
        e[2] = __builtin_amdgcn_exp2f(e[2]); e[3] = __builtin_amdgcn_exp2f(e[3]);
        PIN1(e);
        mv<0, 1, 0>(c);
        e += 1.0f;
        e[0] = __builtin_amdgcn_rcpf(e[0]);
        PIN1(e);
        mv<0, 1, 1>(c);
        e[1] = __builtin_amdgcn_rcpf(e[1]); e[2] = __builtin_amdgcn_rcpf(e[2]);
        PIN1(e);
        mv<0, 1, 2>(c);
        e[3] = __builtin_amdgcn_rcpf(e[3]);
        e *= y;
        store8_sys(optr, u32x2{pack2(e[0], e[1]), pack2(e[2], e[3])});
        SB();
    }
    mv<0, 1, 3>(c);
    pass2<1>(c);
    *(LDS_AS f32x4*)(pw0 + UNIT) = c.acc[1][0];
    *(LDS_AS f32x4*)(pw1 + UNIT) = c.acc[1][1];
    *(LDS_AS f32x4*)(pw0 + UNIT + 128) = c.acc[1][2];
    *(LDS_AS f32x4*)(pw1 + UNIT + 128) = c.acc[1][3];
    lds_barrier();
    {
        const f32x4 b0 = *(LDS_AS f32x4*)(pr0 + UNIT), b1 = *(LDS_AS f32x4*)(pr0 + UNIT + WAVE_LDS);
        const f32x4 b2 = *(LDS_AS f32x4*)(pr2 + UNIT), b3 = *(LDS_AS f32x4*)(pr2 + UNIT + WAVE_LDS);
        SB();
        const f32x4 y = (b0 + b1) + (b2 + b3);
        f32x4 e;
#pragma unroll
        for (int i = 0; i < 4; ++i) e[i] = __builtin_amdgcn_fmed3f(y[i], -60.0f, 60.0f);
        e *= -1.4426950408889634f;
        PIN1(e);
#pragma unroll
        for (int i = 0; i < 4; ++i) e[i] = __builtin_amdgcn_exp2f(e[i]);
        PIN1(e);
        e += 1.0f;
        PIN1(e);
#pragma unroll
        for (int i = 0; i < 4; ++i) e[i] = __builtin_amdgcn_rcpf(e[i]);
        PIN1(e);
        e *= y;
        store8_sys(optr2, u32x2{pack2(e[0], e[1]), pack2(e[2], e[3])});
    }
    if (lane == 0) __hip_atomic_store(cnt, seq, __ATOMIC_RELAXED, __HIP_MEMORY_SCOPE_AGENT);
    if (peer != rank && ok) {
        const GLB_AS char* mine = myout + (size_t)((peer * MLOC + mt * 32 + wid * 8 + (lane >> 3)) * NLOC + sub * 64 + (lane & 7) * 8) * 2;
        if (__builtin_expect(!poll_arrived(mine), 0)) {
            const unsigned long long t0 = __builtin_amdgcn_s_memrealtime();
            while (!poll_arrived(mine))
                if (__builtin_amdgcn_readfirstlane(expired(p, seq, t0))) break;
        }
    }
}

extern "C" void kernel_launch(int rank, int nranks, ncclComm_t comm,
                              hipStream_t stream,
                              void* const* d_in, const int* in_sizes, int n_in,
                              void* d_out, int out_size,
                              const int* in_dtypes, int out_dtype,
                              void* const* d_out_all,
                              void* const* d_ws_all, size_t ws_bytes) {
    if (nranks != NRANK || (unsigned)rank >= (unsigned)NRANK || n_in < 2 || in_sizes[0] != MLOC * KDIM
        || in_sizes[1] != KDIM * NDIM || out_size != NRANK * MLOC * NLOC || ws_bytes < (size_t)WS_END) {
        fprintf(stderr, "kernel_launch: unexpected problem shape\n");
        return;
    }
    Params p;
    const uintptr_t ax = (uintptr_t)d_in[0], aw = (uintptr_t)d_in[1];
    p.srdx[0] = (unsigned)ax; p.srdx[1] = (unsigned)(ax >> 32) & 0xffffu; p.srdx[2] = MLOC * KDIM * 2; p.srdx[3] = 0x00020000u;
    p.srdw[0] = (unsigned)aw; p.srdw[1] = (unsigned)(aw >> 32) & 0xffffu; p.srdw[2] = KDIM * NDIM * 2; p.srdw[3] = 0x00020000u;
    p.myws = (char*)d_ws_all[rank];
    p.myout = (char*)d_out;
    p.rank = (unsigned)rank;
    p.pad[0] = p.pad[1] = p.pad[2] = 0;
    for (int i = 0; i < NRANK; ++i) {
        p.out[i] = (char*)d_out_all[i];
        p.ws[i] = (char*)d_ws_all[i];
    }
    gemm_a2a<<<dim3(NWG), dim3(NW * 64), 0, stream>>>(p);
    HIP_CHECK(hipGetLastError());
}
```

```cpp
#include <hip/hip_runtime.h>
#include <rccl/rccl.h>
#include <cstddef>
#include <cstdint>
#include <cstdio>

#define HIP_CHECK(expr) do { hipError_t _e = (expr); if (_e != hipSuccess) { \
    fprintf(stderr, "HIP error %s at %s:%d\n", hipGetErrorString(_e), __FILE__, __LINE__); \
    return; } } while (0)

typedef __attribute__((ext_vector_type(8))) __bf16 bf16x8;
typedef __attribute__((ext_vector_type(2))) __bf16 bf16x2;
typedef __attribute__((ext_vector_type(4))) short s16x4;
typedef __attribute__((ext_vector_type(4))) float f32x4;
typedef __attribute__((ext_vector_type(4))) unsigned u32x4;
typedef __attribute__((ext_vector_type(2))) unsigned u32x2;
#define LDS_AS __attribute__((address_space(3)))
#define GLB_AS __attribute__((address_space(1)))

constexpr int NRANK = 8;
constexpr int KDIM = 2048, NDIM = 2048, MLOC = 256, NLOC = 256;
constexpr int NWG = 256;
constexpr int NW = 4;
constexpr int UNIT = 12288;
constexpr int NSLOT = 3;
constexpr int WAVE_LDS = NSLOT * UNIT + 4096;
constexpr int LDS_BYTES = NW * WAVE_LDS;
constexpr int RET = 4;
constexpr int NA = 4;
constexpr int NB = 2;
constexpr int DLY = 6, LDLY = 6, FDLY = 4;
static_assert(RET >= 3 && RET <= 5 && NA >= 4 && NA + NB <= 2 * RET, "see voff() and the counted waits");
constexpr int xoff(int u) { return u == 3 ? NSLOT * UNIT : (u % NSLOT) * UNIT + 8192; }
constexpr int voff(int v) { return v < 3 ? v * UNIT + 8192 + 2048 : NSLOT * UNIT + (v - 3) * 2048; }

constexpr int WS_SYNC = 0;
constexpr int WS_TMO = 4096;
constexpr int WS_JUNK = 8192;
constexpr int WS_CNT = 16384;
constexpr int WS_END = WS_CNT + NWG * NW * 128;
constexpr unsigned long long DEADLINE_TICKS = 50000000ull;
constexpr unsigned SENTINEL = 0xFF81FF81u;

struct Params {
    unsigned srdx[4];
    unsigned srdw[4];
    char* myws;
    char* myout;
    unsigned rank;
    unsigned pad[3];
    char* out[NRANK];
    char* ws[NRANK];
};
static_assert(offsetof(Params, srdw) == 0x10 && offsetof(Params, myws) == 0x20 && offsetof(Params, myout) == 0x28
              && offsetof(Params, rank) == 0x30 && offsetof(Params, out) == 0x40, "offsets used by the argument loads");

#define SB() __builtin_amdgcn_sched_barrier(0)
#define PIN1(a) do { asm volatile("" : "+v"(a)); SB(); } while (0)
#define PIN2(a, b) do { asm volatile("" : "+v"(a), "+v"(b)); SB(); } while (0)

template <int LOFF, int GOFF>
__device__ __forceinline__ void dma16(unsigned lds_base, unsigned voff, unsigned sbase, u32x4 srd) {
    unsigned t;
    asm volatile("s_add_u32 m0, %1, %2\n\ts_add_u32 %0, %6, %3\n\tbuffer_load_dwordx4 %4, %5, %0 offen lds"
                 : "=&s"(t) : "s"(lds_base), "i"(LOFF), "i"(GOFF), "v"(voff), "s"(srd), "s"(sbase) : "memory", "scc");
}
template <int N> __device__ __forceinline__ void wait_vm() {
    asm volatile("s_waitcnt vmcnt(%0)" :: "i"(N) : "memory");
}
__device__ __forceinline__ void lds_barrier() { asm volatile("s_waitcnt lgkmcnt(0)\n\ts_barrier" ::: "memory"); }
__device__ __forceinline__ void barriers(unsigned n) {
    for (unsigned i = 0; i < n; ++i) asm volatile("s_barrier" ::: "memory");
}
__device__ __forceinline__ void store8_sys(GLB_AS char* ptr, u32x2 v) {
    asm volatile("global_store_dwordx2 %0, %1, off sc0 sc1\n\ts_nop 1" :: "v"(ptr), "v"(v) : "memory");
}
__device__ __forceinline__ void store16_sys(GLB_AS char* ptr, u32x4 v) {
    asm volatile("global_store_dwordx4 %0, %1, off sc0 sc1\n\ts_nop 1" :: "v"(ptr), "v"(v) : "memory");
}
__device__ __forceinline__ unsigned ssel(unsigned cond, unsigned a, unsigned b) {
    unsigned r;
    asm("s_cmp_lg_u32 %1, 0\n\ts_cselect_b32 %0, %2, %3" : "=s"(r) : "s"(cond), "s"(a), "s"(b) : "scc");
    return r;
}
__device__ __forceinline__ void touch(unsigned& sink, const void* base, unsigned off) {
    asm volatile("s_load_dword %0, %1, %2" : "+s"(sink) : "s"(base), "s"(off) : "memory");
}
__device__ __forceinline__ LDS_AS char* lxor(LDS_AS char* p, unsigned v) {
    return (LDS_AS char*)(uintptr_t)((unsigned)(uintptr_t)p ^ v);
}

struct Ctx {
    unsigned lds_base;
    unsigned kbase, lmask;
    unsigned vx[2], vw;
    u32x4 sx, sw;
    LDS_AS char* ax[2];
    LDS_AS char* aw[4];
    bf16x8 wf[8][2][4];
    bf16x8 xf[8][2][2];
    bf16x8 xv[RET][2];
    f32x4 acc[2][4];
};

template <int U> __device__ __forceinline__ unsigned block_of(const Ctx& c) {
    constexpr unsigned tl = 18u * (U & 1) ^ (U >> 1), tf = 4u * U;
    return c.kbase ^ tf ^ ((tl ^ tf) & c.lmask);
}
constexpr bool full(int u) { return u < 8 - RET; }
constexpr int nloads(int u) { return (u == 2 || u == 3) ? 8 : full(u) ? 12 : 10; }
template <int U, int I, bool ALLX = false> __device__ __forceinline__ void piece(Ctx& c, unsigned kb) {
    constexpr int so = (U % NSLOT) * UNIT;
    if constexpr (U == 8) {
        dma16<voff(I / 2) + (I & 1) * 1024, (16 + (I & 1) * 8) * (KDIM * 2)>(c.lds_base, c.vx[I & 1], block_of<8 - RET + I / 2>(c) << 7, c.sx);
    } else if constexpr (I < 4) {
        if constexpr (I < 2 || full(U) || ALLX) dma16<xoff(U) + I * 1024, I * 8 * (KDIM * 2)>(c.lds_base, c.vx[I & 1], kb << 7, c.sx);
    } else dma16<so + (I - 4) * 1024, (((I - 4) >> 1) * 16 + (I & 1) * 4) * (NDIM * 2)>(c.lds_base, c.vw, kb << 18, c.sw);
}
template <int U, int K> __device__ __forceinline__ void pseq(Ctx& c, unsigned kb) {
    constexpr int order[12] = {0, 1, 4, 5, 6, 2, 7, 3, 8, 9, 10, 11};
    if constexpr (U == 2 || U == 3) {
        if constexpr (K == 0) {
            if (!c.lmask) { piece<U, 0>(c, kb); piece<U, 1>(c, kb); piece<U, 2>(c, kb); piece<U, 3>(c, kb); }
        } else if constexpr (order[K] >= 4) piece<U, order[K]>(c, kb);
    } else if constexpr (U < 8) piece<U, order[K]>(c, kb);
    else if constexpr (K < NA) piece<8, K>(c, kb);
}
template <int U> __device__ __forceinline__ void issue_x(Ctx& c, unsigned kb) {
    piece<U, 0, true>(c, kb); piece<U, 1, true>(c, kb); piece<U, 2, true>(c, kb); piece<U, 3, true>(c, kb);
}
template <int U> __device__ __forceinline__ void issue_w(Ctx& c, unsigned kb) {
    piece<U, 4>(c, kb); piece<U, 5>(c, kb); piece<U, 6>(c, kb); piece<U, 7>(c, kb);
    piece<U, 8>(c, kb); piece<U, 9>(c, kb); piece<U, 10>(c, kb); piece<U, 11>(c, kb);
}
template <int I, int E> __device__ __forceinline__ void issue_v(Ctx& c) {
    if constexpr (I < E) { piece<8, I>(c, 0); issue_v<I + 1, E>(c); }
}
template <int U, int S> __device__ __forceinline__ void read_x(Ctx& c) {
    c.xf[U][S][0] = *(LDS_AS bf16x8*)(c.ax[S] + xoff(U));
    if constexpr (full(U)) c.xf[U][S][1] = *(LDS_AS bf16x8*)(c.ax[S] + xoff(U) + 2048);
}
template <int U, int S, int T> __device__ __forceinline__ void read_w1(Ctx& c) {
    constexpr int so = (U % NSLOT) * UNIT + S * 4096;
    s16x4 lo = __builtin_amdgcn_ds_read_tr16_b64_v4i16((LDS_AS s16x4*)(c.aw[T] + so));
    s16x4 hi = __builtin_amdgcn_ds_read_tr16_b64_v4i16((LDS_AS s16x4*)(c.aw[T] + so + 1024));
    c.wf[U][S][T] = __builtin_bit_cast(bf16x8, __builtin_shufflevector(lo, hi, 0, 1, 2, 3, 4, 5, 6, 7));
}
template <int U, int S> __device__ __forceinline__ void read_w(Ctx& c) {
    read_w1<U, S, 0>(c); read_w1<U, S, 1>(c); read_w1<U, S, 2>(c); read_w1<U, S, 3>(c);
}
template <int V> __device__ __forceinline__ void read_v(Ctx& c) {
    c.xv[V][0] = *(LDS_AS bf16x8*)(c.ax[0] + voff(V));
    c.xv[V][1] = *(LDS_AS bf16x8*)(c.ax[1] + voff(V));
}
template <int U, int S, int T> __device__ __forceinline__ void mma(Ctx& c) {
    c.acc[0][T] = __builtin_amdgcn_mfma_f32_16x16x32_bf16(c.wf[U][S][T], c.xf[U][S][0], c.acc[0][T], 0, 0, 0);
    if constexpr (full(U))
        c.acc[1][T] = __builtin_amdgcn_mfma_f32_16x16x32_bf16(c.wf[U][S][T], c.xf[U][S][1], c.acc[1][T], 0, 0, 0);
    SB();
}
template <int V, int S, int T> __device__ __forceinline__ void mv(Ctx& c) {
    c.acc[1][T] = __builtin_amdgcn_mfma_f32_16x16x32_bf16(c.wf[8 - RET + V][S][T], c.xv[V][S], c.acc[1][T], 0, 0, 0);
    SB();
}
template <int V> __device__ __forceinline__ void pass2(Ctx& c) {
    if constexpr (V < RET) {
        if constexpr (V + 1 < RET) {
            wait_vm<2 * RET - 2 * (V + 2) + 1>();
            read_v<V + 1>(c);
            SB();
        }
        mv<V, 0, 0>(c); mv<V, 0, 1>(c); mv<V, 0, 2>(c); mv<V, 0, 3>(c);
        mv<V, 1, 0>(c); mv<V, 1, 1>(c); mv<V, 1, 2>(c); mv<V, 1, 3>(c);
        pass2<V + 1>(c);
    }
}
template <int U> __device__ __forceinline__ void step(Ctx& c) {
    constexpr int N = U + 2;
    const unsigned kb = block_of<N>(c);
    pseq<N, 0>(c, kb); read_x<U, 0>(c); read_w1<U, 0, 0>(c); SB();
    pseq<N, 1>(c, kb); read_w1<U, 0, 1>(c); read_w1<U, 0, 2>(c); SB();
    pseq<N, 2>(c, kb); read_w1<U, 0, 3>(c); read_x<U, 1>(c); SB();
    pseq<N, 3>(c, kb); read_w1<U, 1, 0>(c); read_w1<U, 1, 1>(c); SB();
    pseq<N, 4>(c, kb); read_w1<U, 1, 2>(c); read_w1<U, 1, 3>(c); SB();
    mma<U, 0, 0>(c); pseq<N, 5>(c, kb); SB();
    mma<U, 0, 1>(c); pseq<N, 6>(c, kb); SB();
    mma<U, 0, 2>(c); pseq<N, 7>(c, kb); SB();
    mma<U, 0, 3>(c); pseq<N, 8>(c, kb); SB();
    mma<U, 1, 0>(c); pseq<N, 9>(c, kb); SB();
    mma<U, 1, 1>(c); pseq<N, 10>(c, kb); SB();
    mma<U, 1, 2>(c); pseq<N, 11>(c, kb); SB();
    mma<U, 1, 3>(c);
}

__device__ __forceinline__ unsigned pack2(float a, float b) {
    bf16x2 v = {(__bf16)a, (__bf16)b};
    return __builtin_bit_cast(unsigned, v);
}
__device__ __forceinline__ bool expired(const Params& p, unsigned seq, unsigned long long t0) {
    unsigned* tmo = (unsigned*)(p.myws + WS_TMO);
    if (__hip_atomic_load(tmo, __ATOMIC_RELAXED, __HIP_MEMORY_SCOPE_AGENT) == seq
        || __builtin_amdgcn_s_memrealtime() - t0 > DEADLINE_TICKS) {
        __hip_atomic_store(tmo, seq, __ATOMIC_RELAXED, __HIP_MEMORY_SCOPE_AGENT);
        __hip_atomic_store((unsigned short*)(p.myout + (size_t)((p.rank & 7) * MLOC * NLOC) * 2),
                           (unsigned short)0x7FC0, __ATOMIC_RELAXED, __HIP_MEMORY_SCOPE_SYSTEM);
        return true;
    }
    return false;
}
__device__ __forceinline__ unsigned poll_arrived(const GLB_AS char* ptr) {
    unsigned n;
    unsigned long long t;
#define POLL_LD(r) "global_load_dwordx4 v[" r "], %2, off sc0 sc1\n\t"
#define POLL_CK(a, b, c, d) \
    "s_waitcnt vmcnt(2)\n\t" \
    "v_cmp_eq_u32 vcc, %3, v" a "\n\tv_cmp_eq_u32 %1, %3, v" b "\n\ts_or_b64 %1, %1, vcc\n\t" \
    "v_cmp_eq_u32 vcc, %3, v" c "\n\ts_or_b64 %1, %1, vcc\n\t" \
    "v_cmp_eq_u32 vcc, %3, v" d "\n\ts_or_b64 %1, %1, vcc\n\ts_cbranch_scc0 2f\n\t"
    asm volatile(POLL_LD("0:3") POLL_LD("4:7") POLL_LD("8:11")
                 "s_movk_i32 %0, 64\n"
                 "1:\n\t"
                 POLL_CK("0", "1", "2", "3") POLL_LD("0:3")
                 POLL_CK("4", "5", "6", "7") POLL_LD("4:7")
                 POLL_CK("8", "9", "10", "11") POLL_LD("8:11")
                 "s_sub_u32 %0, %0, 1\n\ts_cmp_lg_u32 %0, 0\n\ts_cbranch_scc1 1b\n"
                 "2:\n\t"
                 "s_waitcnt vmcnt(0)"
                 : "=&s"(n), "=&s"(t) : "v"(ptr), "s"(SENTINEL)
                 : "memory", "scc", "vcc", "v0", "v1", "v2", "v3", "v4", "v5", "v6", "v7", "v8", "v9", "v10", "v11");
    return n;
}

__global__ __launch_bounds__(512) void gemm_a2a(Params p) {
    __shared__ __attribute__((aligned(1024))) char smem[LDS_BYTES];
    Ctx c;
    u32x4 kws;
    unsigned rank;
    GLB_AS char* outd;
    const void* ka = (const void*)__builtin_amdgcn_kernarg_segment_ptr();
    asm volatile("s_load_dwordx4 %0, %4, 0x0\n\ts_load_dwordx4 %1, %4, 0x10\n\ts_load_dwordx4 %2, %4, 0x20\n\t"
                 "s_load_dword %3, %4, 0x30" : "=&s"(c.sx), "=&s"(c.sw), "=&s"(kws), "=&s"(rank) : "s"(ka));
    SB();
    unsigned b = blockIdx.x;
    asm volatile("" : "+s"(b));
    const unsigned j = b & 7, mt = (b >> 3) & 7, nt = (b >> 6) & 3;
    const unsigned peer = (j & 1) * 4 + nt;
    const unsigned lane = threadIdx.x & 63;
    const unsigned wid = __builtin_amdgcn_readfirstlane(threadIdx.x >> 6);
    const unsigned sub = (j >> 1) ^ nt;
    LDS_AS char* lds = (LDS_AS char*)smem;
    c.lds_base = (unsigned)(uintptr_t)lds + wid * WAVE_LDS;
    c.lmask = ((mt ^ nt) & 1) - 1;
    const unsigned cs = (mt >> 1) ^ (nt & 2);
    c.kbase = ssel(c.lmask, 4 * (wid + 1) + (cs ^ ((wid + 1) >> 1)), wid ^ cs) ^ ((j & 1) * 16 + (j & 2) * 4);
    const unsigned g = lane >> 4, m = lane & 15;
    c.vx[0] = (mt * 32 + (lane >> 3)) * (KDIM * 2) + (((lane & 7) ^ g) << 4);
    c.vx[1] = c.vx[0] ^ 64;
    const unsigned nsl = ssel(c.lmask, wid * DLY + (((peer * 4 + sub) & 7) == 3 ? 0 : LDLY), wid * FDLY);
    asm volatile("" : "+v"(c.vx[0]), "+v"(c.vx[1]));
    asm volatile("" : "+s"(c.kbase), "+s"(c.lds_base));
    const unsigned kb0 = block_of<0>(c), kb1 = block_of<1>(c);
    asm volatile("s_waitcnt lgkmcnt(0)" : "+s"(c.sx), "+s"(c.sw), "+s"(kws), "+s"(rank));
    rank &= 7;
    GLB_AS char* const myws = (GLB_AS char*)(((unsigned long long)kws[1] << 32) | kws[0]);
    GLB_AS char* const myout = (GLB_AS char*)(((unsigned long long)kws[3] << 32) | kws[2]);
    unsigned warm = 0;
    if (__builtin_expect(nsl != 0, 0)) {
        if (wid == 1) touch(warm, (const void*)(((unsigned long long)c.sw[1] << 32) | c.sw[0]), mt << 20);
        if (wid == 2) touch(warm, (const void*)myws, WS_TMO + 256);
        for (unsigned i = 0; i < nsl; i += 2) __builtin_amdgcn_s_sleep(1);
    }
    issue_x<0>(c, kb0);
    c.vw = (((lane >> 3) & 3) + (lane >> 5) * 8) * (NDIM * 2) + (peer * 4 + sub) * 128 + (((lane & 7) ^ (g << 1)) << 4);
    issue_w<0>(c, kb0);
    asm volatile("s_load_dwordx2 %0, %1, %2 offset:0x40" : "=&s"(outd) : "s"(ka), "s"(peer * 8));
    if (c.lmask) {
        issue_x<2>(c, kb0 ^ 1);
        if (wid == 3) {
            touch(warm, p.out[peer], (peer << 17) + (rank << 9));
            touch(warm, (const void*)myout, rank << 17);
        }
    }
    barriers(1 + (wid & c.lmask));
    const unsigned arm = wid * 16 + (j >> 1) * 4 + (mt >> 1);
    const bool armer = ((peer ^ rank) | c.lmask) == 0 && (arm >> 3) != rank;
    if (__builtin_expect(armer, 0)) {
        touch(warm, p.ws[(arm >> 3) & 7], WS_TMO + 256);
        GLB_AS char* fp = myout + (size_t)arm * (32 * NLOC * 2) + lane * 16;
        const u32x4 sv = {SENTINEL, SENTINEL, SENTINEL, SENTINEL};
#pragma unroll
        for (int i = 0; i < 16; ++i) store16_sys(fp + i * 1024, sv);
    }
    GLB_AS unsigned* cnt = (GLB_AS unsigned*)(myws + WS_CNT + (b * NW + wid) * 128);
    unsigned seq;
    asm volatile("global_load_dword %0, %1, %2 sc1" : "=v"(seq) : "v"(0u), "s"(cnt) : "memory");
    LDS_AS char* ring = lds + wid * WAVE_LDS;
    const unsigned q = m >> 2;
    piece<1, 4>(c, kb1); piece<1, 5>(c, kb1); SB();
    c.ax[0] = ring + m * 128 + ((g ^ (m >> 1)) << 4);
    c.ax[1] = lxor(c.ax[0], 64);
    asm volatile("" : "+v"(c.ax[0]), "+v"(c.ax[1]));
    piece<1, 6>(c, kb1); SB();
    c.aw[0] = ring + (g >> 1) * 2048 + (4 * (g & 1) + q) * 128 + 8 * (lane & 3);
    asm volatile("" : "+v"(c.aw[0]));
    piece<1, 7>(c, kb1); SB();
    c.aw[0] += (2 * (g & 1) + (q >> 1)) << 5;
#pragma unroll
    for (int t = 1; t < 4; ++t) c.aw[t] = lxor(c.aw[0], t << 5);
    asm volatile("" : "+v"(c.aw[0]), "+v"(c.aw[1]), "+v"(c.aw[2]), "+v"(c.aw[3]));
    piece<1, 8>(c, kb1); piece<1, 9>(c, kb1); piece<1, 10>(c, kb1); piece<1, 11>(c, kb1);
    issue_x<1>(c, kb1);
    if (c.lmask) issue_x<3>(c, kb1 ^ 1);
    barriers((4 - wid) & c.lmask);
#pragma unroll
    for (int h = 0; h < 2; ++h)
#pragma unroll
        for (int t = 0; t < 4; ++t) c.acc[h][t] = f32x4{0.f, 0.f, 0.f, 0.f};

    if (c.lmask) wait_vm<17>();
    else wait_vm<13>();
    if (__builtin_expect(armer, 0)) {
        asm volatile("s_waitcnt vmcnt(12)" : "+v"(seq) :: "memory");
        if (lane == 0) {
            __hip_atomic_store((unsigned*)(p.ws[(arm >> 3) & 7] + WS_SYNC + (rank * 8 + (arm & 7)) * 8), seq + 1,
                               __ATOMIC_RELAXED, __HIP_MEMORY_SCOPE_SYSTEM);
            __hip_atomic_store((GLB_AS unsigned*)(myws + WS_SYNC + arm * 8 + 4), seq + 1,
                               __ATOMIC_RELAXED, __HIP_MEMORY_SCOPE_SYSTEM);
        }
    }
    asm volatile("s_waitcnt lgkmcnt(0)" : "+s"(warm), "+s"(outd) :: "memory");
    step<0>(c);
    asm volatile("s_waitcnt vmcnt(12)" : "+v"(seq) :: "memory");
    seq += 1;
    step<1>(c); wait_vm<nloads(3)>();
    step<2>(c); wait_vm<nloads(4)>();
    step<3>(c); wait_vm<nloads(5)>();
    step<4>(c); wait_vm<nloads(6)>();
    {
        const unsigned kb7 = block_of<7>(c);
        piece<7, 0>(c, kb7); read_x<5, 0>(c); read_x<5, 1>(c); SB();
        piece<7, 1>(c, kb7); read_w1<5, 0, 0>(c); SB();
        piece<7, 4>(c, kb7); read_w1<5, 0, 1>(c); SB();
        piece<7, 5>(c, kb7); read_w1<5, 0, 2>(c); SB();
        piece<7, 6>(c, kb7); read_w1<5, 0, 3>(c); SB();
        piece<7, 7>(c, kb7); read_w1<5, 1, 0>(c); SB();
        piece<7, 8>(c, kb7); read_w1<5, 1, 1>(c); SB();
        piece<7, 9>(c, kb7); read_w1<5, 1, 2>(c); SB();
        piece<7, 10>(c, kb7); read_w1<5, 1, 3>(c); SB();
        piece<7, 11>(c, kb7); SB();
    }
    LDS_AS char* pw0 = ring + m * 256 + ((g ^ (m & 7)) << 4);
    LDS_AS char* pw1 = lxor(pw0, 64);
    const unsigned row = wid * 4 + g;
    LDS_AS char* pr0 = lds + row * 256 + ((m ^ (row & 7)) << 4);
    LDS_AS char* pr2 = pr0 + 2 * WAVE_LDS;
    GLB_AS char* optr = outd + (size_t)((rank * MLOC + mt * 32 + row) * NLOC + sub * 64 + m * 4) * 2;
    GLB_AS char* optr2 = optr + 16 * NLOC * 2;
    const GLB_AS char* syncp = myws + WS_SYNC + (peer * 8 + mt) * 8;
    u32x2 sy;
    __builtin_amdgcn_s_barrier();
    asm volatile("global_load_dwordx2 %0, %1, %2 sc0 sc1" : "=v"(sy) : "v"(0u), "s"(syncp) : "memory");
    mma<5, 0, 0>(c); mma<5, 0, 1>(c);
    wait_vm<nloads(7) + 1>();
    read_x<6, 0>(c); read_x<6, 1>(c); read_w1<6, 0, 0>(c); SB(); mma<5, 0, 2>(c); pseq<8, 0>(c, 0); SB();
    read_w1<6, 0, 1>(c); read_w1<6, 0, 2>(c); SB(); mma<5, 0, 3>(c); pseq<8, 1>(c, 0); SB();
    read_w1<6, 0, 3>(c); read_w1<6, 1, 0>(c); SB(); mma<5, 1, 0>(c); pseq<8, 2>(c, 0); SB();
    read_w1<6, 1, 1>(c); read_w1<6, 1, 2>(c); SB(); mma<5, 1, 1>(c); pseq<8, 3>(c, 0); SB();
    read_w1<6, 1, 3>(c); SB(); mma<5, 1, 2>(c); mma<5, 1, 3>(c); pseq<8, 4>(c, 0); SB();
    mma<6, 0, 0>(c); mma<6, 0, 1>(c); pseq<8, 5>(c, 0); SB();
    mma<6, 0, 2>(c); mma<6, 0, 3>(c); pseq<8, 6>(c, 0); pseq<8, 7>(c, 0); SB();
    asm volatile("" : "+v"(pw0), "+v"(pw1), "+v"(pr0), "+v"(pr2), "+v"(optr), "+v"(optr2));
    wait_vm<4 + 1 + NA>();
    read_x<7, 0>(c); read_w<7, 0>(c); read_x<7, 1>(c);
    SB();
    mma<6, 1, 0>(c); mma<6, 1, 1>(c); mma<6, 1, 2>(c); mma<6, 1, 3>(c);
    wait_vm<1 + NA>();
    read_w<7, 1>(c);
    SB();
    mma<7, 0, 0>(c); mma<7, 0, 1>(c); mma<7, 0, 2>(c); mma<7, 0, 3>(c);
    mma<7, 1, 0>(c); mma<7, 1, 1>(c); mma<7, 1, 2>(c);
    *(LDS_AS f32x4*)(pw0) = c.acc[0][0];
    SB();
    mma<7, 1, 3>(c);
    *(LDS_AS f32x4*)(pw1) = c.acc[0][1];
    *(LDS_AS f32x4*)(pw0 + 128) = c.acc[0][2];
    *(LDS_AS f32x4*)(pw1 + 128) = c.acc[0][3];
    SB();
    issue_v<NA, NA + NB>(c);
    lds_barrier();
    const f32x4 a0 = *(LDS_AS f32x4*)(pr0), a1 = *(LDS_AS f32x4*)(pr0 + WAVE_LDS);
    const f32x4 a2 = *(LDS_AS f32x4*)(pr2), a3 = *(LDS_AS f32x4*)(pr2 + WAVE_LDS);
    SB();
    asm volatile("s_waitcnt vmcnt(%1)" : "+v"(sy) : "i"(NA + NB - 4) : "memory");
    read_v<0>(c); read_v<1>(c);
    SB();
    issue_v<NA + NB, 2 * RET>(c);
    bool ok = true;
    if (__builtin_expect(peer != rank && __builtin_amdgcn_readfirstlane(sy[0] < seq || sy[1] < seq), 0)) {
        const unsigned long long t0 = __builtin_amdgcn_s_memrealtime();
        for (unsigned spins = 1;; ++spins) {
            asm volatile("global_load_dwordx2 %0, %1, %2 sc0 sc1\n\ts_waitcnt vmcnt(0)" : "=&v"(sy) : "v"(0u), "s"(syncp) : "memory");
            if (__builtin_amdgcn_readfirstlane(sy[0] >= seq && sy[1] >= seq)) break;
            if ((spins & 63) == 0 && __builtin_amdgcn_readfirstlane(expired(p, seq, t0))) {
                ok = false;
                optr = optr2 = myws + WS_JUNK + threadIdx.x * 8;
                break;
            }
            __builtin_amdgcn_s_sleep(1);
        }
    }
    {
        f32x4 y = a0 + a1, z = a2 + a3, e;
        PIN2(y, z);
        mv<0, 0, 0>(c);
        y += z;
        z[0] = __builtin_amdgcn_fmed3f(y[0], -60.0f, 60.0f); z[1] = __builtin_amdgcn_fmed3f(y[1], -60.0f, 60.0f);
        PIN2(y, z);
        mv<0, 0, 1>(c);
        z[2] = __builtin_amdgcn_fmed3f(y[2], -60.0f, 60.0f); z[3] = __builtin_amdgcn_fmed3f(y[3], -60.0f, 60.0f);
        z *= -1.4426950408889634f;
        PIN1(z);
        mv<0, 0, 2>(c);
        e[0] = __builtin_amdgcn_exp2f(z[0]); e[1] = __builtin_amdgcn_exp2f(z[1]); e[2] = z[2]; e[3] = z[3];
        PIN1(e);
        mv<0, 0, 3>(c);
        e[2] = __builtin_amdgcn_exp2f(e[2]); e[3] = __builtin_amdgcn_exp2f(e[3]);
        PIN1(e);
        mv<0, 1, 0>(c);
        e += 1.0f;
        e[0] = __builtin_amdgcn_rcpf(e[0]);
        PIN1(e);
        mv<0, 1, 1>(c);
        e[1] = __builtin_amdgcn_rcpf(e[1]); e[2] = __builtin_amdgcn_rcpf(e[2]);
        PIN1(e);
        mv<0, 1, 2>(c);
        e[3] = __builtin_amdgcn_rcpf(e[3]);
        e *= y;
        store8_sys(optr, u32x2{pack2(e[0], e[1]), pack2(e[2], e[3])});
        SB();
    }
    mv<0, 1, 3>(c);
    pass2<1>(c);
    *(LDS_AS f32x4*)(pw0 + UNIT) = c.acc[1][0];
    *(LDS_AS f32x4*)(pw1 + UNIT) = c.acc[1][1];
    *(LDS_AS f32x4*)(pw0 + UNIT + 128) = c.acc[1][2];
    *(LDS_AS f32x4*)(pw1 + UNIT + 128) = c.acc[1][3];
    lds_barrier();
    {
        const f32x4 b0 = *(LDS_AS f32x4*)(pr0 + UNIT), b1 = *(LDS_AS f32x4*)(pr0 + UNIT + WAVE_LDS);
        const f32x4 b2 = *(LDS_AS f32x4*)(pr2 + UNIT), b3 = *(LDS_AS f32x4*)(pr2 + UNIT + WAVE_LDS);
        SB();
        const f32x4 y = (b0 + b1) + (b2 + b3);
        f32x4 e;
#pragma unroll
        for (int i = 0; i < 4; ++i) e[i] = __builtin_amdgcn_fmed3f(y[i], -60.0f, 60.0f);
        e *= -1.4426950408889634f;
        PIN1(e);
#pragma unroll
        for (int i = 0; i < 4; ++i) e[i] = __builtin_amdgcn_exp2f(e[i]);
        PIN1(e);
        e += 1.0f;
        PIN1(e);
#pragma unroll
        for (int i = 0; i < 4; ++i) e[i] = __builtin_amdgcn_rcpf(e[i]);
        PIN1(e);
        e *= y;
        store8_sys(optr2, u32x2{pack2(e[0], e[1]), pack2(e[2], e[3])});
    }
    if (lane == 0) __hip_atomic_store(cnt, seq, __ATOMIC_RELAXED, __HIP_MEMORY_SCOPE_AGENT);
    if (peer != rank && ok) {
        const GLB_AS char* mine = myout + (size_t)((peer * MLOC + mt * 32 + wid * 8 + (lane >> 3)) * NLOC + sub * 64 + (lane & 7) * 8) * 2;
        if (__builtin_expect(!poll_arrived(mine), 0)) {
            const unsigned long long t0 = __builtin_amdgcn_s_memrealtime();
            while (!poll_arrived(mine))
                if (__builtin_amdgcn_readfirstlane(expired(p, seq, t0))) break;
        }
    }
}

extern "C" void kernel_launch(int rank, int nranks, ncclComm_t comm,
                              hipStream_t stream,
                              void* const* d_in, const int* in_sizes, int n_in,
                              void* d_out, int out_size,
                              const int* in_dtypes, int out_dtype,
                              void* const* d_out_all,
                              void* const* d_ws_all, size_t ws_bytes) {
    if (nranks != NRANK || (unsigned)rank >= (unsigned)NRANK || n_in < 2 || in_sizes[0] != MLOC * KDIM
        || in_sizes[1] != KDIM * NDIM || out_size != NRANK * MLOC * NLOC || ws_bytes < (size_t)WS_END) {
        fprintf(stderr, "kernel_launch: unexpected problem shape\n");
        return;
    }
    Params p;
    const uintptr_t ax = (uintptr_t)d_in[0], aw = (uintptr_t)d_in[1];
    p.srdx[0] = (unsigned)ax; p.srdx[1] = (unsigned)(ax >> 32) & 0xffffu; p.srdx[2] = MLOC * KDIM * 2; p.srdx[3] = 0x00020000u;
    p.srdw[0] = (unsigned)aw; p.srdw[1] = (unsigned)(aw >> 32) & 0xffffu; p.srdw[2] = KDIM * NDIM * 2; p.srdw[3] = 0x00020000u;
    p.myws = (char*)d_ws_all[rank];
    p.myout = (char*)d_out;
    p.rank = (unsigned)rank;
    p.pad[0] = p.pad[1] = p.pad[2] = 0;
    for (int i = 0; i < NRANK; ++i) {
        p.out[i] = (char*)d_out_all[i];
        p.ws[i] = (char*)d_ws_all[i];
    }
    gemm_a2a<<<dim3(NWG), dim3(NW * 64), 0, stream>>>(p);
    HIP_CHECK(hipGetLastError());
}
```

```cpp
#include <hip/hip_runtime.h>
#include <rccl/rccl.h>
#include <cstddef>
#include <cstdint>
#include <cstdio>

#define HIP_CHECK(expr) do { hipError_t _e = (expr); if (_e != hipSuccess) { \
    fprintf(stderr, "HIP error %s at %s:%d\n", hipGetErrorString(_e), __FILE__, __LINE__); \
    return; } } while (0)

typedef __attribute__((ext_vector_type(8))) __bf16 bf16x8;
typedef __attribute__((ext_vector_type(2))) __bf16 bf16x2;
typedef __attribute__((ext_vector_type(4))) short s16x4;
typedef __attribute__((ext_vector_type(4))) float f32x4;
typedef __attribute__((ext_vector_type(4))) unsigned u32x4;
typedef __attribute__((ext_vector_type(2))) unsigned u32x2;
#define LDS_AS __attribute__((address_space(3)))
#define GLB_AS __attribute__((address_space(1)))

constexpr int NRANK = 8;
constexpr int KDIM = 2048, NDIM = 2048, MLOC = 256, NLOC = 256;
constexpr int NWG = 256;
constexpr int NW = 4;
constexpr int UNIT = 12288;
constexpr int NSLOT = 3;
constexpr int WAVE_LDS = NSLOT * UNIT + 4096;
constexpr int LDS_BYTES = NW * WAVE_LDS;
constexpr int RET = 4;
constexpr int NA = 4;
constexpr int NB = 2;
constexpr int DLY = 6, LDLY = 6, FDLY = 4;
static_assert(RET >= 3 && RET <= 5 && NA >= 4 && NA + NB <= 2 * RET, "see voff() and the counted waits");
constexpr int xoff(int u) { return u == 3 ? NSLOT * UNIT : (u % NSLOT) * UNIT + 8192; }
constexpr int voff(int v) { return v < 3 ? v * UNIT + 8192 + 2048 : NSLOT * UNIT + (v - 3) * 2048; }

constexpr int WS_SYNC = 0;
constexpr int WS_TMO = 4096;
constexpr int WS_JUNK = 8192;
constexpr int WS_CNT = 16384;
constexpr int WS_END = WS_CNT + NWG * NW * 128;
constexpr unsigned long long DEADLINE_TICKS = 50000000ull;
constexpr unsigned SENTINEL = 0xFF81FF81u;

struct Params {
    unsigned srdx[4];
    unsigned srdw[4];
    char* myws;
    char* myout;
    unsigned rank;
    unsigned pad[3];
    char* out[NRANK];
    char* ws[NRANK];
};
static_assert(offsetof(Params, srdw) == 0x10 && offsetof(Params, myws) == 0x20 && offsetof(Params, myout) == 0x28
              && offsetof(Params, rank) == 0x30 && offsetof(Params, out) == 0x40, "offsets used by the argument loads");

#define SB() __builtin_amdgcn_sched_barrier(0)
#define PIN1(a) do { asm volatile("" : "+v"(a)); SB(); } while (0)
#define PIN2(a, b) do { asm volatile("" : "+v"(a), "+v"(b)); SB(); } while (0)

template <int LOFF, int GOFF>
__device__ __forceinline__ void dma16(unsigned lds_base, unsigned voff, unsigned sbase, u32x4 srd) {
    unsigned t;
    asm volatile("s_add_u32 m0, %1, %2\n\ts_add_u32 %0, %6, %3\n\tbuffer_load_dwordx4 %4, %5, %0 offen lds"
                 : "=&s"(t) : "s"(lds_base), "i"(LOFF), "i"(GOFF), "v"(voff), "s"(srd), "s"(sbase) : "memory", "scc");
}
template <int N> __device__ __forceinline__ void wait_vm() {
    asm volatile("s_waitcnt vmcnt(%0)" :: "i"(N) : "memory");
}
__device__ __forceinline__ void lds_barrier() { asm volatile("s_waitcnt lgkmcnt(0)\n\ts_barrier" ::: "memory"); }
__device__ __forceinline__ void barriers(unsigned n) {
    for (unsigned i = 0; i < n; ++i) asm volatile("s_barrier" ::: "memory");
}
__device__ __forceinline__ void store8_sys(GLB_AS char* ptr, u32x2 v) {
    asm volatile("global_store_dwordx2 %0, %1, off sc0 sc1\n\ts_nop 1" :: "v"(ptr), "v"(v) : "memory");
}
__device__ __forceinline__ void store16_sys(GLB_AS char* ptr, u32x4 v) {
    asm volatile("global_store_dwordx4 %0, %1, off sc0 sc1\n\ts_nop 1" :: "v"(ptr), "v"(v) : "memory");
}
__device__ __forceinline__ unsigned ssel(unsigned cond, unsigned a, unsigned b) {
    unsigned r;
    asm("s_cmp_lg_u32 %1, 0\n\ts_cselect_b32 %0, %2, %3" : "=s"(r) : "s"(cond), "s"(a), "s"(b) : "scc");
    return r;
}
__device__ __forceinline__ void touch(const void* base, unsigned off) {
    asm volatile("s_load_dword s90, %0, %1" :: "s"(base), "s"(off) : "memory", "s90");
}
__device__ __forceinline__ LDS_AS char* lxor(LDS_AS char* p, unsigned v) {
    return (LDS_AS char*)(uintptr_t)((unsigned)(uintptr_t)p ^ v);
}

struct Ctx {
    unsigned lds_base;
    unsigned kbase, lmask;
    unsigned vx[2], vw;
    u32x4 sx, sw;
    LDS_AS char* ax[2];
    LDS_AS char* aw[4];
    bf16x8 wf[8][2][4];
    bf16x8 xf[8][2][2];
    bf16x8 xv[RET][2];
    f32x4 acc[2][4];
};

template <int U> __device__ __forceinline__ unsigned block_of(const Ctx& c) {
    constexpr unsigned tl = 18u * (U & 1) ^ (U >> 1), tf = 4u * U;
    return c.kbase ^ tf ^ ((tl ^ tf) & c.lmask);
}
constexpr bool full(int u) { return u < 8 - RET; }
constexpr int nloads(int u) { return (u == 2 || u == 3) ? 8 : full(u) ? 12 : 10; }
template <int U, int I, bool ALLX = false> __device__ __forceinline__ void piece(Ctx& c, unsigned kb) {
    constexpr int so = (U % NSLOT) * UNIT;
    if constexpr (U == 8) {
        dma16<voff(I / 2) + (I & 1) * 1024, (16 + (I & 1) * 8) * (KDIM * 2)>(c.lds_base, c.vx[I & 1], block_of<8 - RET + I / 2>(c) << 7, c.sx);
    } else if constexpr (I < 4) {
        if constexpr (I < 2 || full(U) || ALLX) dma16<xoff(U) + I * 1024, I * 8 * (KDIM * 2)>(c.lds_base, c.vx[I & 1], kb << 7, c.sx);
    } else dma16<so + (I - 4) * 1024, (((I - 4) >> 1) * 16 + (I & 1) * 4) * (NDIM * 2)>(c.lds_base, c.vw, kb << 18, c.sw);
}
template <int U, int K> __device__ __forceinline__ void pseq(Ctx& c, unsigned kb) {
    constexpr int order[12] = {0, 1, 4, 5, 6, 2, 7, 3, 8, 9, 10, 11};
    if constexpr (U == 2 || U == 3) {
        if constexpr (K == 0) {
            if (!c.lmask) { piece<U, 0>(c, kb); piece<U, 1>(c, kb); piece<U, 2>(c, kb); piece<U, 3>(c, kb); }
        } else if constexpr (order[K] >= 4) piece<U, order[K]>(c, kb);
    } else if constexpr (U < 8) piece<U, order[K]>(c, kb);
    else if constexpr (K < NA) piece<8, K>(c, kb);
}
template <int U> __device__ __forceinline__ void issue_x(Ctx& c, unsigned kb) {
    piece<U, 0, true>(c, kb); piece<U, 1, true>(c, kb); piece<U, 2, true>(c, kb); piece<U, 3, true>(c, kb);
}
template <int U> __device__ __forceinline__ void issue_w(Ctx& c, unsigned kb) {
    piece<U, 4>(c, kb); piece<U, 5>(c, kb); piece<U, 6>(c, kb); piece<U, 7>(c, kb);
    piece<U, 8>(c, kb); piece<U, 9>(c, kb); piece<U, 10>(c, kb); piece<U, 11>(c, kb);
}
template <int I, int E> __device__ __forceinline__ void issue_v(Ctx& c) {
    if constexpr (I < E) { piece<8, I>(c, 0); issue_v<I + 1, E>(c); }
}
template <int U, int S> __device__ __forceinline__ void read_x(Ctx& c) {
    c.xf[U][S][0] = *(LDS_AS bf16x8*)(c.ax[S] + xoff(U));
    if constexpr (full(U)) c.xf[U][S][1] = *(LDS_AS bf16x8*)(c.ax[S] + xoff(U) + 2048);
}
template <int U, int S, int T> __device__ __forceinline__ void read_w1(Ctx& c) {
    constexpr int so = (U % NSLOT) * UNIT + S * 4096;
    s16x4 lo = __builtin_amdgcn_ds_read_tr16_b64_v4i16((LDS_AS s16x4*)(c.aw[T] + so));
    s16x4 hi = __builtin_amdgcn_ds_read_tr16_b64_v4i16((LDS_AS s16x4*)(c.aw[T] + so + 1024));
    c.wf[U][S][T] = __builtin_bit_cast(bf16x8, __builtin_shufflevector(lo, hi, 0, 1, 2, 3, 4, 5, 6, 7));
}
template <int U, int S> __device__ __forceinline__ void read_w(Ctx& c) {
    read_w1<U, S, 0>(c); read_w1<U, S, 1>(c); read_w1<U, S, 2>(c); read_w1<U, S, 3>(c);
}
template <int V> __device__ __forceinline__ void read_v(Ctx& c) {
    c.xv[V][0] = *(LDS_AS bf16x8*)(c.ax[0] + voff(V));
    c.xv[V][1] = *(LDS_AS bf16x8*)(c.ax[1] + voff(V));
}
template <int U, int S, int T> __device__ __forceinline__ void mma(Ctx& c) {
    c.acc[0][T] = __builtin_amdgcn_mfma_f32_16x16x32_bf16(c.wf[U][S][T], c.xf[U][S][0], c.acc[0][T], 0, 0, 0);
    if constexpr (full(U))
        c.acc[1][T] = __builtin_amdgcn_mfma_f32_16x16x32_bf16(c.wf[U][S][T], c.xf[U][S][1], c.acc[1][T], 0, 0, 0);
    SB();
}
template <int V, int S, int T> __device__ __forceinline__ void mv(Ctx& c) {
    c.acc[1][T] = __builtin_amdgcn_mfma_f32_16x16x32_bf16(c.wf[8 - RET + V][S][T], c.xv[V][S], c.acc[1][T], 0, 0, 0);
    SB();
}
template <int V> __device__ __forceinline__ void pass2(Ctx& c) {
    if constexpr (V < RET) {
        if constexpr (V + 1 < RET) {
            wait_vm<2 * RET - 2 * (V + 2) + 1>();
            read_v<V + 1>(c);
            SB();
        }
        mv<V, 0, 0>(c); mv<V, 0, 1>(c); mv<V, 0, 2>(c); mv<V, 0, 3>(c);
        mv<V, 1, 0>(c); mv<V, 1, 1>(c); mv<V, 1, 2>(c); mv<V, 1, 3>(c);
        pass2<V + 1>(c);
    }
}
template <int U> __device__ __forceinline__ void step(Ctx& c) {
    constexpr int N = U + 2;
    const unsigned kb = block_of<N>(c);
    pseq<N, 0>(c, kb); read_x<U, 0>(c); read_w1<U, 0, 0>(c); SB();
    pseq<N, 1>(c, kb); read_w1<U, 0, 1>(c); read_w1<U, 0, 2>(c); SB();
    pseq<N, 2>(c, kb); read_w1<U, 0, 3>(c); read_x<U, 1>(c); SB();
    pseq<N, 3>(c, kb); read_w1<U, 1, 0>(c); read_w1<U, 1, 1>(c); SB();
    pseq<N, 4>(c, kb); read_w1<U, 1, 2>(c); read_w1<U, 1, 3>(c); SB();
    mma<U, 0, 0>(c); pseq<N, 5>(c, kb); SB();
    mma<U, 0, 1>(c); pseq<N, 6>(c, kb); SB();
    mma<U, 0, 2>(c); pseq<N, 7>(c, kb); SB();
    mma<U, 0, 3>(c); pseq<N, 8>(c, kb); SB();
    mma<U, 1, 0>(c); pseq<N, 9>(c, kb); SB();
    mma<U, 1, 1>(c); pseq<N, 10>(c, kb); SB();
    mma<U, 1, 2>(c); pseq<N, 11>(c, kb); SB();
    mma<U, 1, 3>(c);
}

__device__ __forceinline__ unsigned pack2(float a, float b) {
    bf16x2 v = {(__bf16)a, (__bf16)b};
    return __builtin_bit_cast(unsigned, v);
}
__device__ __forceinline__ bool expired(const Params& p, unsigned seq, unsigned long long t0) {
    unsigned* tmo = (unsigned*)(p.myws + WS_TMO);
    if (__hip_atomic_load(tmo, __ATOMIC_RELAXED, __HIP_MEMORY_SCOPE_AGENT) == seq
        || __builtin_amdgcn_s_memrealtime() - t0 > DEADLINE_TICKS) {
        __hip_atomic_store(tmo, seq, __ATOMIC_RELAXED, __HIP_MEMORY_SCOPE_AGENT);
        __hip_atomic_store((unsigned short*)(p.myout + (size_t)((p.rank & 7) * MLOC * NLOC) * 2),
                           (unsigned short)0x7FC0, __ATOMIC_RELAXED, __HIP_MEMORY_SCOPE_SYSTEM);
        return true;
    }
    return false;
}
__device__ __forceinline__ unsigned poll_arrived(const GLB_AS char* ptr) {
    unsigned n;
    unsigned long long t;
#define POLL_LD(r) "global_load_dwordx4 v[" r "], %2, off sc0 sc1\n\t"
#define POLL_CK(a, b, c, d) \
    "s_waitcnt vmcnt(2)\n\t" \
    "v_cmp_eq_u32 vcc, %3, v" a "\n\tv_cmp_eq_u32 %1, %3, v" b "\n\ts_or_b64 %1, %1, vcc\n\t" \
    "v_cmp_eq_u32 vcc, %3, v" c "\n\ts_or_b64 %1, %1, vcc\n\t" \
    "v_cmp_eq_u32 vcc, %3, v" d "\n\ts_or_b64 %1, %1, vcc\n\ts_cbranch_scc0 2f\n\t"
    asm volatile(POLL_LD("0:3") POLL_LD("4:7") POLL_LD("8:11")
                 "s_movk_i32 %0, 64\n"
                 "1:\n\t"
                 POLL_CK("0", "1", "2", "3") POLL_LD("0:3")
                 POLL_CK("4", "5", "6", "7") POLL_LD("4:7")
                 POLL_CK("8", "9", "10", "11") POLL_LD("8:11")
                 "s_sub_u32 %0, %0, 1\n\ts_cmp_lg_u32 %0, 0\n\ts_cbranch_scc1 1b\n"
                 "2:\n\t"
                 "s_waitcnt vmcnt(0)"
                 : "=&s"(n), "=&s"(t) : "v"(ptr), "s"(SENTINEL)
                 : "memory", "scc", "vcc", "v0", "v1", "v2", "v3", "v4", "v5", "v6", "v7", "v8", "v9", "v10", "v11");
    return n;
}

__global__ __launch_bounds__(512) void gemm_a2a(Params p) {
    __shared__ __attribute__((aligned(1024))) char smem[LDS_BYTES];
    Ctx c;
    u32x4 kws;
    unsigned rank;
    GLB_AS char* outd;
    const void* ka = (const void*)__builtin_amdgcn_kernarg_segment_ptr();
    asm volatile("s_load_dwordx4 %0, %4, 0x0\n\ts_load_dwordx4 %1, %4, 0x10\n\ts_load_dwordx4 %2, %4, 0x20\n\t"
                 "s_load_dword %3, %4, 0x30" : "=&s"(c.sx), "=&s"(c.sw), "=&s"(kws), "=&s"(rank) : "s"(ka));
    SB();
    unsigned b = blockIdx.x;
    asm volatile("" : "+s"(b));
    const unsigned j = b & 7, mt = (b >> 3) & 7, nt = (b >> 6) & 3;
    const unsigned peer = (j & 1) * 4 + nt;
    const unsigned lane = threadIdx.x & 63;
    const unsigned wid = __builtin_amdgcn_readfirstlane(threadIdx.x >> 6);
    const unsigned sub = (j >> 1) ^ nt;
    LDS_AS char* lds = (LDS_AS char*)smem;
    c.lds_base = (unsigned)(uintptr_t)lds + wid * WAVE_LDS;
    c.lmask = ((mt ^ nt) & 1) - 1;
    const unsigned cs = (mt >> 1) ^ (nt & 2);
    c.kbase = ssel(c.lmask, 4 * (wid + 1) + (cs ^ ((wid + 1) >> 1)), wid ^ cs) ^ ((j & 1) * 16 + (j & 2) * 4);
    const unsigned g = lane >> 4, m = lane & 15;
    c.vx[0] = (mt * 32 + (lane >> 3)) * (KDIM * 2) + (((lane & 7) ^ g) << 4);
    c.vx[1] = c.vx[0] ^ 64;
    const unsigned nsl = ssel(c.lmask, wid * DLY + (((peer * 4 + sub) & 7) == 3 ? 0 : LDLY), wid * FDLY);
    asm volatile("" : "+v"(c.vx[0]), "+v"(c.vx[1]));
    asm volatile("" : "+s"(c.kbase), "+s"(c.lds_base));
    const unsigned kb0 = block_of<0>(c);
    asm volatile("s_waitcnt lgkmcnt(0)" : "+s"(c.sx), "+s"(c.sw), "+s"(kws), "+s"(rank));
    GLB_AS char* const myws = (GLB_AS char*)(((unsigned long long)kws[1] << 32) | kws[0]);
    GLB_AS char* const myout = (GLB_AS char*)(((unsigned long long)kws[3] << 32) | kws[2]);
    if (__builtin_expect(nsl != 0, 0)) {
        if (wid == 1) touch((const void*)(((unsigned long long)c.sw[1] << 32) | c.sw[0]), mt << 20);
        if (wid == 2) touch((const void*)myws, WS_TMO + 256);
        for (unsigned i = 0; i < nsl; i += 2) __builtin_amdgcn_s_sleep(1);
    }
    issue_x<0>(c, kb0);
    c.vw = (((lane >> 3) & 3) + (lane >> 5) * 8) * (NDIM * 2) + (peer * 4 + sub) * 128 + (((lane & 7) ^ (g << 1)) << 4);
    issue_w<0>(c, kb0);
    asm volatile("s_load_dwordx2 %0, %1, %2 offset:0x40" : "=&s"(outd) : "s"(ka), "s"(peer * 8));
    if (c.lmask) issue_x<2>(c, kb0 ^ 1);
    rank &= 7;
    GLB_AS unsigned* cnt = (GLB_AS unsigned*)(myws + WS_CNT + (b * NW + wid) * 128);
    unsigned seq;
    asm volatile("global_load_dword %0, %1, %2 sc1" : "=v"(seq) : "v"(0u), "s"(cnt) : "memory");
    const unsigned arm = wid * 16 + (j >> 1) * 4 + (mt >> 1);
    unsigned armer = ssel((peer ^ rank) | c.lmask, 0, (arm >> 3) ^ rank);
    unsigned kb1 = block_of<1>(c), side = (wid & c.lmask) | armer;
    asm volatile("" : "+s"(armer), "+s"(kb1), "+s"(side));
    barriers(1);
    if (side) {
        if (__builtin_expect(armer != 0, 0)) {
            touch(p.ws[(arm >> 3) & 7], WS_TMO + 256);
            GLB_AS char* fp = myout + (size_t)arm * (32 * NLOC * 2) + lane * 16;
            const u32x4 sv = {SENTINEL, SENTINEL, SENTINEL, SENTINEL};
#pragma unroll
            for (int i = 0; i < 16; ++i) store16_sys(fp + i * 1024, sv);
        } else {
            if (wid == 3) {
                touch(p.out[peer], (peer << 17) + (rank << 9));
                touch((const void*)myout, rank << 17);
            }
            barriers(wid);
        }
    }
    LDS_AS char* ring = lds + wid * WAVE_LDS;
    const unsigned q = m >> 2;
    piece<1, 4>(c, kb1); piece<1, 5>(c, kb1); SB();
    c.ax[0] = ring + m * 128 + ((g ^ (m >> 1)) << 4);
    c.ax[1] = lxor(c.ax[0], 64);
    asm volatile("" : "+v"(c.ax[0]), "+v"(c.ax[1]));
    piece<1, 6>(c, kb1); SB();
    c.aw[0] = ring + (g >> 1) * 2048 + (4 * (g & 1) + q) * 128 + 8 * (lane & 3);
    asm volatile("" : "+v"(c.aw[0]));
    piece<1, 7>(c, kb1); SB();
    c.aw[0] += (2 * (g & 1) + (q >> 1)) << 5;
#pragma unroll
    for (int t = 1; t < 4; ++t) c.aw[t] = lxor(c.aw[0], t << 5);
    asm volatile("" : "+v"(c.aw[0]), "+v"(c.aw[1]), "+v"(c.aw[2]), "+v"(c.aw[3]));
    piece<1, 8>(c, kb1); piece<1, 9>(c, kb1); piece<1, 10>(c, kb1); piece<1, 11>(c, kb1);
    issue_x<1>(c, kb1);
    if (c.lmask) {
        issue_x<3>(c, kb1 ^ 1);
        barriers(4 - wid);
    }
#pragma unroll
    for (int h = 0; h < 2; ++h)
#pragma unroll
        for (int t = 0; t < 4; ++t) c.acc[h][t] = f32x4{0.f, 0.f, 0.f, 0.f};

    if (c.lmask) wait_vm<16>();
    else wait_vm<12>();
    if (__builtin_expect(armer != 0, 0)) {
        asm volatile("s_waitcnt vmcnt(12)" : "+v"(seq) :: "memory");
        if (lane == 0) {
            __hip_atomic_store((unsigned*)(p.ws[(arm >> 3) & 7] + WS_SYNC + (rank * 8 + (arm & 7)) * 8), seq + 1,
                               __ATOMIC_RELAXED, __HIP_MEMORY_SCOPE_SYSTEM);
            __hip_atomic_store((GLB_AS unsigned*)(myws + WS_SYNC + arm * 8 + 4), seq + 1,
                               __ATOMIC_RELAXED, __HIP_MEMORY_SCOPE_SYSTEM);
        }
    }
    asm volatile("s_waitcnt lgkmcnt(0)" : "+s"(outd) :: "memory");
    step<0>(c);
    asm volatile("s_waitcnt vmcnt(12)" : "+v"(seq) :: "memory");
    seq += 1;
    step<1>(c); wait_vm<nloads(3)>();
    step<2>(c); wait_vm<nloads(4)>();
    step<3>(c); wait_vm<nloads(5)>();
    step<4>(c); wait_vm<nloads(6)>();
    {
        const unsigned kb7 = block_of<7>(c);
        piece<7, 0>(c, kb7); read_x<5, 0>(c); read_x<5, 1>(c); SB();
        piece<7, 1>(c, kb7); read_w1<5, 0, 0>(c); SB();
        piece<7, 4>(c, kb7); read_w1<5, 0, 1>(c); SB();
        piece<7, 5>(c, kb7); read_w1<5, 0, 2>(c); SB();
        piece<7, 6>(c, kb7); read_w1<5, 0, 3>(c); SB();
        piece<7, 7>(c, kb7); read_w1<5, 1, 0>(c); SB();
        piece<7, 8>(c, kb7); read_w1<5, 1, 1>(c); SB();
        piece<7, 9>(c, kb7); read_w1<5, 1, 2>(c); SB();
        piece<7, 10>(c, kb7); read_w1<5, 1, 3>(c); SB();
        piece<7, 11>(c, kb7); SB();
    }
    LDS_AS char* pw0 = ring + m * 256 + ((g ^ (m & 7)) << 4);
    LDS_AS char* pw1 = lxor(pw0, 64);
    const unsigned row = wid * 4 + g;
    LDS_AS char* pr0 = lds + row * 256 + ((m ^ (row & 7)) << 4);
    LDS_AS char* pr2 = pr0 + 2 * WAVE_LDS;
    GLB_AS char* optr = outd + (size_t)((rank * MLOC + mt * 32 + row) * NLOC + sub * 64 + m * 4) * 2;
    GLB_AS char* optr2 = optr + 16 * NLOC * 2;
    const GLB_AS char* syncp = myws + WS_SYNC + (peer * 8 + mt) * 8;
    u32x2 sy;
    __builtin_amdgcn_s_barrier();
    asm volatile("global_load_dwordx2 %0, %1, %2 sc0 sc1" : "=v"(sy) : "v"(0u), "s"(syncp) : "memory");
    mma<5, 0, 0>(c); mma<5, 0, 1>(c);
    wait_vm<nloads(7) + 1>();
    read_x<6, 0>(c); read_x<6, 1>(c); read_w1<6, 0, 0>(c); SB(); mma<5, 0, 2>(c); pseq<8, 0>(c, 0); SB();
    read_w1<6, 0, 1>(c); read_w1<6, 0, 2>(c); SB(); mma<5, 0, 3>(c); pseq<8, 1>(c, 0); SB();
    read_w1<6, 0, 3>(c); read_w1<6, 1, 0>(c); SB(); mma<5, 1, 0>(c); pseq<8, 2>(c, 0); SB();
    read_w1<6, 1, 1>(c); read_w1<6, 1, 2>(c); SB(); mma<5, 1, 1>(c); pseq<8, 3>(c, 0); SB();
    read_w1<6, 1, 3>(c); SB(); mma<5, 1, 2>(c); mma<5, 1, 3>(c); pseq<8, 4>(c, 0); SB();
    mma<6, 0, 0>(c); mma<6, 0, 1>(c); pseq<8, 5>(c, 0); SB();
    mma<6, 0, 2>(c); mma<6, 0, 3>(c); pseq<8, 6>(c, 0); pseq<8, 7>(c, 0); SB();
    asm volatile("" : "+v"(pw0), "+v"(pw1), "+v"(pr0), "+v"(pr2), "+v"(optr), "+v"(optr2));
    wait_vm<4 + 1 + NA>();
    read_x<7, 0>(c); read_w<7, 0>(c); read_x<7, 1>(c);
    SB();
    mma<6, 1, 0>(c); mma<6, 1, 1>(c); mma<6, 1, 2>(c); mma<6, 1, 3>(c);
    wait_vm<1 + NA>();
    read_w<7, 1>(c);
    SB();
    mma<7, 0, 0>(c); mma<7, 0, 1>(c); mma<7, 0, 2>(c); mma<7, 0, 3>(c);
    mma<7, 1, 0>(c); mma<7, 1, 1>(c); mma<7, 1, 2>(c);
    *(LDS_AS f32x4*)(pw0) = c.acc[0][0];
    SB();
    mma<7, 1, 3>(c);
    *(LDS_AS f32x4*)(pw1) = c.acc[0][1];
    *(LDS_AS f32x4*)(pw0 + 128) = c.acc[0][2];
    *(LDS_AS f32x4*)(pw1 + 128) = c.acc[0][3];
    SB();
    issue_v<NA, NA + NB>(c);
    lds_barrier();
    const f32x4 a0 = *(LDS_AS f32x4*)(pr0), a1 = *(LDS_AS f32x4*)(pr0 + WAVE_LDS);
    const f32x4 a2 = *(LDS_AS f32x4*)(pr2), a3 = *(LDS_AS f32x4*)(pr2 + WAVE_LDS);
    SB();
    asm volatile("s_waitcnt vmcnt(%1)" : "+v"(sy) : "i"(NA + NB - 4) : "memory");
    read_v<0>(c); read_v<1>(c);
    SB();
    issue_v<NA + NB, 2 * RET>(c);
    bool ok = true;
    if (__builtin_expect(peer != rank && __builtin_amdgcn_readfirstlane(sy[0] < seq || sy[1] < seq), 0)) {
        const unsigned long long t0 = __builtin_amdgcn_s_memrealtime();
        for (unsigned spins = 1;; ++spins) {
            asm volatile("global_load_dwordx2 %0, %1, %2 sc0 sc1\n\ts_waitcnt vmcnt(0)" : "=&v"(sy) : "v"(0u), "s"(syncp) : "memory");
            if (__builtin_amdgcn_readfirstlane(sy[0] >= seq && sy[1] >= seq)) break;
            if ((spins & 63) == 0 && __builtin_amdgcn_readfirstlane(expired(p, seq, t0))) {
                ok = false;
                optr = optr2 = myws + WS_JUNK + threadIdx.x * 8;
                break;
            }
            __builtin_amdgcn_s_sleep(1);
        }
    }
    {
        f32x4 y = a0 + a1, z = a2 + a3, e;
        PIN2(y, z);
        mv<0, 0, 0>(c);
        y += z;
        z[0] = __builtin_amdgcn_fmed3f(y[0], -60.0f, 60.0f); z[1] = __builtin_amdgcn_fmed3f(y[1], -60.0f, 60.0f);
        PIN2(y, z);
        mv<0, 0, 1>(c);
        z[2] = __builtin_amdgcn_fmed3f(y[2], -60.0f, 60.0f); z[3] = __builtin_amdgcn_fmed3f(y[3], -60.0f, 60.0f);
        z *= -1.4426950408889634f;
        PIN1(z);
        mv<0, 0, 2>(c);
        e[0] = __builtin_amdgcn_exp2f(z[0]); e[1] = __builtin_amdgcn_exp2f(z[1]); e[2] = z[2]; e[3] = z[3];
        PIN1(e);
        mv<0, 0, 3>(c);
        e[2] = __builtin_amdgcn_exp2f(e[2]); e[3] = __builtin_amdgcn_exp2f(e[3]);
        PIN1(e);
        mv<0, 1, 0>(c);
        e += 1.0f;
        e[0] = __builtin_amdgcn_rcpf(e[0]);
        PIN1(e);
        mv<0, 1, 1>(c);
        e[1] = __builtin_amdgcn_rcpf(e[1]); e[2] = __builtin_amdgcn_rcpf(e[2]);
        PIN1(e);
        mv<0, 1, 2>(c);
        e[3] = __builtin_amdgcn_rcpf(e[3]);
        e *= y;
        store8_sys(optr, u32x2{pack2(e[0], e[1]), pack2(e[2], e[3])});
        SB();
    }
    mv<0, 1, 3>(c);
    pass2<1>(c);
    *(LDS_AS f32x4*)(pw0 + UNIT) = c.acc[1][0];
    *(LDS_AS f32x4*)(pw1 + UNIT) = c.acc[1][1];
    *(LDS_AS f32x4*)(pw0 + UNIT + 128) = c.acc[1][2];
    *(LDS_AS f32x4*)(pw1 + UNIT + 128) = c.acc[1][3];
    lds_barrier();
    {
        const f32x4 b0 = *(LDS_AS f32x4*)(pr0 + UNIT), b1 = *(LDS_AS f32x4*)(pr0 + UNIT + WAVE_LDS);
        const f32x4 b2 = *(LDS_AS f32x4*)(pr2 + UNIT), b3 = *(LDS_AS f32x4*)(pr2 + UNIT + WAVE_LDS);
        SB();
        const f32x4 y = (b0 + b1) + (b2 + b3);
        f32x4 e;
#pragma unroll
        for (int i = 0; i < 4; ++i) e[i] = __builtin_amdgcn_fmed3f(y[i], -60.0f, 60.0f);
        e *= -1.4426950408889634f;
        PIN1(e);
#pragma unroll
        for (int i = 0; i < 4; ++i) e[i] = __builtin_amdgcn_exp2f(e[i]);
        PIN1(e);
        e += 1.0f;
        PIN1(e);
#pragma unroll
        for (int i = 0; i < 4; ++i) e[i] = __builtin_amdgcn_rcpf(e[i]);
        PIN1(e);
        e *= y;
        store8_sys(optr2, u32x2{pack2(e[0], e[1]), pack2(e[2], e[3])});
    }
    if (lane == 0) __hip_atomic_store(cnt, seq, __ATOMIC_RELAXED, __HIP_MEMORY_SCOPE_AGENT);
    if (peer != rank && ok) {
        const GLB_AS char* mine = myout + (size_t)((peer * MLOC + mt * 32 + wid * 8 + (lane >> 3)) * NLOC + sub * 64 + (lane & 7) * 8) * 2;
        if (__builtin_expect(!poll_arrived(mine), 0)) {
            const unsigned long long t0 = __builtin_amdgcn_s_memrealtime();
            while (!poll_arrived(mine))
                if (__builtin_amdgcn_readfirstlane(expired(p, seq, t0))) break;
        }
    }
}

extern "C" void kernel_launch(int rank, int nranks, ncclComm_t comm,
                              hipStream_t stream,
                              void* const* d_in, const int* in_sizes, int n_in,
                              void* d_out, int out_size,
                              const int* in_dtypes, int out_dtype,
                              void* const* d_out_all,
                              void* const* d_ws_all, size_t ws_bytes) {
    if (nranks != NRANK || (unsigned)rank >= (unsigned)NRANK || n_in < 2 || in_sizes[0] != MLOC * KDIM
        || in_sizes[1] != KDIM * NDIM || out_size != NRANK * MLOC * NLOC || ws_bytes < (size_t)WS_END) {
        fprintf(stderr, "kernel_launch: unexpected problem shape\n");
        return;
    }
    Params p;
    const uintptr_t ax = (uintptr_t)d_in[0], aw = (uintptr_t)d_in[1];
    p.srdx[0] = (unsigned)ax; p.srdx[1] = (unsigned)(ax >> 32) & 0xffffu; p.srdx[2] = MLOC * KDIM * 2; p.srdx[3] = 0x00020000u;
    p.srdw[0] = (unsigned)aw; p.srdw[1] = (unsigned)(aw >> 32) & 0xffffu; p.srdw[2] = KDIM * NDIM * 2; p.srdw[3] = 0x00020000u;
    p.myws = (char*)d_ws_all[rank];
    p.myout = (char*)d_out;
    p.rank = (unsigned)rank;
    p.pad[0] = p.pad[1] = p.pad[2] = 0;
    for (int i = 0; i < NRANK; ++i) {
        p.out[i] = (char*)d_out_all[i];
        p.ws[i] = (char*)d_ws_all[i];
    }
    gemm_a2a<<<dim3(NWG), dim3(NW * 64), 0, stream>>>(p);
    HIP_CHECK(hipGetLastError());
}
```

```cpp
#include <hip/hip_runtime.h>
#include <rccl/rccl.h>
#include <cstddef>
#include <cstdint>
#include <cstdio>

#define HIP_CHECK(expr) do { hipError_t _e = (expr); if (_e != hipSuccess) { \
    fprintf(stderr, "HIP error %s at %s:%d\n", hipGetErrorString(_e), __FILE__, __LINE__); \
    return; } } while (0)

typedef __attribute__((ext_vector_type(8))) __bf16 bf16x8;
typedef __attribute__((ext_vector_type(2))) __bf16 bf16x2;
typedef __attribute__((ext_vector_type(4))) short s16x4;
typedef __attribute__((ext_vector_type(4))) float f32x4;
typedef __attribute__((ext_vector_type(4))) unsigned u32x4;
typedef __attribute__((ext_vector_type(2))) unsigned u32x2;
#define LDS_AS __attribute__((address_space(3)))
#define GLB_AS __attribute__((address_space(1)))

constexpr int NRANK = 8;
constexpr int KDIM = 2048, NDIM = 2048, MLOC = 256, NLOC = 256;
constexpr int NWG = 256;
constexpr int NW = 4;
constexpr int UNIT = 12288;
constexpr int NSLOT = 3;
constexpr int WAVE_LDS = NSLOT * UNIT + 4096;
constexpr int LDS_BYTES = NW * WAVE_LDS;
constexpr int RET = 4;
constexpr int NA = 4;
constexpr int NB = 2;
constexpr int DLY = 8, LDLY = 8, FDLY = 6;
static_assert(RET >= 3 && RET <= 5 && NA >= 4 && NA + NB <= 2 * RET, "see voff() and the counted waits");
constexpr int xoff(int u) { return u == 3 ? NSLOT * UNIT : (u % NSLOT) * UNIT + 8192; }
constexpr int voff(int v) { return v < 3 ? v * UNIT + 8192 + 2048 : NSLOT * UNIT + (v - 3) * 2048; }

constexpr int WS_SYNC = 0;
constexpr int WS_TMO = 4096;
constexpr int WS_JUNK = 8192;
constexpr int WS_CNT = 16384;
constexpr int WS_END = WS_CNT + NWG * NW * 128;
constexpr unsigned long long DEADLINE_TICKS = 50000000ull;
constexpr unsigned SENTINEL = 0xFF81FF81u;

struct Params {
    unsigned srdx[4];
    unsigned srdw[4];
    char* myws;
    char* myout;
    unsigned rank;
    unsigned pad[3];
    char* out[NRANK];
    char* ws[NRANK];
};
static_assert(offsetof(Params, srdw) == 0x10 && offsetof(Params, myws) == 0x20 && offsetof(Params, myout) == 0x28
              && offsetof(Params, rank) == 0x30 && offsetof(Params, out) == 0x40, "offsets used by the argument loads");

#define SB() __builtin_amdgcn_sched_barrier(0)
#define PIN1(a) do { asm volatile("" : "+v"(a)); SB(); } while (0)
#define PIN2(a, b) do { asm volatile("" : "+v"(a), "+v"(b)); SB(); } while (0)

template <int LOFF, int GOFF>
__device__ __forceinline__ void dma16(unsigned lds_base, unsigned voff, unsigned sbase, u32x4 srd) {
    unsigned t;
    asm volatile("s_add_u32 m0, %1, %2\n\ts_add_u32 %0, %6, %3\n\tbuffer_load_dwordx4 %4, %5, %0 offen lds"
                 : "=&s"(t) : "s"(lds_base), "i"(LOFF), "i"(GOFF), "v"(voff), "s"(srd), "s"(sbase) : "memory", "scc");
}
template <int N> __device__ __forceinline__ void wait_vm() {
    asm volatile("s_waitcnt vmcnt(%0)" :: "i"(N) : "memory");
}
__device__ __forceinline__ void lds_barrier() { asm volatile("s_waitcnt lgkmcnt(0)\n\ts_barrier" ::: "memory"); }
__device__ __forceinline__ void barriers(unsigned n) {
    for (unsigned i = 0; i < n; ++i) asm volatile("s_barrier" ::: "memory");
}
__device__ __forceinline__ void store8_sys(GLB_AS char* ptr, u32x2 v) {
    asm volatile("global_store_dwordx2 %0, %1, off sc0 sc1\n\ts_nop 1" :: "v"(ptr), "v"(v) : "memory");
}
__device__ __forceinline__ void store16_sys(GLB_AS char* ptr, u32x4 v) {
    asm volatile("global_store_dwordx4 %0, %1, off sc0 sc1\n\ts_nop 1" :: "v"(ptr), "v"(v) : "memory");
}
__device__ __forceinline__ unsigned ssel(unsigned cond, unsigned a, unsigned b) {
    unsigned r;
    asm("s_cmp_lg_u32 %1, 0\n\ts_cselect_b32 %0, %2, %3" : "=s"(r) : "s"(cond), "s"(a), "s"(b) : "scc");
    return r;
}
__device__ __forceinline__ void touch(const void* base, unsigned off) {
    asm volatile("s_load_dword s90, %0, %1" :: "s"(base), "s"(off) : "memory", "s90");
}
__device__ __forceinline__ LDS_AS char* lxor(LDS_AS char* p, unsigned v) {
    return (LDS_AS char*)(uintptr_t)((unsigned)(uintptr_t)p ^ v);
}

struct Ctx {
    unsigned lds_base;
    unsigned kbase, lmask;
    unsigned vx[2], vw;
    u32x4 sx, sw;
    LDS_AS char* ax[2];
    LDS_AS char* aw[4];
    bf16x8 wf[8][2][4];
    bf16x8 xf[8][2][2];
    bf16x8 xv[RET][2];
    f32x4 acc[2][4];
};

template <int U> __device__ __forceinline__ unsigned block_of(const Ctx& c) {
    constexpr unsigned tl = 18u * (U & 1) ^ (U >> 1), tf = 4u * U;
    return c.kbase ^ tf ^ ((tl ^ tf) & c.lmask);
}
constexpr bool full(int u) { return u < 8 - RET; }
constexpr int nloads(int u) { return (u == 2 || u == 3) ? 8 : full(u) ? 12 : 10; }
template <int U, int I, bool ALLX = false> __device__ __forceinline__ void piece(Ctx& c, unsigned kb) {
    constexpr int so = (U % NSLOT) * UNIT;
    if constexpr (U == 8) {
        dma16<voff(I / 2) + (I & 1) * 1024, (16 + (I & 1) * 8) * (KDIM * 2)>(c.lds_base, c.vx[I & 1], block_of<8 - RET + I / 2>(c) << 7, c.sx);
    } else if constexpr (I < 4) {
        if constexpr (I < 2 || full(U) || ALLX) dma16<xoff(U) + I * 1024, I * 8 * (KDIM * 2)>(c.lds_base, c.vx[I & 1], kb << 7, c.sx);
    } else dma16<so + (I - 4) * 1024, (((I - 4) >> 1) * 16 + (I & 1) * 4) * (NDIM * 2)>(c.lds_base, c.vw, kb << 18, c.sw);
}
template <int U, int K> __device__ __forceinline__ void pseq(Ctx& c, unsigned kb) {
    constexpr int order[12] = {0, 1, 4, 5, 6, 2, 7, 3, 8, 9, 10, 11};
    if constexpr (U == 2 || U == 3) {
        if constexpr (K == 0) {
            if (!c.lmask) { piece<U, 0>(c, kb); piece<U, 1>(c, kb); piece<U, 2>(c, kb); piece<U, 3>(c, kb); }
        } else if constexpr (order[K] >= 4) piece<U, order[K]>(c, kb);
    } else if constexpr (U < 8) piece<U, order[K]>(c, kb);
    else if constexpr (K < NA) piece<8, K>(c, kb);
}
template <int U> __device__ __forceinline__ void issue_x(Ctx& c, unsigned kb) {
    piece<U, 0, true>(c, kb); piece<U, 1, true>(c, kb); piece<U, 2, true>(c, kb); piece<U, 3, true>(c, kb);
}
template <int U> __device__ __forceinline__ void issue_w(Ctx& c, unsigned kb) {
    piece<U, 4>(c, kb); piece<U, 5>(c, kb); piece<U, 6>(c, kb); piece<U, 7>(c, kb);
    piece<U, 8>(c, kb); piece<U, 9>(c, kb); piece<U, 10>(c, kb); piece<U, 11>(c, kb);
}
template <int I, int E> __device__ __forceinline__ void issue_v(Ctx& c) {
    if constexpr (I < E) { piece<8, I>(c, 0); issue_v<I + 1, E>(c); }
}
template <int U, int S> __device__ __forceinline__ void read_x(Ctx& c) {
    c.xf[U][S][0] = *(LDS_AS bf16x8*)(c.ax[S] + xoff(U));
    if constexpr (full(U)) c.xf[U][S][1] = *(LDS_AS bf16x8*)(c.ax[S] + xoff(U) + 2048);
}
template <int U, int S, int T> __device__ __forceinline__ void read_w1(Ctx& c) {
    constexpr int so = (U % NSLOT) * UNIT + S * 4096;
    s16x4 lo = __builtin_amdgcn_ds_read_tr16_b64_v4i16((LDS_AS s16x4*)(c.aw[T] + so));
    s16x4 hi = __builtin_amdgcn_ds_read_tr16_b64_v4i16((LDS_AS s16x4*)(c.aw[T] + so + 1024));
    c.wf[U][S][T] = __builtin_bit_cast(bf16x8, __builtin_shufflevector(lo, hi, 0, 1, 2, 3, 4, 5, 6, 7));
}
template <int U, int S> __device__ __forceinline__ void read_w(Ctx& c) {
    read_w1<U, S, 0>(c); read_w1<U, S, 1>(c); read_w1<U, S, 2>(c); read_w1<U, S, 3>(c);
}
template <int V> __device__ __forceinline__ void read_v(Ctx& c) {
    c.xv[V][0] = *(LDS_AS bf16x8*)(c.ax[0] + voff(V));
    c.xv[V][1] = *(LDS_AS bf16x8*)(c.ax[1] + voff(V));
}
template <int U, int S, int T> __device__ __forceinline__ void mma(Ctx& c) {
    c.acc[0][T] = __builtin_amdgcn_mfma_f32_16x16x32_bf16(c.wf[U][S][T], c.xf[U][S][0], c.acc[0][T], 0, 0, 0);
    if constexpr (full(U))
        c.acc[1][T] = __builtin_amdgcn_mfma_f32_16x16x32_bf16(c.wf[U][S][T], c.xf[U][S][1], c.acc[1][T], 0, 0, 0);
    SB();
}
template <int V, int S, int T> __device__ __forceinline__ void mv(Ctx& c) {
    c.acc[1][T] = __builtin_amdgcn_mfma_f32_16x16x32_bf16(c.wf[8 - RET + V][S][T], c.xv[V][S], c.acc[1][T], 0, 0, 0);
    SB();
}
template <int V> __device__ __forceinline__ void pass2(Ctx& c) {
    if constexpr (V < RET) {
        if constexpr (V + 1 < RET) {
            wait_vm<2 * RET - 2 * (V + 2) + 1>();
            read_v<V + 1>(c);
            SB();
        }
        mv<V, 0, 0>(c); mv<V, 0, 1>(c); mv<V, 0, 2>(c); mv<V, 0, 3>(c);
        mv<V, 1, 0>(c); mv<V, 1, 1>(c); mv<V, 1, 2>(c); mv<V, 1, 3>(c);
        pass2<V + 1>(c);
    }
}
template <int U> __device__ __forceinline__ void step(Ctx& c) {
    constexpr int N = U + 2;
    const unsigned kb = block_of<N>(c);
    pseq<N, 0>(c, kb); read_x<U, 0>(c); read_w1<U, 0, 0>(c); SB();
    pseq<N, 1>(c, kb); read_w1<U, 0, 1>(c); read_w1<U, 0, 2>(c); SB();
    pseq<N, 2>(c, kb); read_w1<U, 0, 3>(c); read_x<U, 1>(c); SB();
    pseq<N, 3>(c, kb); read_w1<U, 1, 0>(c); read_w1<U, 1, 1>(c); SB();
    pseq<N, 4>(c, kb); read_w1<U, 1, 2>(c); read_w1<U, 1, 3>(c); SB();
    mma<U, 0, 0>(c); pseq<N, 5>(c, kb); SB();
    mma<U, 0, 1>(c); pseq<N, 6>(c, kb); SB();
    mma<U, 0, 2>(c); pseq<N, 7>(c, kb); SB();
    mma<U, 0, 3>(c); pseq<N, 8>(c, kb); SB();
    mma<U, 1, 0>(c); pseq<N, 9>(c, kb); SB();
    mma<U, 1, 1>(c); pseq<N, 10>(c, kb); SB();
    mma<U, 1, 2>(c); pseq<N, 11>(c, kb); SB();
    mma<U, 1, 3>(c);
}

__device__ __forceinline__ unsigned pack2(float a, float b) {
    bf16x2 v = {(__bf16)a, (__bf16)b};
    return __builtin_bit_cast(unsigned, v);
}
__device__ __forceinline__ bool expired(const Params& p, unsigned seq, unsigned long long t0) {
    unsigned* tmo = (unsigned*)(p.myws + WS_TMO);
    if (__hip_atomic_load(tmo, __ATOMIC_RELAXED, __HIP_MEMORY_SCOPE_AGENT) == seq
        || __builtin_amdgcn_s_memrealtime() - t0 > DEADLINE_TICKS) {
        __hip_atomic_store(tmo, seq, __ATOMIC_RELAXED, __HIP_MEMORY_SCOPE_AGENT);
        __hip_atomic_store((unsigned short*)(p.myout + (size_t)((p.rank & 7) * MLOC * NLOC) * 2),
                           (unsigned short)0x7FC0, __ATOMIC_RELAXED, __HIP_MEMORY_SCOPE_SYSTEM);
        return true;
    }
    return false;
}
__device__ __forceinline__ unsigned poll_arrived(const GLB_AS char* ptr) {
    unsigned n;
    unsigned long long t;
#define POLL_LD(r) "global_load_dwordx4 v[" r "], %2, off sc0 sc1\n\t"
#define POLL_CK(a, b, c, d) \
    "s_waitcnt vmcnt(2)\n\t" \
    "v_cmp_eq_u32 vcc, %3, v" a "\n\tv_cmp_eq_u32 %1, %3, v" b "\n\ts_or_b64 %1, %1, vcc\n\t" \
    "v_cmp_eq_u32 vcc, %3, v" c "\n\ts_or_b64 %1, %1, vcc\n\t" \
    "v_cmp_eq_u32 vcc, %3, v" d "\n\ts_or_b64 %1, %1, vcc\n\ts_cbranch_scc0 2f\n\t"
    asm volatile(POLL_LD("0:3") POLL_LD("4:7") POLL_LD("8:11")
                 "s_movk_i32 %0, 64\n"
                 "1:\n\t"
                 POLL_CK("0", "1", "2", "3") POLL_LD("0:3")
                 POLL_CK("4", "5", "6", "7") POLL_LD("4:7")
                 POLL_CK("8", "9", "10", "11") POLL_LD("8:11")
                 "s_sub_u32 %0, %0, 1\n\ts_cmp_lg_u32 %0, 0\n\ts_cbranch_scc1 1b\n"
                 "2:\n\t"
                 "s_waitcnt vmcnt(0)"
                 : "=&s"(n), "=&s"(t) : "v"(ptr), "s"(SENTINEL)
                 : "memory", "scc", "vcc", "v0", "v1", "v2", "v3", "v4", "v5", "v6", "v7", "v8", "v9", "v10", "v11");
    return n;
}

__global__ __launch_bounds__(512) void gemm_a2a(Params p) {
    __shared__ __attribute__((aligned(1024))) char smem[LDS_BYTES];
    Ctx c;
    u32x4 kws;
    unsigned rank;
    GLB_AS char* outd;
    const void* ka = (const void*)__builtin_amdgcn_kernarg_segment_ptr();
    asm volatile("s_load_dwordx4 %0, %4, 0x0\n\ts_load_dwordx4 %1, %4, 0x10\n\ts_load_dwordx4 %2, %4, 0x20\n\t"
                 "s_load_dword %3, %4, 0x30" : "=&s"(c.sx), "=&s"(c.sw), "=&s"(kws), "=&s"(rank) : "s"(ka));
    SB();
    unsigned b = blockIdx.x;
    asm volatile("" : "+s"(b));
    const unsigned j = b & 7, mt = (b >> 3) & 7, nt = (b >> 6) & 3;
    const unsigned peer = (j & 1) * 4 + nt;
    const unsigned lane = threadIdx.x & 63;
    const unsigned wid = __builtin_amdgcn_readfirstlane(threadIdx.x >> 6);
    const unsigned sub = (j >> 1) ^ nt;
    LDS_AS char* lds = (LDS_AS char*)smem;
    c.lds_base = (unsigned)(uintptr_t)lds + wid * WAVE_LDS;
    c.lmask = ((mt ^ nt) & 1) - 1;
    const unsigned cs = (mt >> 1) ^ (nt & 2);
    c.kbase = ssel(c.lmask, 4 * (wid + 1) + (cs ^ ((wid + 1) >> 1)), wid ^ cs) ^ ((j & 1) * 16 + (j & 2) * 4);
    const unsigned g = lane >> 4, m = lane & 15;
    c.vx[0] = (mt * 32 + (lane >> 3)) * (KDIM * 2) + (((lane & 7) ^ g) << 4);
    c.vx[1] = c.vx[0] ^ 64;
    const unsigned nsl = ssel(c.lmask, wid * DLY + (((peer * 4 + sub) & 7) == 3 ? 0 : LDLY), wid * FDLY);
    asm volatile("" : "+v"(c.vx[0]), "+v"(c.vx[1]));
    asm volatile("" : "+s"(c.kbase), "+s"(c.lds_base));
    const unsigned kb0 = block_of<0>(c);
    asm volatile("s_waitcnt lgkmcnt(0)" : "+s"(c.sx), "+s"(c.sw), "+s"(kws), "+s"(rank));
    GLB_AS char* const myws = (GLB_AS char*)(((unsigned long long)kws[1] << 32) | kws[0]);
    GLB_AS char* const myout = (GLB_AS char*)(((unsigned long long)kws[3] << 32) | kws[2]);
    if (__builtin_expect(nsl != 0, 1)) {
        touch((const void*)(((unsigned long long)ssel(wid & 1, c.sw[1], kws[1]) << 32) | ssel(wid & 1, c.sw[0], kws[0])),
              ssel(wid & 1, mt << 20, WS_TMO + 256));
        for (unsigned i = 0; i < nsl; i += 2) __builtin_amdgcn_s_sleep(1);
    }
    issue_x<0>(c, kb0);
    c.vw = (((lane >> 3) & 3) + (lane >> 5) * 8) * (NDIM * 2) + (peer * 4 + sub) * 128 + (((lane & 7) ^ (g << 1)) << 4);
    issue_w<0>(c, kb0);
    asm volatile("s_load_dwordx2 %0, %1, %2 offset:0x40" : "=&s"(outd) : "s"(ka), "s"(peer * 8));
    if (c.lmask) issue_x<2>(c, kb0 ^ 1);
    rank &= 7;
    GLB_AS unsigned* cnt = (GLB_AS unsigned*)(myws + WS_CNT + (b * NW + wid) * 128);
    unsigned seq;
    asm volatile("global_load_dword %0, %1, %2 sc1" : "=v"(seq) : "v"(0u), "s"(cnt) : "memory");
    const unsigned arm = wid * 16 + (j >> 1) * 4 + (mt >> 1);
    unsigned armer = ssel((peer ^ rank) | c.lmask, 0, (arm >> 3) ^ rank);
    unsigned kb1 = block_of<1>(c), side = (wid & c.lmask) | armer;
    asm volatile("" : "+s"(armer), "+s"(kb1), "+s"(side));
    barriers(1);
    if (side) {
        if (__builtin_expect(armer != 0, 0)) {
            touch(p.ws[(arm >> 3) & 7], WS_TMO + 256);
            GLB_AS char* fp = myout + (size_t)arm * (32 * NLOC * 2) + lane * 16;
            const u32x4 sv = {SENTINEL, SENTINEL, SENTINEL, SENTINEL};
#pragma unroll
            for (int i = 0; i < 16; ++i) store16_sys(fp + i * 1024, sv);
        } else {
            if (wid == 3) {
                touch(p.out[peer], (peer << 17) + (rank << 9));
                touch((const void*)myout, rank << 17);
            }
            barriers(wid);
        }
    }
    LDS_AS char* ring = lds + wid * WAVE_LDS;
    const unsigned q = m >> 2;
    piece<1, 4>(c, kb1); piece<1, 5>(c, kb1); SB();
    c.ax[0] = ring + m * 128 + ((g ^ (m >> 1)) << 4);
    c.ax[1] = lxor(c.ax[0], 64);
    asm volatile("" : "+v"(c.ax[0]), "+v"(c.ax[1]));
    piece<1, 6>(c, kb1); SB();
    c.aw[0] = ring + (g >> 1) * 2048 + (4 * (g & 1) + q) * 128 + 8 * (lane & 3);
    asm volatile("" : "+v"(c.aw[0]));
    piece<1, 7>(c, kb1); SB();
    c.aw[0] += (2 * (g & 1) + (q >> 1)) << 5;
#pragma unroll
    for (int t = 1; t < 4; ++t) c.aw[t] = lxor(c.aw[0], t << 5);
    asm volatile("" : "+v"(c.aw[0]), "+v"(c.aw[1]), "+v"(c.aw[2]), "+v"(c.aw[3]));
    piece<1, 8>(c, kb1); piece<1, 9>(c, kb1); piece<1, 10>(c, kb1); piece<1, 11>(c, kb1);
    issue_x<1>(c, kb1);
    if (c.lmask) {
        issue_x<3>(c, kb1 ^ 1);
        barriers(4 - wid);
    }
#pragma unroll
    for (int h = 0; h < 2; ++h)
#pragma unroll
        for (int t = 0; t < 4; ++t) c.acc[h][t] = f32x4{0.f, 0.f, 0.f, 0.f};

    if (c.lmask) wait_vm<16>();
    else wait_vm<12>();
    if (__builtin_expect(armer != 0, 0)) {
        asm volatile("s_waitcnt vmcnt(12)" : "+v"(seq) :: "memory");
        if (lane == 0) {
            __hip_atomic_store((unsigned*)(p.ws[(arm >> 3) & 7] + WS_SYNC + (rank * 8 + (arm & 7)) * 8), seq + 1,
                               __ATOMIC_RELAXED, __HIP_MEMORY_SCOPE_SYSTEM);
            __hip_atomic_store((GLB_AS unsigned*)(myws + WS_SYNC + arm * 8 + 4), seq + 1,
                               __ATOMIC_RELAXED, __HIP_MEMORY_SCOPE_SYSTEM);
        }
    }
    asm volatile("s_waitcnt lgkmcnt(0)" : "+s"(outd) :: "memory");
    step<0>(c);
    asm volatile("s_waitcnt vmcnt(12)" : "+v"(seq) :: "memory");
    seq += 1;
    step<1>(c); wait_vm<nloads(3)>();
    step<2>(c); wait_vm<nloads(4)>();
    step<3>(c); wait_vm<nloads(5)>();
    step<4>(c); wait_vm<nloads(6)>();
    {
        const unsigned kb7 = block_of<7>(c);
        piece<7, 0>(c, kb7); read_x<5, 0>(c); read_x<5, 1>(c); SB();
        piece<7, 1>(c, kb7); read_w1<5, 0, 0>(c); SB();
        piece<7, 4>(c, kb7); read_w1<5, 0, 1>(c); SB();
        piece<7, 5>(c, kb7); read_w1<5, 0, 2>(c); SB();
        piece<7, 6>(c, kb7); read_w1<5, 0, 3>(c); SB();
        piece<7, 7>(c, kb7); read_w1<5, 1, 0>(c); SB();
        piece<7, 8>(c, kb7); read_w1<5, 1, 1>(c); SB();
        piece<7, 9>(c, kb7); read_w1<5, 1, 2>(c); SB();
        piece<7, 10>(c, kb7); read_w1<5, 1, 3>(c); SB();
        piece<7, 11>(c, kb7); SB();
    }
    LDS_AS char* pw0 = ring + m * 256 + ((g ^ (m & 7)) << 4);
    LDS_AS char* pw1 = lxor(pw0, 64);
    const unsigned row = wid * 4 + g;
    LDS_AS char* pr0 = lds + row * 256 + ((m ^ (row & 7)) << 4);
    LDS_AS char* pr2 = pr0 + 2 * WAVE_LDS;
    GLB_AS char* optr = outd + (size_t)((rank * MLOC + mt * 32 + row) * NLOC + sub * 64 + m * 4) * 2;
    GLB_AS char* optr2 = optr + 16 * NLOC * 2;
    const GLB_AS char* syncp = myws + WS_SYNC + (peer * 8 + mt) * 8;
    u32x2 sy;
    __builtin_amdgcn_s_barrier();
    asm volatile("global_load_dwordx2 %0, %1, %2 sc0 sc1" : "=v"(sy) : "v"(0u), "s"(syncp) : "memory");
    mma<5, 0, 0>(c); mma<5, 0, 1>(c);
    wait_vm<nloads(7) + 1>();
    read_x<6, 0>(c); read_x<6, 1>(c); read_w1<6, 0, 0>(c); SB(); mma<5, 0, 2>(c); pseq<8, 0>(c, 0); SB();
    read_w1<6, 0, 1>(c); read_w1<6, 0, 2>(c); SB(); mma<5, 0, 3>(c); pseq<8, 1>(c, 0); SB();
    read_w1<6, 0, 3>(c); read_w1<6, 1, 0>(c); SB(); mma<5, 1, 0>(c); pseq<8, 2>(c, 0); SB();
    read_w1<6, 1, 1>(c); read_w1<6, 1, 2>(c); SB(); mma<5, 1, 1>(c); pseq<8, 3>(c, 0); SB();
    read_w1<6, 1, 3>(c); SB(); mma<5, 1, 2>(c); mma<5, 1, 3>(c); pseq<8, 4>(c, 0); SB();
    mma<6, 0, 0>(c); mma<6, 0, 1>(c); pseq<8, 5>(c, 0); SB();
    mma<6, 0, 2>(c); mma<6, 0, 3>(c); pseq<8, 6>(c, 0); pseq<8, 7>(c, 0); SB();
    asm volatile("" : "+v"(pw0), "+v"(pw1), "+v"(pr0), "+v"(pr2), "+v"(optr), "+v"(optr2));
    wait_vm<4 + 1 + NA>();
    read_x<7, 0>(c); read_w<7, 0>(c); read_x<7, 1>(c);
    SB();
    mma<6, 1, 0>(c); mma<6, 1, 1>(c); mma<6, 1, 2>(c); mma<6, 1, 3>(c);
    wait_vm<1 + NA>();
    read_w<7, 1>(c);
    SB();
    mma<7, 0, 0>(c); mma<7, 0, 1>(c); mma<7, 0, 2>(c); mma<7, 0, 3>(c);
    mma<7, 1, 0>(c); mma<7, 1, 1>(c); mma<7, 1, 2>(c);
    *(LDS_AS f32x4*)(pw0) = c.acc[0][0];
    SB();
    mma<7, 1, 3>(c);
    *(LDS_AS f32x4*)(pw1) = c.acc[0][1];
    *(LDS_AS f32x4*)(pw0 + 128) = c.acc[0][2];
    *(LDS_AS f32x4*)(pw1 + 128) = c.acc[0][3];
    SB();
    issue_v<NA, NA + NB>(c);
    lds_barrier();
    const f32x4 a0 = *(LDS_AS f32x4*)(pr0), a1 = *(LDS_AS f32x4*)(pr0 + WAVE_LDS);
    const f32x4 a2 = *(LDS_AS f32x4*)(pr2), a3 = *(LDS_AS f32x4*)(pr2 + WAVE_LDS);
    SB();
    asm volatile("s_waitcnt vmcnt(%1)" : "+v"(sy) : "i"(NA + NB - 4) : "memory");
    read_v<0>(c); read_v<1>(c);
    SB();
    issue_v<NA + NB, 2 * RET>(c);
    bool ok = true;
    if (__builtin_expect(peer != rank && __builtin_amdgcn_readfirstlane(sy[0] < seq || sy[1] < seq), 0)) {
        const unsigned long long t0 = __builtin_amdgcn_s_memrealtime();
        for (unsigned spins = 1;; ++spins) {
            asm volatile("global_load_dwordx2 %0, %1, %2 sc0 sc1\n\ts_waitcnt vmcnt(0)" : "=&v"(sy) : "v"(0u), "s"(syncp) : "memory");
            if (__builtin_amdgcn_readfirstlane(sy[0] >= seq && sy[1] >= seq)) break;
            if ((spins & 63) == 0 && __builtin_amdgcn_readfirstlane(expired(p, seq, t0))) {
                ok = false;
                optr = optr2 = myws + WS_JUNK + threadIdx.x * 8;
                break;
            }
            __builtin_amdgcn_s_sleep(1);
        }
    }
    {
        f32x4 y = a0 + a1, z = a2 + a3, e;
        PIN2(y, z);
        mv<0, 0, 0>(c);
        y += z;
        z[0] = __builtin_amdgcn_fmed3f(y[0], -60.0f, 60.0f); z[1] = __builtin_amdgcn_fmed3f(y[1], -60.0f, 60.0f);
        PIN2(y, z);
        mv<0, 0, 1>(c);
        z[2] = __builtin_amdgcn_fmed3f(y[2], -60.0f, 60.0f); z[3] = __builtin_amdgcn_fmed3f(y[3], -60.0f, 60.0f);
        z *= -1.4426950408889634f;
        PIN1(z);
        mv<0, 0, 2>(c);
        e[0] = __builtin_amdgcn_exp2f(z[0]); e[1] = __builtin_amdgcn_exp2f(z[1]); e[2] = z[2]; e[3] = z[3];
        PIN1(e);
        mv<0, 0, 3>(c);
        e[2] = __builtin_amdgcn_exp2f(e[2]); e[3] = __builtin_amdgcn_exp2f(e[3]);
        PIN1(e);
        mv<0, 1, 0>(c);
        e += 1.0f;
        e[0] = __builtin_amdgcn_rcpf(e[0]);
        PIN1(e);
        mv<0, 1, 1>(c);
        e[1] = __builtin_amdgcn_rcpf(e[1]); e[2] = __builtin_amdgcn_rcpf(e[2]);
        PIN1(e);
        mv<0, 1, 2>(c);
        e[3] = __builtin_amdgcn_rcpf(e[3]);
        e *= y;
        store8_sys(optr, u32x2{pack2(e[0], e[1]), pack2(e[2], e[3])});
        SB();
    }
    mv<0, 1, 3>(c);
    pass2<1>(c);
    *(LDS_AS f32x4*)(pw0 + UNIT) = c.acc[1][0];
    *(LDS_AS f32x4*)(pw1 + UNIT) = c.acc[1][1];
    *(LDS_AS f32x4*)(pw0 + UNIT + 128) = c.acc[1][2];
    *(LDS_AS f32x4*)(pw1 + UNIT + 128) = c.acc[1][3];
    lds_barrier();
    {
        const f32x4 b0 = *(LDS_AS f32x4*)(pr0 + UNIT), b1 = *(LDS_AS f32x4*)(pr0 + UNIT + WAVE_LDS);
        const f32x4 b2 = *(LDS_AS f32x4*)(pr2 + UNIT), b3 = *(LDS_AS f32x4*)(pr2 + UNIT + WAVE_LDS);
        SB();
        const f32x4 y = (b0 + b1) + (b2 + b3);
        f32x4 e;
#pragma unroll
        for (int i = 0; i < 4; ++i) e[i] = __builtin_amdgcn_fmed3f(y[i], -60.0f, 60.0f);
        e *= -1.4426950408889634f;
        PIN1(e);
#pragma unroll
        for (int i = 0; i < 4; ++i) e[i] = __builtin_amdgcn_exp2f(e[i]);
        PIN1(e);
        e += 1.0f;
        PIN1(e);
#pragma unroll
        for (int i = 0; i < 4; ++i) e[i] = __builtin_amdgcn_rcpf(e[i]);
        PIN1(e);
        e *= y;
        store8_sys(optr2, u32x2{pack2(e[0], e[1]), pack2(e[2], e[3])});
    }
    if (lane == 0) __hip_atomic_store(cnt, seq, __ATOMIC_RELAXED, __HIP_MEMORY_SCOPE_AGENT);
    if (peer != rank && ok) {
        const GLB_AS char* mine = myout + (size_t)((peer * MLOC + mt * 32 + wid * 8 + (lane >> 3)) * NLOC + sub * 64 + (lane & 7) * 8) * 2;
        if (__builtin_expect(!poll_arrived(mine), 0)) {
            const unsigned long long t0 = __builtin_amdgcn_s_memrealtime();
            while (!poll_arrived(mine))
                if (__builtin_amdgcn_readfirstlane(expired(p, seq, t0))) break;
        }
    }
}

extern "C" void kernel_launch(int rank, int nranks, ncclComm_t comm,
                              hipStream_t stream,
                              void* const* d_in, const int* in_sizes, int n_in,
                              void* d_out, int out_size,
                              const int* in_dtypes, int out_dtype,
                              void* const* d_out_all,
                              void* const* d_ws_all, size_t ws_bytes) {
    if (nranks != NRANK || (unsigned)rank >= (unsigned)NRANK || n_in < 2 || in_sizes[0] != MLOC * KDIM
        || in_sizes[1] != KDIM * NDIM || out_size != NRANK * MLOC * NLOC || ws_bytes < (size_t)WS_END) {
        fprintf(stderr, "kernel_launch: unexpected problem shape\n");
        return;
    }
    Params p;
    const uintptr_t ax = (uintptr_t)d_in[0], aw = (uintptr_t)d_in[1];
    p.srdx[0] = (unsigned)ax; p.srdx[1] = (unsigned)(ax >> 32) & 0xffffu; p.srdx[2] = MLOC * KDIM * 2; p.srdx[3] = 0x00020000u;
    p.srdw[0] = (unsigned)aw; p.srdw[1] = (unsigned)(aw >> 32) & 0xffffu; p.srdw[2] = KDIM * NDIM * 2; p.srdw[3] = 0x00020000u;
    p.myws = (char*)d_ws_all[rank];
    p.myout = (char*)d_out;
    p.rank = (unsigned)rank;
    p.pad[0] = p.pad[1] = p.pad[2] = 0;
    for (int i = 0; i < NRANK; ++i) {
        p.out[i] = (char*)d_out_all[i];
        p.ws[i] = (char*)d_ws_all[i];
    }
    gemm_a2a<<<dim3(NWG), dim3(NW * 64), 0, stream>>>(p);
    HIP_CHECK(hipGetLastError());
}
```
